```python
import numpy as np

kernel_name = "gpu_gemm_rs_m4096_k4096_n4096_e5m2e5m2_dqsilu"


ABSMAX_RTOL = 0.05

COMPUTE_DTYPE = "float32"
EPILOGUE = "dequant_silu"
IN_QUANT = None
OUT_QUANT = None
IN_FP8 = "e5m2e5m2"
IN_DTYPES = {"x": "e5m2", "w_mat": "e5m2",
             "scale_x": "float32", "scale_w": "float32"}
WORLD_SIZE = 8

def _snap_fp8(x, mant_bits, max_abs, min_normal):
    a = np.abs(x.astype(np.float64))
    _m, _e = np.frexp(a)
    q = np.where(a >= min_normal, np.ldexp(1.0, _e - 1 - mant_bits),
                 min_normal * 2.0 ** -mant_bits)
    snapped = np.minimum(np.round(a / q) * q, max_abs)
    return (np.sign(x) * snapped).astype(np.float32)

def setup_inputs():

    x = _snap_fp8(np.random.randn(4096, 4096).astype(np.float32), 2, 57344.0, 2.0 ** -14)
    w_mat = _snap_fp8(np.random.randn(4096, 4096).astype(np.float32), 2, 57344.0, 2.0 ** -14)
    scale_x = (0.02 * (1.0 + np.random.rand(1))).astype(np.float32)
    scale_w = (0.02 * (1.0 + np.random.rand(1))).astype(np.float32)
    return {"x": x, "w_mat": w_mat, "scale_x": scale_x, "scale_w": scale_w}

def reference(x, w_mat, scale_x, scale_w):
    acc = (x @ w_mat).astype(np.float32)
    y = (acc * (scale_x * scale_w)).astype(np.float32)
    return (y / (1.0 + np.exp(-np.clip(y, -60.0, 60.0)))).astype(np.float32)
```

```cpp
#include <hip/hip_runtime.h>
#include <rccl/rccl.h>
#include <cstdio>
#include <cstdint>

#define HIP_CHECK(expr) do { hipError_t _e = (expr); if (_e != hipSuccess) { \
    fprintf(stderr, "HIP error %s at %s:%d\n", hipGetErrorString(_e), __FILE__, __LINE__); \
    return; } } while (0)

typedef unsigned int u32;
typedef unsigned long long u64;
typedef u32   u32x4  __attribute__((ext_vector_type(4)));
typedef int   i32x4  __attribute__((ext_vector_type(4)));
typedef int   i32x8  __attribute__((ext_vector_type(8)));
typedef float f32x4  __attribute__((ext_vector_type(4)));
typedef float f32x16 __attribute__((ext_vector_type(16)));

#define PF_WIN 4
#define C_PAIRS 2
#define NBUF 2
#define OUT_AUX 0
#define DO_INV 1

constexpr int NR   = 8;
constexpr int NPF  = 16;
constexpr int NGW  = 256;
constexpr int NDIM = 4096;
constexpr int KS   = 512;
constexpr u64 DEADLINE = 100000000ull;

constexpr u32 OFF_CTR   = 0;
constexpr u32 OFF_TMO   = 32768;
constexpr u32 OFF_ELECT = 32768 + 1024;
constexpr u32 OFF_INVD  = 32768 + 4096;
constexpr u32 OFF_WFLAG = 65536;
constexpr u32 OFF_XFLAG = 65536 + 16384;
constexpr u32 OFF_STAMP = 131072;
constexpr u32 CTRL_BYTES = 1u << 20;
constexpr u32 WST_BYTES = 2u << 20;
constexpr u32 XB_BYTES  = 2u << 20;
constexpr u32 BUF_BYTES = WST_BYTES + XB_BYTES;
constexpr u32 WS_NEED   = CTRL_BYTES + NBUF * BUF_BYTES;

struct Args {
    const unsigned char* x;
    const unsigned char* w;
    const float* sx;
    const float* sw;
    float* out;
    unsigned char* ws[NR];
    int rank;
};

#define RSRC(p, bytes) __builtin_amdgcn_make_buffer_rsrc((void*)(p), 0, (int)(bytes), 0x00020000)
#define AUX_SYS 17
#define AUX_AGT 16

__device__ __forceinline__ u32 ld_sys(const unsigned char* p) {
    return __hip_atomic_load((const u32*)p, __ATOMIC_RELAXED, __HIP_MEMORY_SCOPE_SYSTEM);
}
__device__ __forceinline__ void st_sys(unsigned char* p, u32 v) {
    __hip_atomic_store((u32*)p, v, __ATOMIC_RELAXED, __HIP_MEMORY_SCOPE_SYSTEM);
}
__device__ __forceinline__ u32 xcc_id() {
    return (u32)__builtin_amdgcn_s_getreg((3 << 11) | 20) & 0xFu;
}
__device__ __forceinline__ void wait_vm0() { asm volatile("s_waitcnt vmcnt(0)" ::: "memory"); }
__device__ __forceinline__ void wg_barrier() {
    asm volatile("s_waitcnt lgkmcnt(0)" ::: "memory");
    __builtin_amdgcn_s_barrier();
    asm volatile("" ::: "memory");
}

#define STAMP(i) do {} while (0)

__device__ __forceinline__ bool poll_flags(const unsigned char* p0, bool use0,
                                           const unsigned char* p1, bool use1,
                                           unsigned char* tmo, u32 call) {
    const u64 t0 = __builtin_amdgcn_s_memrealtime();
    for (;;) {
        u32 v0 = ld_sys(p0);
        u32 v1 = ld_sys(p1);
        bool ok = (!use0 || v0 >= call) && (!use1 || v1 >= call);
        if (__all(ok)) return true;
        u32 t = __hip_atomic_load((u32*)tmo, __ATOMIC_RELAXED, __HIP_MEMORY_SCOPE_AGENT);
        if (t == call || __builtin_amdgcn_s_memrealtime() - t0 > DEADLINE) {
            __hip_atomic_store((u32*)tmo, call, __ATOMIC_RELAXED, __HIP_MEMORY_SCOPE_AGENT);
            return false;
        }
        __builtin_amdgcn_s_sleep(4);
    }
}

__device__ __forceinline__ void elect_invalidate(unsigned char* myws, u32 call, u32 xcc, int lane) {
    u32 old = 0;
    if (lane == 0)
        old = __hip_atomic_fetch_max((u32*)(myws + OFF_ELECT + xcc * 128), call,
                                     __ATOMIC_RELAXED, __HIP_MEMORY_SCOPE_AGENT);
    old = __builtin_amdgcn_readfirstlane(old);
    if (old < call) {
        __builtin_amdgcn_fence(__ATOMIC_ACQUIRE, "");
        wait_vm0();
        if (lane == 0) st_sys(myws + OFF_INVD + xcc * 128, call);
    }
}

__global__ void __launch_bounds__(512, 4) fused_gemm_rs(Args a) {
    __shared__ __attribute__((aligned(16))) float red[2 * 8 * 1024 + 4];
    const int lane = threadIdx.x & 63;
    const int wv = __builtin_amdgcn_readfirstlane(threadIdx.x >> 6);
    const int bid = blockIdx.x;
    const int rank = a.rank;
    unsigned char* myws = a.ws[rank];
    STAMP(0);

    unsigned char* ctr = myws + OFF_CTR + bid * 64;
    const u32 xcc = xcc_id();

    if (bid < NPF) {
        const u32 call = __builtin_amdgcn_readfirstlane(ld_sys(ctr)) + 1u;
        wait_vm0();
        wg_barrier();
        const int stream = (bid & 7) * 2 + (bid >> 3);
        if (wv == 7) {
            if (lane == 0) st_sys(ctr, call);
            elect_invalidate(myws, call, xcc, lane);
            return;
        }
        const int src = (rank + 1 + wv) & 7;
        const unsigned char* f0 = myws + OFF_WFLAG + (src * 16 + stream) * 128 + (lane & 7) * 4;
        const unsigned char* f1 = myws + OFF_INVD + xcc * 128;
        if (!poll_flags(f0, true, f1, DO_INV != 0, myws + OFF_TMO, call)) return;
        STAMP(1);
        const unsigned char* base = a.ws[src] + CTRL_BYTES + (call % NBUF) * BUF_BYTES + stream * 131072;
        __amdgpu_buffer_rsrc_t rs = RSRC(base, 131072);
        u32x4 v[PF_WIN];
        const u32 lo = lane * 16;
#pragma unroll
        for (int k = 0; k < PF_WIN; ++k) v[k] = __builtin_amdgcn_raw_buffer_load_b128(rs, lo + k * 1024, 0, 0);
        for (int n = PF_WIN; n < 128 + PF_WIN; n += PF_WIN) {
#pragma unroll
            for (int k = 0; k < PF_WIN; ++k) {
                asm volatile("" :: "v"(v[k]));
                v[k] = __builtin_amdgcn_raw_buffer_load_b128(rs, lo + (n + k) * 1024, 0, 0);
                __builtin_amdgcn_sched_barrier(0);
            }
        }
        STAMP(2);
        return;
    }

    const int gb = bid - NPF;
    const int stream = (gb & 7) * 2 + ((gb >> 3) & 1);
    const int slab = gb >> 4;

    u32 call;
    if (wv < 4) {
        if (gb < 128) {
            const int T = ((gb >> 3) & 3) + 4 * ((gb & 7) + 8 * (gb >> 5));
            const int cq = lane & 7, kg = lane >> 3;
            __amdgpu_buffer_rsrc_t rw = RSRC(a.w, KS * NDIM);
            const u32 o = (u32)(wv * 128 + kg * 16) * NDIM + T * 32 + cq * 4;
            u32 d[16];
#pragma unroll
            for (int j = 0; j < 16; ++j) d[j] = __builtin_amdgcn_raw_buffer_load_b32(rw, o + j * NDIM, 0, 0);
            call = __builtin_amdgcn_readfirstlane(ld_sys(ctr)) + 1u;
            u32x4 q[4];
#pragma unroll
            for (int u = 0; u < 4; ++u) {
                u32 t0 = __builtin_amdgcn_perm(d[4 * u + 1], d[4 * u + 0], 0x05010400u);
                u32 t1 = __builtin_amdgcn_perm(d[4 * u + 1], d[4 * u + 0], 0x07030602u);
                u32 t2 = __builtin_amdgcn_perm(d[4 * u + 3], d[4 * u + 2], 0x05010400u);
                u32 t3 = __builtin_amdgcn_perm(d[4 * u + 3], d[4 * u + 2], 0x07030602u);
                q[0][u] = __builtin_amdgcn_perm(t2, t0, 0x05040100u);
                q[1][u] = __builtin_amdgcn_perm(t2, t0, 0x07060302u);
                q[2][u] = __builtin_amdgcn_perm(t3, t1, 0x05040100u);
                q[3][u] = __builtin_amdgcn_perm(t3, t1, 0x07060302u);
            }
            __amdgpu_buffer_rsrc_t rd = RSRC(myws + CTRL_BYTES + (call % NBUF) * BUF_BYTES + T * 16384, 16384);
            const int G = wv * 8 + kg;
            const u32 so = (u32)(G >> 1) * 1024 + ((G & 1) * 32 + cq) * 16;
#pragma unroll
            for (int c = 0; c < 4; ++c) __builtin_amdgcn_raw_buffer_store_b128(q[c], rd, so + c * 128, 0, AUX_SYS);
        } else {
            const int xb = gb - 128;
            const int dst = (rank + 1 + (xb >> 4)) & 7;
            const int sl = xb & 15;
            __amdgpu_buffer_rsrc_t rx = RSRC(a.x + (size_t)(dst * 512 + sl * 32) * KS, 32 * KS);
            const u32 o = (u32)(lane & 31) * KS + (lane >> 5) * 16 + wv * 128;
            u32x4 v[4];
#pragma unroll
            for (int c = 0; c < 4; ++c) v[c] = __builtin_amdgcn_raw_buffer_load_b128(rx, o + c * 32, 0, 0);
            call = __builtin_amdgcn_readfirstlane(ld_sys(ctr)) + 1u;
            __amdgpu_buffer_rsrc_t rd = RSRC(a.ws[dst] + CTRL_BYTES + (call % NBUF) * BUF_BYTES + WST_BYTES
                                             + (rank * 16 + sl) * 16384, 16384);
#pragma unroll
            for (int c = 0; c < 4; ++c)
                __builtin_amdgcn_raw_buffer_store_b128(v[c], rd, (wv * 4 + c) * 1024 + lane * 16, 0, AUX_SYS);
        }
    } else {
        call = __builtin_amdgcn_readfirstlane(ld_sys(ctr)) + 1u;
        if (wv == 7) elect_invalidate(myws, call, xcc, lane);
    }
    const float scale = a.sx[0] * a.sw[0];
    STAMP(1);
    wait_vm0();
    wg_barrier();
    STAMP(2);
    int* okw = (int*)&red[2 * 8 * 1024];
    if (wv == 0) {
        if (gb < 128) {
            const int T = ((gb >> 3) & 3) + 4 * ((gb & 7) + 8 * (gb >> 5));
            if (lane < NR) st_sys(a.ws[lane] + OFF_WFLAG + (rank * 16 + (T >> 3)) * 128 + (T & 7) * 4, call);
        } else {
            const int xb = gb - 128;
            const int dst = (rank + 1 + (xb >> 4)) & 7;
            if (lane == 0) st_sys(a.ws[dst] + OFF_XFLAG + (xb & 15) * 128 + rank * 4, call);
        }
        if (lane == 0) st_sys(ctr, call);
    } else if (wv == 7) {
        const unsigned char* f0 = myws + OFF_WFLAG + ((lane >> 3) * 16 + stream) * 128 + (lane & 7) * 4;
        const unsigned char* f1 = (lane < 8) ? (myws + OFF_XFLAG + slab * 128 + lane * 4)
                                             : (myws + OFF_INVD + xcc * 128);
        bool ok = poll_flags(f0, true, f1, lane < 8 || DO_INV != 0, myws + OFF_TMO, call);
        if (lane == 0) *okw = ok ? 1 : 0;
    }
    wg_barrier();
    STAMP(3);
    const int ok = __builtin_amdgcn_readfirstlane(*okw);

    float* outp = a.out + (size_t)(slab * 32) * NDIM + stream * 256;
    __amdgpu_buffer_rsrc_t ro = RSRC(outp, 31 * NDIM * 4 + 1024);
    if (!ok) {
        const float qnan = __builtin_nanf("");
        const f32x4 nn = {qnan, qnan, qnan, qnan};
#pragma unroll
        for (int r = 0; r < 4; ++r)
            __builtin_amdgcn_raw_buffer_store_b128(__builtin_bit_cast(u32x4, nn), ro,
                                                   (u32)(wv * 4 + r) * NDIM * 4 + lane * 16, 0, 0);
        return;
    }

    const u32 bufo = CTRL_BYTES + (call % NBUF) * BUF_BYTES;
    __amdgpu_buffer_rsrc_t rw = RSRC(a.ws[wv] + bufo + stream * 131072, 131072);
    __amdgpu_buffer_rsrc_t rx = RSRC(myws + bufo + WST_BYTES + (wv * 16 + slab) * 16384, 16384);
    const u32 lo = lane * 16;

    u32x4 wf[2 * C_PAIRS];
#pragma unroll
    for (int k = 0; k < 2 * C_PAIRS; ++k) wf[k] = __builtin_amdgcn_raw_buffer_load_b128(rw, lo + k * 1024, 0, 0);
    u32x4 xf[16];
#pragma unroll
    for (int i = 0; i < 16; ++i) xf[i] = __builtin_amdgcn_raw_buffer_load_b128(rx, lo + i * 1024, 0, AUX_AGT);

    const int m = lane & 31, hh = lane >> 5;
    const u32 wr_base = wv * 4096 + m * 128;
    const int rrow = wv * 4 + ((lane >> 3) & 3);
    const int re = lane & 7;
    const u32 rd_base = hh * 4 * 4096 + rrow * 128 + ((re ^ (rrow & 7)) * 16);
    const u32 out_off = (u32)rrow * NDIM * 4 + re * 16;
    const int one = 0x7F7F7F7F;

    static_assert(8 % C_PAIRS == 0, "pairs must divide the MFMAs of a tile");
#pragma unroll 1
    for (int j = 0; j < 8; ++j) {
        f32x16 acc = {0.f, 0.f, 0.f, 0.f, 0.f, 0.f, 0.f, 0.f, 0.f, 0.f, 0.f, 0.f, 0.f, 0.f, 0.f, 0.f};
#pragma unroll
        for (int t = 0; t < 8; ++t) {
            const int s = (t % C_PAIRS) * 2;
            i32x8 A = __builtin_shufflevector(__builtin_bit_cast(i32x4, wf[s]), __builtin_bit_cast(i32x4, wf[s + 1]),
                                              0, 1, 2, 3, 4, 5, 6, 7);
            i32x8 B = __builtin_shufflevector(__builtin_bit_cast(i32x4, xf[2 * t]), __builtin_bit_cast(i32x4, xf[2 * t + 1]),
                                              0, 1, 2, 3, 4, 5, 6, 7);
            acc = __builtin_amdgcn_mfma_scale_f32_32x32x64_f8f6f4(A, B, acc, 1, 1, 0, one, 0, one);
            __builtin_amdgcn_sched_barrier(0);
            const u32 n = (u32)(j * 16 + 2 * t + 2 * C_PAIRS) * 1024 + lo;
            wf[s]     = __builtin_amdgcn_raw_buffer_load_b128(rw, n, 0, 0);
            wf[s + 1] = __builtin_amdgcn_raw_buffer_load_b128(rw, n + 1024, 0, 0);
            __builtin_amdgcn_sched_barrier(0);
        }
        char* slot = (char*)red + (j & 1) * 32768;
#pragma unroll
        for (int b = 0; b < 4; ++b) {
            f32x4 p = {acc[b], acc[4 + b], acc[8 + b], acc[12 + b]};
            *(f32x4*)(slot + wr_base + (((hh * 4 + b) ^ (m & 7)) * 16)) = p;
        }
        wg_barrier();
        f32x4 s0 = *(const f32x4*)(slot + rd_base);
        f32x4 s1 = *(const f32x4*)(slot + rd_base + 4096);
        f32x4 s2 = *(const f32x4*)(slot + rd_base + 8192);
        f32x4 s3 = *(const f32x4*)(slot + rd_base + 12288);
        f32x4 s = (s0 + s1) + (s2 + s3);
        f32x4 r;
#pragma unroll
        for (int c = 0; c < 4; ++c) {
            float tot = s[c] + __shfl_xor(s[c], 32);
            float y = tot * scale;
            float z = fminf(fmaxf(y, -60.f), 60.f);
            r[c] = y / (1.f + __expf(-z));
        }
        if (lane < 32)
            __builtin_amdgcn_raw_buffer_store_b128(__builtin_bit_cast(u32x4, r), ro, out_off + j * 128, 0, OUT_AUX);
    }
    STAMP(7);
}

extern "C" void kernel_launch(int rank, int nranks, ncclComm_t comm,
                              hipStream_t stream,
                              void* const* d_in, const int* in_sizes, int n_in,
                              void* d_out, int out_size,
                              const int* in_dtypes, int out_dtype,
                              void* const* d_out_all,
                              void* const* d_ws_all, size_t ws_bytes) {
    (void)comm; (void)in_sizes; (void)in_dtypes; (void)out_dtype; (void)d_out_all;
    if (nranks != NR || n_in < 4 || (unsigned)rank >= (unsigned)NR || ws_bytes < WS_NEED ||
        out_size != 512 * NDIM) {
        fprintf(stderr, "kernel_launch: unexpected problem shape\n");
        return;
    }
    Args a;
    a.x = (const unsigned char*)d_in[0];
    a.w = (const unsigned char*)d_in[1];
    a.sx = (const float*)d_in[2];
    a.sw = (const float*)d_in[3];
    a.out = (float*)d_out;
    for (int p = 0; p < NR; ++p) a.ws[p] = (unsigned char*)d_ws_all[p];
    a.rank = rank;
    hipLaunchKernelGGL(fused_gemm_rs, dim3(NPF + NGW), dim3(512), 0, stream, a);
    HIP_CHECK(hipGetLastError());
}
```

```cpp
#include <hip/hip_runtime.h>
#include <rccl/rccl.h>
#include <cstdio>
#include <cstdint>

#define HIP_CHECK(expr) do { hipError_t _e = (expr); if (_e != hipSuccess) { \
    fprintf(stderr, "HIP error %s at %s:%d\n", hipGetErrorString(_e), __FILE__, __LINE__); \
    return; } } while (0)

typedef unsigned int u32;
typedef unsigned long long u64;
typedef u32   u32x4  __attribute__((ext_vector_type(4)));
typedef int   i32x4  __attribute__((ext_vector_type(4)));
typedef int   i32x8  __attribute__((ext_vector_type(8)));
typedef float f32x4  __attribute__((ext_vector_type(4)));
typedef float f32x16 __attribute__((ext_vector_type(16)));

#define PF_WIN 4
#define C_PAIRS 2
#define NBUF 2
#define X_WIN 1
#define OUT_AUX 16
#define DO_INV 1

constexpr int NR   = 8;
constexpr int NWG  = 256;
constexpr int NXP  = 14;
constexpr int NGM  = 16;
constexpr int NDIM = 4096;
constexpr int KS   = 512;
constexpr u64 DEADLINE = 100000000ull;

constexpr u32 OFF_CTR   = 0;
constexpr u32 OFF_TMO   = 32768;
constexpr u32 OFF_ELECT = 32768 + 1024;
constexpr u32 OFF_INVD  = 32768 + 4096;
constexpr u32 OFF_WFLAG = 65536;
constexpr u32 OFF_XFLAG = 65536 + 16384;
constexpr u32 OFF_STAMP = 131072;
constexpr u32 CTRL_BYTES = 1u << 20;
constexpr u32 WST_BYTES = 2u << 20;
constexpr u32 XB_BYTES  = 2u << 20;
constexpr u32 BUF_BYTES = WST_BYTES + XB_BYTES;
constexpr u32 WS_NEED   = CTRL_BYTES + NBUF * BUF_BYTES;

struct Args {
    const unsigned char* x;
    const unsigned char* w;
    const float* sx;
    const float* sw;
    float* out;
    unsigned char* ws[NR];
    int rank;
};

#define RSRC(p, bytes) __builtin_amdgcn_make_buffer_rsrc((void*)(p), 0, (int)(bytes), 0x00020000)
#define AUX_SYS 17
#define AUX_AGT 16

__device__ __forceinline__ u32 ld_sys(const unsigned char* p) {
    return __hip_atomic_load((const u32*)p, __ATOMIC_RELAXED, __HIP_MEMORY_SCOPE_SYSTEM);
}
__device__ __forceinline__ void st_sys(unsigned char* p, u32 v) {
    __hip_atomic_store((u32*)p, v, __ATOMIC_RELAXED, __HIP_MEMORY_SCOPE_SYSTEM);
}
__device__ __forceinline__ u32 xcc_id() {
    return (u32)__builtin_amdgcn_s_getreg((3 << 11) | 20) & 0xFu;
}
__device__ __forceinline__ void wait_vm0() { asm volatile("s_waitcnt vmcnt(0)" ::: "memory"); }
__device__ __forceinline__ void wg_barrier() {
    asm volatile("s_waitcnt lgkmcnt(0)" ::: "memory");
    __builtin_amdgcn_s_barrier();
    asm volatile("" ::: "memory");
}

#define STAMP(i) do {} while (0)

__device__ __forceinline__ bool poll_flags(const unsigned char* p0, bool use0,
                                           const unsigned char* p1, bool use1,
                                           unsigned char* tmo, u32 call) {
    const u64 t0 = __builtin_amdgcn_s_memrealtime();
    for (;;) {
        u32 v0 = ld_sys(p0);
        u32 v1 = ld_sys(p1);
        bool ok = (!use0 || v0 >= call) && (!use1 || v1 >= call);
        if (__all(ok)) return true;
        u32 t = __hip_atomic_load((u32*)tmo, __ATOMIC_RELAXED, __HIP_MEMORY_SCOPE_AGENT);
        if (t == call || __builtin_amdgcn_s_memrealtime() - t0 > DEADLINE) {
            __hip_atomic_store((u32*)tmo, call, __ATOMIC_RELAXED, __HIP_MEMORY_SCOPE_AGENT);
            return false;
        }
        __builtin_amdgcn_s_sleep(4);
    }
}

__device__ __forceinline__ void elect_invalidate(unsigned char* myws, u32 call, u32 xcc, int lane) {
    u32 old = 0;
    if (lane == 0)
        old = __hip_atomic_fetch_max((u32*)(myws + OFF_ELECT + xcc * 128), call,
                                     __ATOMIC_RELAXED, __HIP_MEMORY_SCOPE_AGENT);
    old = __builtin_amdgcn_readfirstlane(old);
    if (old < call) {
        __builtin_amdgcn_fence(__ATOMIC_ACQUIRE, "");
        wait_vm0();
        if (lane == 0) st_sys(myws + OFF_INVD + xcc * 128, call);
    }
}

__global__ void __launch_bounds__(512, 2) fused_gemm_rs(Args a) {
    __shared__ __attribute__((aligned(16))) float red[2 * 8 * 2048 + 4];
    const int lane = threadIdx.x & 63;
    const int wv = __builtin_amdgcn_readfirstlane(threadIdx.x >> 6);
    const int bid = blockIdx.x;
    const int xg = bid & 7;
    const int idx = bid >> 3;
    const int rank = a.rank;
    unsigned char* myws = a.ws[rank];
    STAMP(0);

    unsigned char* ctr = myws + OFF_CTR + bid * 64;

    if (idx < NXP || idx >= NXP + NGM) {
        const bool pf = idx >= NXP + NGM;
        const int u = pf ? 112 + (idx - NXP - NGM) * 8 + xg : idx * 8 + xg;
        const int dst = (rank + 1 + (u >> 4)) & 7;
        const int sl = u & 15;
        u32 call;
        if (wv < 4) {
            __amdgpu_buffer_rsrc_t rx = RSRC(a.x + (size_t)(dst * 512 + sl * 32) * KS, 32 * KS);
            const u32 o = (u32)(lane & 31) * KS + (lane >> 5) * 16 + wv * 128;
            u32x4 v[4];
#pragma unroll
            for (int c = 0; c < 4; ++c) v[c] = __builtin_amdgcn_raw_buffer_load_b128(rx, o + c * 32, 0, 0);
            call = __builtin_amdgcn_readfirstlane(ld_sys(ctr)) + 1u;
            __amdgpu_buffer_rsrc_t rd = RSRC(a.ws[dst] + CTRL_BYTES + (call % NBUF) * BUF_BYTES + WST_BYTES
                                             + (rank * 16 + sl) * 16384, 16384);
            STAMP(1);
#pragma unroll
            for (int c = 0; c < 4; ++c) {
                if (c >= X_WIN) asm volatile("s_waitcnt vmcnt(%0)" :: "i"(X_WIN - 1) : "memory");
                __builtin_amdgcn_raw_buffer_store_b128(v[c], rd, (wv * 4 + c) * 1024 + lane * 16, 0, AUX_SYS);
            }
        } else {
            call = __builtin_amdgcn_readfirstlane(ld_sys(ctr)) + 1u;
            if (pf && wv == 7) elect_invalidate(myws, call, xcc_id(), lane);
            STAMP(1);
        }
        wait_vm0();
        wg_barrier();
        STAMP(2);
        if (wv == 7) {
            if (lane == 0) {
                st_sys(a.ws[dst] + OFF_XFLAG + (sl >> 1) * 128 + (sl & 1) * 32 + rank * 4, call);
                st_sys(ctr, call);
            }
            return;
        }
        if (!pf) return;

        const int stream = xg * 2 + (idx - NXP - NGM);
        const int src = (rank + 1 + wv) & 7;
        const unsigned char* f0 = myws + OFF_WFLAG + (src * 16 + stream) * 128 + (lane & 7) * 4;
        const unsigned char* f1 = myws + OFF_INVD + xcc_id() * 128;
        if (!poll_flags(f0, true, f1, DO_INV != 0, myws + OFF_TMO, call)) return;
        STAMP(3);
        const unsigned char* base = a.ws[src] + CTRL_BYTES + (call % NBUF) * BUF_BYTES + stream * 131072;
        __amdgpu_buffer_rsrc_t rs = RSRC(base, 131072);
        u32x4 v[PF_WIN];
        const u32 lo = lane * 16;
#pragma unroll
        for (int k = 0; k < PF_WIN; ++k) v[k] = __builtin_amdgcn_raw_buffer_load_b128(rs, lo + k * 1024, 0, 0);
        for (int n = PF_WIN; n < 128 + PF_WIN; n += PF_WIN) {
#pragma unroll
            for (int k = 0; k < PF_WIN; ++k) {
                asm volatile("" :: "v"(v[k]));
                v[k] = __builtin_amdgcn_raw_buffer_load_b128(rs, lo + (n + k) * 1024, 0, 0);
                __builtin_amdgcn_sched_barrier(0);
            }
        }
        STAMP(4);
        return;
    }

    const int gi = idx - NXP;
    const int stream = xg * 2 + (gi & 1);
    const int slab = gi >> 1;
    const int T = (gi & 3) + 4 * (xg + 8 * (gi >> 2));
    const u32 xcc = xcc_id();

    u32 call;
    if (wv < 4) {
        const int cq = lane & 7, kg = lane >> 3;
        __amdgpu_buffer_rsrc_t rw = RSRC(a.w, KS * NDIM);
        const u32 o = (u32)(wv * 128 + kg * 16) * NDIM + T * 32 + cq * 4;
        u32 d[16];
#pragma unroll
        for (int j = 0; j < 16; ++j) d[j] = __builtin_amdgcn_raw_buffer_load_b32(rw, o + j * NDIM, 0, 0);
        call = __builtin_amdgcn_readfirstlane(ld_sys(ctr)) + 1u;
        u32x4 q[4];
#pragma unroll
        for (int u = 0; u < 4; ++u) {
            u32 t0 = __builtin_amdgcn_perm(d[4 * u + 1], d[4 * u + 0], 0x05010400u);
            u32 t1 = __builtin_amdgcn_perm(d[4 * u + 1], d[4 * u + 0], 0x07030602u);
            u32 t2 = __builtin_amdgcn_perm(d[4 * u + 3], d[4 * u + 2], 0x05010400u);
            u32 t3 = __builtin_amdgcn_perm(d[4 * u + 3], d[4 * u + 2], 0x07030602u);
            q[0][u] = __builtin_amdgcn_perm(t2, t0, 0x05040100u);
            q[1][u] = __builtin_amdgcn_perm(t2, t0, 0x07060302u);
            q[2][u] = __builtin_amdgcn_perm(t3, t1, 0x05040100u);
            q[3][u] = __builtin_amdgcn_perm(t3, t1, 0x07060302u);
        }
        __amdgpu_buffer_rsrc_t rd = RSRC(myws + CTRL_BYTES + (call % NBUF) * BUF_BYTES + T * 16384, 16384);
        const int G = wv * 8 + kg;
        const u32 so = (u32)(G >> 1) * 1024 + ((G & 1) * 32 + cq) * 16;
#pragma unroll
        for (int c = 0; c < 4; ++c) __builtin_amdgcn_raw_buffer_store_b128(q[c], rd, so + c * 128, 0, AUX_SYS);
    } else {
        call = __builtin_amdgcn_readfirstlane(ld_sys(ctr)) + 1u;
        if (wv == 7) elect_invalidate(myws, call, xcc, lane);
    }
    const float scale = a.sx[0] * a.sw[0];
    STAMP(1);
    wait_vm0();
    wg_barrier();
    STAMP(2);
    int* okw = (int*)&red[2 * 8 * 2048];
    if (wv == 0) {
        if (lane < NR) st_sys(a.ws[lane] + OFF_WFLAG + (rank * 16 + (T >> 3)) * 128 + (T & 7) * 4, call);
        if (lane == 0) st_sys(ctr, call);
    } else if (wv == 7) {
        const unsigned char* f0 = myws + OFF_WFLAG + ((lane >> 3) * 16 + stream) * 128 + (lane & 7) * 4;
        const unsigned char* f1 = (lane < 16) ? (myws + OFF_XFLAG + slab * 128 + lane * 4)
                                              : (myws + OFF_INVD + xcc * 128);
        bool ok = poll_flags(f0, true, f1, lane < 16 || DO_INV != 0, myws + OFF_TMO, call);
        if (lane == 0) *okw = ok ? 1 : 0;
    }
    wg_barrier();
    STAMP(3);
    const int ok = __builtin_amdgcn_readfirstlane(*okw);

    float* outp = a.out + (size_t)(slab * 64) * NDIM + stream * 256;
    __amdgpu_buffer_rsrc_t ro = RSRC(outp, 63 * NDIM * 4 + 1024);
    if (!ok) {
        const float qnan = __builtin_nanf("");
        const f32x4 nn = {qnan, qnan, qnan, qnan};
#pragma unroll
        for (int r = 0; r < 8; ++r)
            __builtin_amdgcn_raw_buffer_store_b128(__builtin_bit_cast(u32x4, nn), ro,
                                                   (u32)(wv * 8 + r) * NDIM * 4 + lane * 16, 0, 0);
        return;
    }

    const u32 bufo = CTRL_BYTES + (call % NBUF) * BUF_BYTES;
    __amdgpu_buffer_rsrc_t rw = RSRC(a.ws[wv] + bufo + stream * 131072, 131072);
    __amdgpu_buffer_rsrc_t rx = RSRC(myws + bufo + WST_BYTES + (wv * 16 + slab * 2) * 16384, 32768);
    const u32 lo = lane * 16;

    u32x4 wf[8];
#pragma unroll
    for (int k = 0; k < 2 * C_PAIRS; ++k) wf[k] = __builtin_amdgcn_raw_buffer_load_b128(rw, lo + k * 1024, 0, 0);
    u32x4 xf[32];
#pragma unroll
    for (int i = 0; i < 32; ++i) xf[i] = __builtin_amdgcn_raw_buffer_load_b128(rx, lo + i * 1024, 0, AUX_AGT);

    const int m = lane & 31, hh = lane >> 5;
    const u32 wr_base = wv * 8192 + m * 128;
    const int rrow = wv * 8 + (lane >> 3);
    const int re = lane & 7;
    const u32 rd_base = rrow * 128 + ((re ^ (rrow & 7)) * 16);
    const u32 out_off = (u32)rrow * NDIM * 4 + re * 16;
    const int one = 0x7F7F7F7F;

    static_assert(C_PAIRS >= 1 && C_PAIRS <= 3, "4 operand buffers");
#define CAT8(lo4, hi4) __builtin_shufflevector(__builtin_bit_cast(i32x4, lo4), __builtin_bit_cast(i32x4, hi4), 0, 1, 2, 3, 4, 5, 6, 7)
#pragma unroll 1
    for (int j = 0; j < 8; ++j) {
        f32x16 acc0 = {0.f, 0.f, 0.f, 0.f, 0.f, 0.f, 0.f, 0.f, 0.f, 0.f, 0.f, 0.f, 0.f, 0.f, 0.f, 0.f};
        f32x16 acc1 = acc0;
#pragma unroll
        for (int t = 0; t < 8; ++t) {
            const int s = (t & 3) * 2, sn = ((t + C_PAIRS) & 3) * 2;
            asm volatile("" :: "v"(wf[s]), "v"(wf[s + 1]) : "memory");
            const u32 n = (u32)(j * 16 + 2 * t + 2 * C_PAIRS) * 1024 + lo;
            wf[sn]     = __builtin_amdgcn_raw_buffer_load_b128(rw, n, 0, 0);
            wf[sn + 1] = __builtin_amdgcn_raw_buffer_load_b128(rw, n + 1024, 0, 0);
            __builtin_amdgcn_sched_barrier(0);
            i32x8 A = CAT8(wf[s], wf[s + 1]);
            acc0 = __builtin_amdgcn_mfma_scale_f32_32x32x64_f8f6f4(A, CAT8(xf[2 * t], xf[2 * t + 1]), acc0, 1, 1, 0, one, 0, one);
            acc1 = __builtin_amdgcn_mfma_scale_f32_32x32x64_f8f6f4(A, CAT8(xf[16 + 2 * t], xf[17 + 2 * t]), acc1, 1, 1, 0, one, 0, one);
            asm volatile("" : "+v"(acc0), "+v"(acc1));
        }
        char* slot = (char*)red + (j & 1) * 65536;
#pragma unroll
        for (int b = 0; b < 4; ++b) {
            f32x4 p = {acc0[b], acc0[4 + b], acc0[8 + b], acc0[12 + b]};
            *(f32x4*)(slot + wr_base + (((hh * 4 + b) ^ (m & 7)) * 16)) = p;
        }
#pragma unroll
        for (int b = 0; b < 4; ++b) {
            f32x4 p = {acc1[b], acc1[4 + b], acc1[8 + b], acc1[12 + b]};
            *(f32x4*)(slot + wr_base + 4096 + (((hh * 4 + b) ^ (m & 7)) * 16)) = p;
        }
        wg_barrier();
        f32x4 sv[8];
#pragma unroll
        for (int w = 0; w < 8; ++w) sv[w] = *(const f32x4*)(slot + rd_base + w * 8192);
        f32x4 s = ((sv[0] + sv[1]) + (sv[2] + sv[3])) + ((sv[4] + sv[5]) + (sv[6] + sv[7]));
        f32x4 r;
#pragma unroll
        for (int c = 0; c < 4; ++c) {
            float y = s[c] * scale;
            float z = fminf(fmaxf(y, -60.f), 60.f);
            r[c] = y / (1.f + __expf(-z));
        }
        __builtin_amdgcn_raw_buffer_store_b128(__builtin_bit_cast(u32x4, r), ro, out_off + j * 128, 0, OUT_AUX);
    }
    STAMP(7);
}

extern "C" void kernel_launch(int rank, int nranks, ncclComm_t comm,
                              hipStream_t stream,
                              void* const* d_in, const int* in_sizes, int n_in,
                              void* d_out, int out_size,
                              const int* in_dtypes, int out_dtype,
                              void* const* d_out_all,
                              void* const* d_ws_all, size_t ws_bytes) {
    (void)comm; (void)in_sizes; (void)in_dtypes; (void)out_dtype; (void)d_out_all;
    if (nranks != NR || n_in < 4 || (unsigned)rank >= (unsigned)NR || ws_bytes < WS_NEED ||
        out_size != 512 * NDIM) {
        fprintf(stderr, "kernel_launch: unexpected problem shape\n");
        return;
    }
    Args a;
    a.x = (const unsigned char*)d_in[0];
    a.w = (const unsigned char*)d_in[1];
    a.sx = (const float*)d_in[2];
    a.sw = (const float*)d_in[3];
    a.out = (float*)d_out;
    for (int p = 0; p < NR; ++p) a.ws[p] = (unsigned char*)d_ws_all[p];
    a.rank = rank;
    hipLaunchKernelGGL(fused_gemm_rs, dim3(NWG), dim3(512), 0, stream, a);
    HIP_CHECK(hipGetLastError());
}
```

```cpp
#include <hip/hip_runtime.h>
#include <rccl/rccl.h>
#include <cstdio>
#include <cstdint>

#define HIP_CHECK(expr) do { hipError_t _e = (expr); if (_e != hipSuccess) { \
    fprintf(stderr, "HIP error %s at %s:%d\n", hipGetErrorString(_e), __FILE__, __LINE__); \
    return; } } while (0)

typedef unsigned int u32;
typedef unsigned long long u64;
typedef u32   u32x4  __attribute__((ext_vector_type(4)));
typedef int   i32x4  __attribute__((ext_vector_type(4)));
typedef int   i32x8  __attribute__((ext_vector_type(8)));
typedef float f32x4  __attribute__((ext_vector_type(4)));
typedef float f32x16 __attribute__((ext_vector_type(16)));

#define PF_WIN 4
#define C_PAIRS 2
#define NBUF 2
#define X_WIN 2
#define OUT_AUX 16
#define X_PUSH 5
#define POLL_SLEEP 2
#define DO_INV 1

constexpr int NR   = 8;
constexpr int NGM  = 16;
constexpr int NAX  = 14;
constexpr int NWG  = 8 * (NGM + NAX);
constexpr int NDIM = 4096;
constexpr int KS   = 512;
constexpr u64 DEADLINE = 100000000ull;

constexpr u32 OFF_CTR   = 0;
constexpr u32 OFF_TMO   = 32768;
constexpr u32 OFF_ELECT = 32768 + 1024;
constexpr u32 OFF_INVD  = 32768 + 4096;
constexpr u32 OFF_XFLAG = 65536;
constexpr u32 OFF_YFLAG = 98304;
constexpr u32 OFF_RFLAG = 131072;
constexpr u32 OFF_STAMP = 524288;
constexpr u32 CTRL_BYTES = 1u << 20;
constexpr u32 STR_BYTES = 131072;
constexpr u32 STR_PITCH = STR_BYTES + 4096;
constexpr u32 WST_BYTES = 16 * STR_PITCH;
constexpr u32 XB_BYTES  = 2u << 20;
constexpr u32 XS_PITCH  = 16384 + 128;
constexpr u32 XS_BYTES  = 128 * XS_PITCH;
constexpr u32 BUF_BYTES = WST_BYTES + XB_BYTES + XS_BYTES;
constexpr u32 WS_NEED   = CTRL_BYTES + NBUF * BUF_BYTES;

struct Args {
    const unsigned char* x;
    const unsigned char* w;
    unsigned char* myws;
    int rank;
    int pad;
    unsigned char* rot[NR];
    const float* sx;
    const float* sw;
    float* out;
};

#define RSRC(p, bytes) __builtin_amdgcn_make_buffer_rsrc((void*)(p), 0, (int)(bytes), 0x00020000)
#define AUX_SYS 17
#define AUX_AGT 16

__device__ __forceinline__ u32 ld_sys(const unsigned char* p) {
    return __hip_atomic_load((const u32*)p, __ATOMIC_RELAXED, __HIP_MEMORY_SCOPE_SYSTEM);
}
__device__ __forceinline__ void st_sys(unsigned char* p, u32 v) {
    __hip_atomic_store((u32*)p, v, __ATOMIC_RELAXED, __HIP_MEMORY_SCOPE_SYSTEM);
}
__device__ __forceinline__ u32 xcc_id() {
    return (u32)__builtin_amdgcn_s_getreg((3 << 11) | 20) & 0xFu;
}
__device__ __forceinline__ void wait_vm0() { asm volatile("s_waitcnt vmcnt(0)" ::: "memory"); }
__device__ __forceinline__ void wg_barrier() {
    asm volatile("s_waitcnt lgkmcnt(0)" ::: "memory");
    __builtin_amdgcn_s_barrier();
    asm volatile("" ::: "memory");
}

#define STAMP(i) do {} while (0)

__device__ __forceinline__ bool poll_flags(const unsigned char* p, unsigned char* tmo, u32 call) {
    const u64 t0 = __builtin_amdgcn_s_memrealtime();
    for (u32 it = 1;; ++it) {
        u32 v = ld_sys(p);
        if (__all(v >= call)) return true;
        if ((it & 15u) == 0u) {
            u32 t = __hip_atomic_load((u32*)tmo, __ATOMIC_RELAXED, __HIP_MEMORY_SCOPE_AGENT);
            if (t == call || __builtin_amdgcn_s_memrealtime() - t0 > DEADLINE) {
                __hip_atomic_store((u32*)tmo, call, __ATOMIC_RELAXED, __HIP_MEMORY_SCOPE_AGENT);
                return false;
            }
        }
        __builtin_amdgcn_s_sleep(POLL_SLEEP);
    }
}

__device__ __forceinline__ void elect_invalidate(unsigned char* myws, u32 call, u32 xcc, int lane) {
    u32 old = 0;
    if (lane == 0)
        old = __hip_atomic_fetch_max((u32*)(myws + OFF_ELECT + xcc * 128), call,
                                     __ATOMIC_RELAXED, __HIP_MEMORY_SCOPE_AGENT);
    old = __builtin_amdgcn_readfirstlane(old);
    if (old < call) {
        __builtin_amdgcn_fence(__ATOMIC_ACQUIRE, "");
        wait_vm0();
        if (lane == 0) st_sys(myws + OFF_INVD + xcc * 128, call);
    }
}

__global__ void __launch_bounds__(512, 2) fused_gemm_rs(Args a) {
    __shared__ __attribute__((aligned(16))) float red[2 * 8 * 2048 + 4];
    const int lane = threadIdx.x & 63;
    const int wv = __builtin_amdgcn_readfirstlane(threadIdx.x >> 6);
    const int bid = blockIdx.x;
    const int xg = bid & 7;
    const int idx = bid >> 3;
    const int rank = a.rank;
    unsigned char* myws = a.myws;
    const unsigned char* xin = a.x;
    const unsigned char* win = a.w;
    unsigned char* pws = a.rot[((idx - NGM) >> 1) & 7];
    asm volatile("" :: "s"(rank), "s"(myws), "s"(xin), "s"(win), "s"(pws));
    STAMP(0);

    unsigned char* ctr = myws + OFF_CTR + bid * 64;

    if (idx >= NGM) {
        const int hi = idx - NGM;
        const int peer = (rank + 1 + (hi >> 1)) & 7;
        const int blk = (hi & 1) * 8 + xg;
        if (wv > 3) return;
        unsigned char* myctr = ctr + wv * 4;
        constexpr int NX = 16 - X_PUSH;
        const u32 lo = lane * 16;
        if (wv == 0 || wv == 3) {
            __amdgpu_buffer_rsrc_t rx = RSRC(xin + (size_t)(peer * 512 + blk * 32) * KS, 32 * KS);
            const u32 o = (u32)(lane & 31) * KS + (lane >> 5) * 16;
            if (wv == 0) {
                if (X_PUSH == 0) return;
                u32x4 v[X_PUSH > 0 ? X_PUSH : 1];
#pragma unroll
                for (int c = 0; c < X_PUSH; ++c) v[c] = __builtin_amdgcn_raw_buffer_load_b128(rx, o + c * 32, 0, 0);
                const u32 call = __builtin_amdgcn_readfirstlane(ld_sys(myctr)) + 1u;
                __amdgpu_buffer_rsrc_t rd = RSRC(pws + CTRL_BYTES + (call % NBUF) * BUF_BYTES + WST_BYTES
                                                 + (rank * 16 + blk) * 16384, 16384);
                STAMP(1);
#pragma unroll
                for (int c = 0; c < X_PUSH; ++c) {
                    if (c >= X_WIN) asm volatile("s_waitcnt vmcnt(%0)" :: "i"(X_WIN - 1) : "memory");
                    __builtin_amdgcn_raw_buffer_store_b128(v[c], rd, c * 1024 + lo, 0, AUX_SYS);
                }
                wait_vm0();
                STAMP(2);
                if (lane == 0) {
                    st_sys(pws + OFF_XFLAG + (blk * 8 + rank) * 128, call);
                    st_sys(myctr, call);
                }
            } else {
                if (NX == 0) return;
                u32x4 v[NX > 0 ? NX : 1];
#pragma unroll
                for (int c = 0; c < NX; ++c) v[c] = __builtin_amdgcn_raw_buffer_load_b128(rx, o + (X_PUSH + c) * 32, 0, 0);
                const u32 call = __builtin_amdgcn_readfirstlane(ld_sys(myctr)) + 1u;
                unsigned char* xs = myws + CTRL_BYTES + (call % NBUF) * BUF_BYTES + WST_BYTES + XB_BYTES
                                    + (peer * 16 + blk) * XS_PITCH;
                __amdgpu_buffer_rsrc_t rl = RSRC(xs, 16384);
                STAMP(1);
#pragma unroll
                for (int c = 0; c < NX; ++c)
                    __builtin_amdgcn_raw_buffer_store_b128(v[c], rl, (X_PUSH + c) * 1024 + lo, 0, AUX_SYS);
                wait_vm0();
                STAMP(2);
                if (lane == 0) {
                    st_sys(xs + 16384, call);
                    st_sys(myctr, call);
                }
            }
            return;
        }
        const u32 call = __builtin_amdgcn_readfirstlane(ld_sys(myctr)) + 1u;
        if (lane == 0) st_sys(myctr, call);
        if (wv == 2) {
            elect_invalidate(myws, call, xcc_id(), lane);
            return;
        }
        const int stream = xg * 2 + (hi & 1);
        const u32 bufo = CTRL_BYTES + (call % NBUF) * BUF_BYTES;
        const unsigned char* base = pws + bufo + stream * STR_PITCH;
        const unsigned char* xs = pws + bufo + WST_BYTES + XB_BYTES + (rank * 16 + blk) * XS_PITCH;
        const unsigned char* fw = base + STR_BYTES + (lane & 7) * 4;
        if (DO_INV && lane >= 8) fw = myws + OFF_INVD + xcc_id() * 128;
        __amdgpu_buffer_rsrc_t rs = RSRC(base, STR_BYTES);
        __amdgpu_buffer_rsrc_t rxs = RSRC(xs, 16384);
        __amdgpu_buffer_rsrc_t rxd = RSRC(myws + bufo + WST_BYTES + (peer * 16 + blk) * 16384, 16384);
        u32x4 v[PF_WIN];
        static_assert(NX == 0 || NX >= 2 * PF_WIN, "x phase must cover the w flag check");
        STAMP(1);
        if (NX > 0) {
            if (!poll_flags(xs + 16384, myws + OFF_TMO, call)) return;
            STAMP(3);
#pragma unroll
            for (int k = 0; k < PF_WIN; ++k)
                v[k] = __builtin_amdgcn_raw_buffer_load_b128(rxs, lo + (X_PUSH + k) * 1024, 0, AUX_SYS);
            u32 fv = 0;
#pragma unroll
            for (int i = 0; i < NX - PF_WIN; ++i) {
                const int k = i % PF_WIN;
                __builtin_amdgcn_raw_buffer_store_b128(v[k], rxd, lo + (X_PUSH + i) * 1024, 0, AUX_SYS);
                if (i == NX - 2 * PF_WIN) fv = ld_sys(fw);
                v[k] = __builtin_amdgcn_raw_buffer_load_b128(rxs, lo + (X_PUSH + i + PF_WIN) * 1024, 0, AUX_SYS);
                __builtin_amdgcn_sched_barrier(0);
            }
            if (!__all(fv >= call) && !poll_flags(fw, myws + OFF_TMO, call)) return;
        } else {
            if (!poll_flags(fw, myws + OFF_TMO, call)) return;
            STAMP(3);
        }
        if (lane == 0) st_sys(myws + OFF_RFLAG + (stream * 8 + peer) * 128, call);
#pragma unroll
        for (int i = 0; i < PF_WIN; ++i) {
            const int k = (NX + i) % PF_WIN;
            if (NX > 0) __builtin_amdgcn_raw_buffer_store_b128(v[k], rxd, lo + (16 - PF_WIN + i) * 1024, 0, AUX_SYS);
            v[k] = __builtin_amdgcn_raw_buffer_load_b128(rs, lo + i * 1024, 0, 0);
            __builtin_amdgcn_sched_barrier(0);
        }
        for (int n = PF_WIN; n < 128 + PF_WIN; n += PF_WIN) {
#pragma unroll
            for (int i = 0; i < PF_WIN; ++i) {
                const int k = (NX + i) % PF_WIN;
                asm volatile("" :: "v"(v[k]) : "memory");
                v[k] = __builtin_amdgcn_raw_buffer_load_b128(rs, lo + (n + i) * 1024, 0, 0);
                __builtin_amdgcn_sched_barrier(0);
            }
            if (NX > 0 && n == PF_WIN) {
                asm volatile("s_waitcnt vmcnt(%0)" :: "i"(PF_WIN) : "memory");
                if (lane == 0) st_sys(myws + OFF_YFLAG + (blk * 8 + peer) * 128, call);
                STAMP(2);
            }
        }
        STAMP(4);
        return;
    }

    const int gi = idx;
    const int stream = xg * 2 + (gi & 1);
    const int slab = gi >> 1;
    const u32 xcc = xcc_id();

    u32 call;
    if (wv < 4) {
        const int cq = lane & 7;
        const int G = xg * 4 + wv;
        __amdgpu_buffer_rsrc_t rw = RSRC(win, KS * NDIM);
        const u32 o = (u32)(G * 16) * NDIM + gi * 256 + lane * 4;
        u32 d[16];
#pragma unroll
        for (int j = 0; j < 16; ++j) d[j] = __builtin_amdgcn_raw_buffer_load_b32(rw, o + j * NDIM, 0, 0);
        call = __builtin_amdgcn_readfirstlane(ld_sys(ctr)) + 1u;
        u32x4 q[4];
#pragma unroll
        for (int u = 0; u < 4; ++u) {
            u32 t0 = __builtin_amdgcn_perm(d[4 * u + 1], d[4 * u + 0], 0x05010400u);
            u32 t1 = __builtin_amdgcn_perm(d[4 * u + 1], d[4 * u + 0], 0x07030602u);
            u32 t2 = __builtin_amdgcn_perm(d[4 * u + 3], d[4 * u + 2], 0x05010400u);
            u32 t3 = __builtin_amdgcn_perm(d[4 * u + 3], d[4 * u + 2], 0x07030602u);
            q[0][u] = __builtin_amdgcn_perm(t2, t0, 0x05040100u);
            q[1][u] = __builtin_amdgcn_perm(t2, t0, 0x07060302u);
            q[2][u] = __builtin_amdgcn_perm(t3, t1, 0x05040100u);
            q[3][u] = __builtin_amdgcn_perm(t3, t1, 0x07060302u);
        }
        __amdgpu_buffer_rsrc_t rd = RSRC(myws + CTRL_BYTES + (call % NBUF) * BUF_BYTES + gi * STR_PITCH, STR_BYTES);
        const u32 so = (u32)(lane >> 3) * 16384 + (G >> 1) * 1024 + ((G & 1) * 32 + cq) * 16;
#pragma unroll
        for (int c = 0; c < 4; ++c) __builtin_amdgcn_raw_buffer_store_b128(q[c], rd, so + c * 128, 0, AUX_SYS);
    } else {
        call = __builtin_amdgcn_readfirstlane(ld_sys(ctr)) + 1u;
        if (wv == 7) elect_invalidate(myws, call, xcc, lane);
    }
    const float scale = a.sx[0] * a.sw[0];
    STAMP(1);
    wait_vm0();
    wg_barrier();
    STAMP(2);
    int* okw = (int*)&red[2 * 8 * 2048];
    const u32 bufo = CTRL_BYTES + (call % NBUF) * BUF_BYTES;
    if (wv == 0) {
        if (lane == 0) {
            st_sys(myws + bufo + gi * STR_PITCH + STR_BYTES + xg * 4, call);
            st_sys(ctr, call);
        }
    } else if (wv == 7) {
        const int l7 = lane & 7;
        const unsigned char* own = myws + bufo + stream * STR_PITCH + STR_BYTES + l7 * 4;
        const unsigned char* f = myws + OFF_INVD + xcc * 128;
        if (lane < 16) f = myws + OFF_XFLAG + (slab * 16 + lane) * 128;
        else if (lane < 32) f = myws + OFF_YFLAG + (slab * 16 + lane - 16) * 128;
        else if (lane < 40) f = myws + OFF_RFLAG + (stream * 8 + l7) * 128;
        if ((lane < 40 && l7 == rank) || (lane >= 40 && lane < 48) || (!DO_INV && lane >= 48) ||
            (X_PUSH == 0 && lane < 16) || (X_PUSH == 16 && lane >= 16 && lane < 32)) f = own;
        bool ok = poll_flags(f, myws + OFF_TMO, call);
        if (lane == 0) *okw = ok ? 1 : 0;
    }
    wg_barrier();
    STAMP(3);
    const int ok = __builtin_amdgcn_readfirstlane(*okw);

    float* outp = a.out + (size_t)(slab * 64) * NDIM + stream * 256;
    __amdgpu_buffer_rsrc_t ro = RSRC(outp, 63 * NDIM * 4 + 1024);
    if (!ok) {
        const float qnan = __builtin_nanf("");
        const f32x4 nn = {qnan, qnan, qnan, qnan};
#pragma unroll
        for (int r = 0; r < 8; ++r)
            __builtin_amdgcn_raw_buffer_store_b128(__builtin_bit_cast(u32x4, nn), ro,
                                                   (u32)(wv * 8 + r) * NDIM * 4 + lane * 16, 0, 0);
        return;
    }

    __amdgpu_buffer_rsrc_t rw = RSRC(a.rot[(wv - rank - 1) & 7] + bufo + stream * STR_PITCH, STR_BYTES);
    const bool ownk = (wv == rank);
    __amdgpu_buffer_rsrc_t rx = RSRC(ownk ? xin + (size_t)(rank * 512 + slab * 64) * KS
                                          : myws + bufo + WST_BYTES + (wv * 16 + slab * 2) * 16384, 32768);
    const u32 lo = lane * 16;
    const u32 xo = ownk ? (u32)(lane & 31) * KS + (lane >> 5) * 16 : lo;
    const u32 xstep = ownk ? 32u : 1024u;

    u32x4 wf[8];
#pragma unroll
    for (int k = 0; k < 2 * C_PAIRS; ++k) wf[k] = __builtin_amdgcn_raw_buffer_load_b128(rw, lo + k * 1024, 0, 0);
    u32x4 xf[32];
#pragma unroll
    for (int i = 0; i < 32; ++i)
        xf[i] = __builtin_amdgcn_raw_buffer_load_b128(rx, xo + (i >> 4) * 16384, (i & 15) * xstep, AUX_AGT);

    const int m = lane & 31, hh = lane >> 5;
    const u32 wr_base = wv * 8192 + m * 128;
    const int rrow = wv * 8 + (lane >> 3);
    const int re = lane & 7;
    const u32 rd_base = rrow * 128 + ((re ^ (rrow & 7)) * 16);
    const u32 out_off = (u32)rrow * NDIM * 4 + re * 16;
    const int one = 0x7F7F7F7F;

    static_assert(C_PAIRS >= 1 && C_PAIRS <= 3, "4 operand buffers");
#define CAT8(lo4, hi4) __builtin_shufflevector(__builtin_bit_cast(i32x4, lo4), __builtin_bit_cast(i32x4, hi4), 0, 1, 2, 3, 4, 5, 6, 7)
#pragma unroll 1
    for (int j = 0; j < 8; ++j) {
        f32x16 acc0 = {0.f, 0.f, 0.f, 0.f, 0.f, 0.f, 0.f, 0.f, 0.f, 0.f, 0.f, 0.f, 0.f, 0.f, 0.f, 0.f};
        f32x16 acc1 = acc0;
#pragma unroll
        for (int t = 0; t < 8; ++t) {
            const int s = (t & 3) * 2, sn = ((t + C_PAIRS) & 3) * 2;
            asm volatile("" :: "v"(wf[s]), "v"(wf[s + 1]) : "memory");
            const u32 n = (u32)(j * 16 + 2 * t + 2 * C_PAIRS) * 1024 + lo;
            wf[sn]     = __builtin_amdgcn_raw_buffer_load_b128(rw, n, 0, 0);
            wf[sn + 1] = __builtin_amdgcn_raw_buffer_load_b128(rw, n + 1024, 0, 0);
            __builtin_amdgcn_sched_barrier(0);
            i32x8 A = CAT8(wf[s], wf[s + 1]);
            acc0 = __builtin_amdgcn_mfma_scale_f32_32x32x64_f8f6f4(A, CAT8(xf[2 * t], xf[2 * t + 1]), acc0, 1, 1, 0, one, 0, one);
            acc1 = __builtin_amdgcn_mfma_scale_f32_32x32x64_f8f6f4(A, CAT8(xf[16 + 2 * t], xf[17 + 2 * t]), acc1, 1, 1, 0, one, 0, one);
            asm volatile("" : "+v"(acc0), "+v"(acc1));
        }
        char* slot = (char*)red + (j & 1) * 65536;
#pragma unroll
        for (int b = 0; b < 4; ++b) {
            f32x4 p = {acc0[b], acc0[4 + b], acc0[8 + b], acc0[12 + b]};
            *(f32x4*)(slot + wr_base + (((hh * 4 + b) ^ (m & 7)) * 16)) = p;
        }
#pragma unroll
        for (int b = 0; b < 4; ++b) {
            f32x4 p = {acc1[b], acc1[4 + b], acc1[8 + b], acc1[12 + b]};
            *(f32x4*)(slot + wr_base + 4096 + (((hh * 4 + b) ^ (m & 7)) * 16)) = p;
        }
        wg_barrier();
        f32x4 sv[8];
#pragma unroll
        for (int w = 0; w < 8; ++w) sv[w] = *(const f32x4*)(slot + rd_base + w * 8192);
        f32x4 s = ((sv[0] + sv[1]) + (sv[2] + sv[3])) + ((sv[4] + sv[5]) + (sv[6] + sv[7]));
        f32x4 r;
#pragma unroll
        for (int c = 0; c < 4; ++c) {
            float y = s[c] * scale;
            float z = fminf(fmaxf(y, -60.f), 60.f);
            r[c] = y * __builtin_amdgcn_rcpf(1.f + __expf(-z));
        }
        __builtin_amdgcn_raw_buffer_store_b128(__builtin_bit_cast(u32x4, r), ro, out_off + j * 128, 0, OUT_AUX);
    }
    STAMP(7);
}

extern "C" void kernel_launch(int rank, int nranks, ncclComm_t comm,
                              hipStream_t stream,
                              void* const* d_in, const int* in_sizes, int n_in,
                              void* d_out, int out_size,
                              const int* in_dtypes, int out_dtype,
                              void* const* d_out_all,
                              void* const* d_ws_all, size_t ws_bytes) {
    (void)comm; (void)in_sizes; (void)in_dtypes; (void)out_dtype; (void)d_out_all;
    if (nranks != NR || n_in < 4 || (unsigned)rank >= (unsigned)NR || ws_bytes < WS_NEED ||
        out_size != 512 * NDIM) {
        fprintf(stderr, "kernel_launch: unexpected problem shape\n");
        return;
    }
    Args a;
    a.x = (const unsigned char*)d_in[0];
    a.w = (const unsigned char*)d_in[1];
    a.sx = (const float*)d_in[2];
    a.sw = (const float*)d_in[3];
    a.out = (float*)d_out;
    for (int d = 0; d < NR; ++d) a.rot[d] = (unsigned char*)d_ws_all[(rank + 1 + d) & 7];
    a.myws = (unsigned char*)d_ws_all[rank];
    a.pad = 0;
    a.rank = rank;
    hipLaunchKernelGGL(fused_gemm_rs, dim3(NWG), dim3(512), 0, stream, a);
    HIP_CHECK(hipGetLastError());
}
```

```cpp
#include <hip/hip_runtime.h>
#include <rccl/rccl.h>
#include <cstdio>
#include <cstdint>

#define HIP_CHECK(expr) do { hipError_t _e = (expr); if (_e != hipSuccess) { \
    fprintf(stderr, "HIP error %s at %s:%d\n", hipGetErrorString(_e), __FILE__, __LINE__); \
    return; } } while (0)

typedef unsigned int u32;
typedef unsigned long long u64;
typedef u32   u32x4  __attribute__((ext_vector_type(4)));
typedef int   i32x4  __attribute__((ext_vector_type(4)));
typedef int   i32x8  __attribute__((ext_vector_type(8)));
typedef float f32x4  __attribute__((ext_vector_type(4)));
typedef float f32x16 __attribute__((ext_vector_type(16)));

#define PF_WIN 3
#define PF_WIN2 3
#define PF_SW 64
#define C_PAIRS 2
#define NBUF 2
#define X_WIN 2
#define OUT_AUX 16
#define X_PUSH 5
#define Z_PUSH 1
#define POLL_SLEEP 2
#define DO_INV 1

constexpr int NR   = 8;
constexpr int NGM  = 16;
constexpr int NAX  = 14;
constexpr int NWG  = 8 * (NGM + NAX);
constexpr int NDIM = 4096;
constexpr int KS   = 512;
constexpr u64 DEADLINE = 100000000ull;

constexpr u32 OFF_CTR   = 0;
constexpr u32 OFF_TMO   = 32768;
constexpr u32 OFF_ELECT = 32768 + 1024;
constexpr u32 OFF_INVD  = 32768 + 4096;
constexpr u32 OFF_XFLAG = 65536;
constexpr u32 OFF_YFLAG = 98304;
constexpr u32 OFF_RFLAG = 131072;
constexpr u32 OFF_ZFLAG = 163840;
constexpr u32 OFF_STAMP = 524288;
constexpr u32 CTRL_BYTES = 1u << 20;
constexpr u32 STR_BYTES = 131072;
constexpr u32 STR_PITCH = STR_BYTES + 4096;
constexpr u32 WST_BYTES = 16 * STR_PITCH;
constexpr u32 XB_BYTES  = 2u << 20;
constexpr u32 XS_PITCH  = 16384 + 128;
constexpr u32 XS_BYTES  = 128 * XS_PITCH;
constexpr u32 BUF_BYTES = WST_BYTES + XB_BYTES + XS_BYTES;
constexpr u32 WS_NEED   = CTRL_BYTES + NBUF * BUF_BYTES;

struct Args {
    const unsigned char* x;
    const unsigned char* w;
    unsigned char* myws;
    int rank;
    int pad;
    unsigned char* rot[NR];
    const float* sx;
    const float* sw;
    float* out;
};

#define LIKELY(c) __builtin_expect(!!(c), 1)
#define RSRC(p, bytes) __builtin_amdgcn_make_buffer_rsrc((void*)(p), 0, (int)(bytes), 0x00020000)
#define AUX_SYS 17
#define AUX_AGT 16

__device__ __forceinline__ u32 ld_sys(const unsigned char* p) {
    return __hip_atomic_load((const u32*)p, __ATOMIC_RELAXED, __HIP_MEMORY_SCOPE_SYSTEM);
}
__device__ __forceinline__ void st_sys(unsigned char* p, u32 v) {
    __hip_atomic_store((u32*)p, v, __ATOMIC_RELAXED, __HIP_MEMORY_SCOPE_SYSTEM);
}
__device__ __forceinline__ u32 xcc_id() {
    return (u32)__builtin_amdgcn_s_getreg((3 << 11) | 20) & 0xFu;
}
__device__ __forceinline__ void wait_vm0() { asm volatile("s_waitcnt vmcnt(0)" ::: "memory"); }
__device__ __forceinline__ void wg_barrier() {
    asm volatile("s_waitcnt lgkmcnt(0)" ::: "memory");
    __builtin_amdgcn_s_barrier();
    asm volatile("" ::: "memory");
}

#define STAMP(i) do {} while (0)

__device__ __forceinline__ bool poll_flags(const unsigned char* p, unsigned char* tmo, u32 call) {
    const u64 t0 = __builtin_amdgcn_s_memrealtime();
    for (u32 it = 1;; ++it) {
        u32 v = ld_sys(p);
        if (__all(v >= call)) return true;
        if ((it & 15u) == 0u) {
            u32 t = __hip_atomic_load((u32*)tmo, __ATOMIC_RELAXED, __HIP_MEMORY_SCOPE_AGENT);
            if (t == call || __builtin_amdgcn_s_memrealtime() - t0 > DEADLINE) {
                __hip_atomic_store((u32*)tmo, call, __ATOMIC_RELAXED, __HIP_MEMORY_SCOPE_AGENT);
                return false;
            }
        }
        __builtin_amdgcn_s_sleep(POLL_SLEEP);
    }
}

__device__ __forceinline__ void elect_invalidate(unsigned char* myws, u32 call, u32 xcc, int lane) {
    u32 old = 0;
    if (lane == 0)
        old = __hip_atomic_fetch_max((u32*)(myws + OFF_ELECT + xcc * 128), call,
                                     __ATOMIC_RELAXED, __HIP_MEMORY_SCOPE_AGENT);
    old = __builtin_amdgcn_readfirstlane(old);
    if (old < call) {
        __builtin_amdgcn_fence(__ATOMIC_ACQUIRE, "");
        wait_vm0();
        if (lane == 0) st_sys(myws + OFF_INVD + xcc * 128, call);
    }
}

__global__ void __launch_bounds__(512, 2) fused_gemm_rs(Args a) {
    __shared__ __attribute__((aligned(16))) float red[2 * 8 * 2048 + 4];
    const int lane = threadIdx.x & 63;
    const int wv = __builtin_amdgcn_readfirstlane(threadIdx.x >> 6);
    const int bid = blockIdx.x;
    const int xg = bid & 7;
    const int idx = bid >> 3;
    const int rank = a.rank;
    unsigned char* myws = a.myws;
    const unsigned char* xin = a.x;
    const unsigned char* win = a.w;
    unsigned char* pws = a.rot[((idx - NGM) >> 1) & 7];
    asm volatile("" :: "s"(rank), "s"(myws), "s"(xin), "s"(win), "s"(pws));
    STAMP(0);

    unsigned char* ctr = myws + OFF_CTR + bid * 64;

    if (LIKELY(idx >= NGM)) {
        const int hi = idx - NGM;
        const int peer = (rank + 1 + (hi >> 1)) & 7;
        const int blk = (hi & 1) * 8 + xg;
        if (wv > 3) return;
        unsigned char* myctr = ctr + wv * 4;
        constexpr int NX = 16 - X_PUSH;
        const u32 lo = lane * 16;
        if (LIKELY(wv == 0 || wv == 3)) {
            __amdgpu_buffer_rsrc_t rx = RSRC(xin + (size_t)(peer * 512 + blk * 32) * KS, 32 * KS);
            const u32 o = (u32)(lane & 31) * KS + (lane >> 5) * 16;
            if (wv == 0) {
                if (X_PUSH == 0) return;
                u32x4 v[X_PUSH > 0 ? X_PUSH : 1];
                const u32 cv = ld_sys(myctr);
                __builtin_amdgcn_sched_barrier(0);
#pragma unroll
                for (int c = 0; c < X_PUSH; ++c) v[c] = __builtin_amdgcn_raw_buffer_load_b128(rx, o + c * 32, 0, 0);
                __builtin_amdgcn_sched_barrier(0);
                const u32 call = __builtin_amdgcn_readfirstlane(cv) + 1u;
                __amdgpu_buffer_rsrc_t rd = RSRC(pws + CTRL_BYTES + (call % NBUF) * BUF_BYTES + WST_BYTES
                                                 + (rank * 16 + blk) * 16384, 16384);
                STAMP(1);
#pragma unroll
                for (int c = 0; c < X_PUSH; ++c) {
                    if (c >= X_WIN) asm volatile("s_waitcnt vmcnt(%0)" :: "i"(X_WIN - 1) : "memory");
                    __builtin_amdgcn_raw_buffer_store_b128(v[c], rd, c * 1024 + lo, 0, AUX_SYS);
                }
                wait_vm0();
                STAMP(2);
                if (lane == 0) {
                    st_sys(pws + OFF_XFLAG + (blk * 8 + rank) * 128, call);
                    st_sys(myctr, call);
                }
            } else {
                if (NX == 0) return;
                u32x4 v[NX > 0 ? NX : 1];
                const u32 cv = ld_sys(myctr);
                __builtin_amdgcn_sched_barrier(0);
#pragma unroll
                for (int c = 0; c < NX; ++c) v[c] = __builtin_amdgcn_raw_buffer_load_b128(rx, o + (X_PUSH + c) * 32, 0, 0);
                __builtin_amdgcn_sched_barrier(0);
                const u32 call = __builtin_amdgcn_readfirstlane(cv) + 1u;
                unsigned char* xs = myws + CTRL_BYTES + (call % NBUF) * BUF_BYTES + WST_BYTES + XB_BYTES
                                    + (peer * 16 + blk) * XS_PITCH;
                __amdgpu_buffer_rsrc_t rl = RSRC(xs, 16384);
                STAMP(1);
#pragma unroll
                for (int c = 0; c < NX; ++c)
                    __builtin_amdgcn_raw_buffer_store_b128(v[c], rl, (X_PUSH + c) * 1024 + lo, 0, AUX_SYS);
                wait_vm0();
                STAMP(2);
                if (lane == 0) {
                    st_sys(pws + OFF_ZFLAG + (blk * 8 + rank) * 128, call);
                    st_sys(myctr, call);
                }
            }
            return;
        }
        const u32 call = __builtin_amdgcn_readfirstlane(ld_sys(myctr)) + 1u;
        if (lane == 0) st_sys(myctr, call);
        if (wv == 2) {
            elect_invalidate(myws, call, xcc_id(), lane);
            return;
        }
        const int stream = xg * 2 + (hi & 1);
        const u32 bufo = CTRL_BYTES + (call % NBUF) * BUF_BYTES;
        const unsigned char* base = pws + bufo + stream * STR_PITCH;
        const unsigned char* xs = pws + bufo + WST_BYTES + XB_BYTES + (rank * 16 + blk) * XS_PITCH;
        const unsigned char* fw = base + STR_BYTES + (lane & 7) * 4;
        if (DO_INV && lane >= 8) fw = myws + OFF_INVD + xcc_id() * 128;
        __amdgpu_buffer_rsrc_t rs = RSRC(base, STR_BYTES);
        __amdgpu_buffer_rsrc_t rxs = RSRC(xs, 16384);
        __amdgpu_buffer_rsrc_t rxd = RSRC(myws + bufo + WST_BYTES + (peer * 16 + blk) * 16384, 16384);
        u32x4 v[PF_WIN];
        static_assert(NX == 0 || NX >= 2 * PF_WIN, "x phase must cover the w flag check");
        STAMP(1);
        if (NX > 0) {
            if (!poll_flags(myws + OFF_ZFLAG + (blk * 8 + peer) * 128, myws + OFF_TMO, call)) return;
            STAMP(3);
#pragma unroll
            for (int k = 0; k < PF_WIN; ++k)
                v[k] = __builtin_amdgcn_raw_buffer_load_b128(rxs, lo + (X_PUSH + k) * 1024, 0, AUX_SYS);
            u32 fv = 0;
#pragma unroll
            for (int i = 0; i < NX - PF_WIN; ++i) {
                const int k = i % PF_WIN;
                __builtin_amdgcn_raw_buffer_store_b128(v[k], rxd, lo + (X_PUSH + i) * 1024, 0, AUX_SYS);
                if (i == NX - 2 * PF_WIN) fv = ld_sys(fw);
                v[k] = __builtin_amdgcn_raw_buffer_load_b128(rxs, lo + (X_PUSH + i + PF_WIN) * 1024, 0, AUX_SYS);
                __builtin_amdgcn_sched_barrier(0);
            }
            if (!__all(fv >= call) && !poll_flags(fw, myws + OFF_TMO, call)) return;
        } else {
            if (!poll_flags(fw, myws + OFF_TMO, call)) return;
            STAMP(3);
        }
        if (lane == 0) st_sys(myws + OFF_RFLAG + (stream * 8 + peer) * 128, call);
#pragma unroll
        for (int i = 0; i < PF_WIN; ++i) {
            const int k = (NX + i) % PF_WIN;
            if (NX > 0) __builtin_amdgcn_raw_buffer_store_b128(v[k], rxd, lo + (16 - PF_WIN + i) * 1024, 0, AUX_SYS);
            v[k] = __builtin_amdgcn_raw_buffer_load_b128(rs, lo + i * 1024, 0, 0);
            __builtin_amdgcn_sched_barrier(0);
        }
#pragma unroll
        for (int i = 0; i < PF_WIN; ++i) {
            const int k = (NX + i) % PF_WIN;
            asm volatile("" :: "v"(v[k]) : "memory");
            v[k] = __builtin_amdgcn_raw_buffer_load_b128(rs, lo + (PF_WIN + i) * 1024, 0, 0);
            __builtin_amdgcn_sched_barrier(0);
        }
        if (NX > 0) {
            asm volatile("s_waitcnt vmcnt(%0)" :: "i"(PF_WIN) : "memory");
            if (lane == 0) st_sys(myws + OFF_YFLAG + (blk * 8 + peer) * 128, call);
            STAMP(2);
        }
        {
            u32x4 dsc;
            dsc.x = __builtin_amdgcn_readfirstlane((u32)(size_t)base);
            dsc.y = __builtin_amdgcn_readfirstlane((u32)((size_t)base >> 32) & 0xffffu);
            dsc.z = STR_BYTES;
            dsc.w = 0x00020000u;
            u32x4 sink = {0u, 0u, 0u, 0u};
            u32 off = lo + 2 * PF_WIN * 1024;
#pragma unroll 2
            for (int n = 2 * PF_WIN; n < PF_SW; ++n) {
                asm volatile("s_waitcnt vmcnt(%3)\n\tbuffer_load_dwordx4 %0, %1, %2, 0 offen"
                             : "+v"(sink) : "v"(off), "s"(dsc), "i"(PF_WIN - 1) : "memory");
                off += 1024;
            }
#pragma unroll 2
            for (int n = PF_SW; n < 128; ++n) {
                asm volatile("s_waitcnt vmcnt(%3)\n\tbuffer_load_dwordx4 %0, %1, %2, 0 offen"
                             : "+v"(sink) : "v"(off), "s"(dsc), "i"(PF_WIN2 - 1) : "memory");
                off += 1024;
            }
            wait_vm0();
            asm volatile("" :: "v"(sink));
#pragma unroll
            for (int k = 0; k < PF_WIN; ++k) asm volatile("" :: "v"(v[k]));
        }
        STAMP(4);
        return;
    }

    const int gi = idx;
    const int stream = xg * 2 + (gi & 1);
    const int slab = gi >> 1;
    const u32 xcc = xcc_id();

    u32 call;
    if (LIKELY(wv < 4)) {
        const int cq = lane & 7;
        const int G = xg * 4 + wv;
        __amdgpu_buffer_rsrc_t rw = RSRC(win, KS * NDIM);
        const u32 o = (u32)(G * 16) * NDIM + gi * 256 + lane * 4;
        u32 d[16];
#pragma unroll
        for (int j = 0; j < 16; ++j) d[j] = __builtin_amdgcn_raw_buffer_load_b32(rw, o + j * NDIM, 0, 0);
        call = __builtin_amdgcn_readfirstlane(ld_sys(ctr)) + 1u;
        u32x4 q[4];
#pragma unroll
        for (int u = 0; u < 4; ++u) {
            u32 t0 = __builtin_amdgcn_perm(d[4 * u + 1], d[4 * u + 0], 0x05010400u);
            u32 t1 = __builtin_amdgcn_perm(d[4 * u + 1], d[4 * u + 0], 0x07030602u);
            u32 t2 = __builtin_amdgcn_perm(d[4 * u + 3], d[4 * u + 2], 0x05010400u);
            u32 t3 = __builtin_amdgcn_perm(d[4 * u + 3], d[4 * u + 2], 0x07030602u);
            q[0][u] = __builtin_amdgcn_perm(t2, t0, 0x05040100u);
            q[1][u] = __builtin_amdgcn_perm(t2, t0, 0x07060302u);
            q[2][u] = __builtin_amdgcn_perm(t3, t1, 0x05040100u);
            q[3][u] = __builtin_amdgcn_perm(t3, t1, 0x07060302u);
        }
        __amdgpu_buffer_rsrc_t rd = RSRC(myws + CTRL_BYTES + (call % NBUF) * BUF_BYTES + gi * STR_PITCH, STR_BYTES);
        const u32 so = (u32)(lane >> 3) * 16384 + (G >> 1) * 1024 + ((G & 1) * 32 + cq) * 16;
#pragma unroll
        for (int c = 0; c < 4; ++c) __builtin_amdgcn_raw_buffer_store_b128(q[c], rd, so + c * 128, 0, AUX_SYS);
    } else {
        call = __builtin_amdgcn_readfirstlane(ld_sys(ctr)) + 1u;
        if (wv == 7) elect_invalidate(myws, call, xcc, lane);
    }
    const float scale = a.sx[0] * a.sw[0];
    STAMP(1);
    wait_vm0();
    wg_barrier();
    STAMP(2);
    int* okw = (int*)&red[2 * 8 * 2048];
    const u32 bufo = CTRL_BYTES + (call % NBUF) * BUF_BYTES;
    if (wv == 0) {
        if (lane == 0) {
            st_sys(myws + bufo + gi * STR_PITCH + STR_BYTES + xg * 4, call);
            st_sys(ctr, call);
        }
    } else if (wv == 7) {
        const int l7 = lane & 7;
        const unsigned char* own = myws + bufo + stream * STR_PITCH + STR_BYTES + l7 * 4;
        const unsigned char* f = myws + OFF_INVD + xcc * 128;
        if (lane < 16) f = myws + OFF_XFLAG + (slab * 16 + lane) * 128;
        else if (lane < 32) f = myws + OFF_YFLAG + (slab * 16 + lane - 16) * 128;
        else if (lane < 40) f = myws + OFF_RFLAG + (stream * 8 + l7) * 128;
        if ((lane < 40 && l7 == rank) || (lane >= 40 && lane < 48) || (!DO_INV && lane >= 48) ||
            (X_PUSH == 0 && lane < 16) || (X_PUSH == 16 && lane >= 16 && lane < 32)) f = own;
        bool ok = poll_flags(f, myws + OFF_TMO, call);
        if (lane == 0) *okw = ok ? 1 : 0;
    }
    wg_barrier();
    STAMP(3);
    const int ok = __builtin_amdgcn_readfirstlane(*okw);

    float* outp = a.out + (size_t)(slab * 64) * NDIM + stream * 256;
    __amdgpu_buffer_rsrc_t ro = RSRC(outp, 63 * NDIM * 4 + 1024);
    if (!ok) {
        const float qnan = __builtin_nanf("");
        const f32x4 nn = {qnan, qnan, qnan, qnan};
#pragma unroll
        for (int r = 0; r < 8; ++r)
            __builtin_amdgcn_raw_buffer_store_b128(__builtin_bit_cast(u32x4, nn), ro,
                                                   (u32)(wv * 8 + r) * NDIM * 4 + lane * 16, 0, 0);
        return;
    }

    __amdgpu_buffer_rsrc_t rw = RSRC(a.rot[(wv - rank - 1) & 7] + bufo + stream * STR_PITCH, STR_BYTES);
    const bool ownk = (wv == rank);
    __amdgpu_buffer_rsrc_t rx = RSRC(ownk ? xin + (size_t)(rank * 512 + slab * 64) * KS
                                          : myws + bufo + WST_BYTES + (wv * 16 + slab * 2) * 16384, 32768);
    const u32 lo = lane * 16;
    const u32 xo = ownk ? (u32)(lane & 31) * KS + (lane >> 5) * 16 : lo;
    const u32 xstep = ownk ? 32u : 1024u;

    u32x4 wf[8];
#pragma unroll
    for (int k = 0; k < 2 * C_PAIRS; ++k) wf[k] = __builtin_amdgcn_raw_buffer_load_b128(rw, lo + k * 1024, 0, 0);
    u32x4 xf[32];
#pragma unroll
    for (int i = 0; i < 32; ++i)
        xf[i] = __builtin_amdgcn_raw_buffer_load_b128(rx, xo + (i >> 4) * 16384, (i & 15) * xstep, AUX_AGT);

    const int m = lane & 31, hh = lane >> 5;
    const u32 wr_base = wv * 8192 + m * 128;
    const int rrow = wv * 8 + (lane >> 3);
    const int re = lane & 7;
    const u32 rd_base = rrow * 128 + ((re ^ (rrow & 7)) * 16);
    const u32 out_off = (u32)rrow * NDIM * 4 + re * 16;
    const int one = 0x7F7F7F7F;

    static_assert(C_PAIRS >= 1 && C_PAIRS <= 3, "4 operand buffers");
#define CAT8(lo4, hi4) __builtin_shufflevector(__builtin_bit_cast(i32x4, lo4), __builtin_bit_cast(i32x4, hi4), 0, 1, 2, 3, 4, 5, 6, 7)
#pragma unroll 1
    for (int j = 0; j < 8; ++j) {
        f32x16 acc0 = {0.f, 0.f, 0.f, 0.f, 0.f, 0.f, 0.f, 0.f, 0.f, 0.f, 0.f, 0.f, 0.f, 0.f, 0.f, 0.f};
        f32x16 acc1 = acc0;
#pragma unroll
        for (int t = 0; t < 8; ++t) {
            const int s = (t & 3) * 2, sn = ((t + C_PAIRS) & 3) * 2;
            asm volatile("" :: "v"(wf[s]), "v"(wf[s + 1]) : "memory");
            const u32 n = (u32)(j * 16 + 2 * t + 2 * C_PAIRS) * 1024 + lo;
            wf[sn]     = __builtin_amdgcn_raw_buffer_load_b128(rw, n, 0, 0);
            wf[sn + 1] = __builtin_amdgcn_raw_buffer_load_b128(rw, n + 1024, 0, 0);
            __builtin_amdgcn_sched_barrier(0);
            i32x8 A = CAT8(wf[s], wf[s + 1]);
            acc0 = __builtin_amdgcn_mfma_scale_f32_32x32x64_f8f6f4(A, CAT8(xf[2 * t], xf[2 * t + 1]), acc0, 1, 1, 0, one, 0, one);
            acc1 = __builtin_amdgcn_mfma_scale_f32_32x32x64_f8f6f4(A, CAT8(xf[16 + 2 * t], xf[17 + 2 * t]), acc1, 1, 1, 0, one, 0, one);
            asm volatile("" : "+v"(acc0), "+v"(acc1));
        }
        char* slot = (char*)red + (j & 1) * 65536;
#pragma unroll
        for (int b = 0; b < 4; ++b) {
            f32x4 p = {acc0[b], acc0[4 + b], acc0[8 + b], acc0[12 + b]};
            *(f32x4*)(slot + wr_base + (((hh * 4 + b) ^ (m & 7)) * 16)) = p;
        }
#pragma unroll
        for (int b = 0; b < 4; ++b) {
            f32x4 p = {acc1[b], acc1[4 + b], acc1[8 + b], acc1[12 + b]};
            *(f32x4*)(slot + wr_base + 4096 + (((hh * 4 + b) ^ (m & 7)) * 16)) = p;
        }
        wg_barrier();
        f32x4 sv[8];
#pragma unroll
        for (int w = 0; w < 8; ++w) sv[w] = *(const f32x4*)(slot + rd_base + w * 8192);
        f32x4 s = ((sv[0] + sv[1]) + (sv[2] + sv[3])) + ((sv[4] + sv[5]) + (sv[6] + sv[7]));
        f32x4 r;
#pragma unroll
        for (int c = 0; c < 4; ++c) {
            float y = s[c] * scale;
            float z = fminf(fmaxf(y, -60.f), 60.f);
            r[c] = y * __builtin_amdgcn_rcpf(1.f + __expf(-z));
        }
        __builtin_amdgcn_raw_buffer_store_b128(__builtin_bit_cast(u32x4, r), ro, out_off + j * 128, 0, OUT_AUX);
    }
    STAMP(7);
}

extern "C" void kernel_launch(int rank, int nranks, ncclComm_t comm,
                              hipStream_t stream,
                              void* const* d_in, const int* in_sizes, int n_in,
                              void* d_out, int out_size,
                              const int* in_dtypes, int out_dtype,
                              void* const* d_out_all,
                              void* const* d_ws_all, size_t ws_bytes) {
    (void)comm; (void)in_sizes; (void)in_dtypes; (void)out_dtype; (void)d_out_all;
    if (nranks != NR || n_in < 4 || (unsigned)rank >= (unsigned)NR || ws_bytes < WS_NEED ||
        out_size != 512 * NDIM) {
        fprintf(stderr, "kernel_launch: unexpected problem shape\n");
        return;
    }
    Args a;
    a.x = (const unsigned char*)d_in[0];
    a.w = (const unsigned char*)d_in[1];
    a.sx = (const float*)d_in[2];
    a.sw = (const float*)d_in[3];
    a.out = (float*)d_out;
    for (int d = 0; d < NR; ++d) a.rot[d] = (unsigned char*)d_ws_all[(rank + 1 + d) & 7];
    a.myws = (unsigned char*)d_ws_all[rank];
    a.pad = 0;
    a.rank = rank;
    hipLaunchKernelGGL(fused_gemm_rs, dim3(NWG), dim3(512), 0, stream, a);
    HIP_CHECK(hipGetLastError());
}
```

```cpp
#include <hip/hip_runtime.h>
#include <rccl/rccl.h>
#include <cstdio>
#include <cstdint>

#define HIP_CHECK(expr) do { hipError_t _e = (expr); if (_e != hipSuccess) { \
    fprintf(stderr, "HIP error %s at %s:%d\n", hipGetErrorString(_e), __FILE__, __LINE__); \
    return; } } while (0)

typedef unsigned int u32;
typedef unsigned long long u64;
typedef u32   u32x4  __attribute__((ext_vector_type(4)));
typedef int   i32x4  __attribute__((ext_vector_type(4)));
typedef int   i32x8  __attribute__((ext_vector_type(8)));
typedef float f32x4  __attribute__((ext_vector_type(4)));
typedef float f32x16 __attribute__((ext_vector_type(16)));

constexpr int NR   = 8;
constexpr int NGM  = 16;
constexpr int NAX  = 14;
constexpr int NWG  = 8 * (NGM + NAX);
constexpr int NDIM = 4096;
constexpr int KS   = 512;
constexpr u64 DEADLINE = 100000000ull;

constexpr int PF_WIN  = 3;
constexpr int C_PAIRS = 2;
constexpr int X_PUSH  = 5;
constexpr int X_WIN   = 2;
constexpr int NX      = 16 - X_PUSH;
constexpr u32 NBUF    = 2;

constexpr u32 OFF_CTR   = 0;
constexpr u32 OFF_TMO   = 32768;
constexpr u32 OFF_ELECT = 32768 + 1024;
constexpr u32 OFF_INVD  = 32768 + 4096;
constexpr u32 OFF_XFLAG = 65536;
constexpr u32 OFF_YFLAG = 98304;
constexpr u32 OFF_RFLAG = 131072;
constexpr u32 OFF_ZFLAG = 163840;
constexpr u32 CTRL_BYTES = 1u << 20;
constexpr u32 STR_BYTES = 131072;
constexpr u32 STR_PITCH = STR_BYTES + 4096;
constexpr u32 WST_BYTES = 16 * STR_PITCH;
constexpr u32 XB_BYTES  = 2u << 20;
constexpr u32 XS_BYTES  = 2u << 20;
constexpr u32 BUF_BYTES = WST_BYTES + XB_BYTES + XS_BYTES;
constexpr u32 WS_NEED   = CTRL_BYTES + NBUF * BUF_BYTES;

struct Args {
    const unsigned char* x;
    const unsigned char* w;
    unsigned char* myws;
    int rank;
    int pad;
    unsigned char* rot[NR];
    const float* sx;
    const float* sw;
    float* out;
};

#define LIKELY(c) __builtin_expect(!!(c), 1)
#define RSRC(p, bytes) __builtin_amdgcn_make_buffer_rsrc((void*)(p), 0, (int)(bytes), 0x00020000)
#define AUX_SYS 17
#define AUX_AGT 16
#define CAT8(lo4, hi4) __builtin_shufflevector(__builtin_bit_cast(i32x4, lo4), __builtin_bit_cast(i32x4, hi4), \
                                               0, 1, 2, 3, 4, 5, 6, 7)

__device__ __forceinline__ u32 ld_sys(const unsigned char* p) {
    return __hip_atomic_load((const u32*)p, __ATOMIC_RELAXED, __HIP_MEMORY_SCOPE_SYSTEM);
}
__device__ __forceinline__ void st_sys(unsigned char* p, u32 v) {
    __hip_atomic_store((u32*)p, v, __ATOMIC_RELAXED, __HIP_MEMORY_SCOPE_SYSTEM);
}
__device__ __forceinline__ u32 xcc_id() {
    return (u32)__builtin_amdgcn_s_getreg((3 << 11) | 20) & 0xFu;
}
__device__ __forceinline__ void wait_vm0() { asm volatile("s_waitcnt vmcnt(0)" ::: "memory"); }
__device__ __forceinline__ void wg_barrier() {
    asm volatile("s_waitcnt lgkmcnt(0)" ::: "memory");
    __builtin_amdgcn_s_barrier();
    asm volatile("" ::: "memory");
}

template <int SLEEP>
__device__ __forceinline__ bool poll_flags(const unsigned char* p, unsigned char* tmo, u32 call) {
    const u64 t0 = __builtin_amdgcn_s_memrealtime();
    for (u32 it = 1;; ++it) {
        u32 v = ld_sys(p);
        if (__all(v >= call)) { asm volatile("" ::: "memory"); return true; }
        if ((it & 15u) == 0u) {
            u32 t = __hip_atomic_load((u32*)tmo, __ATOMIC_RELAXED, __HIP_MEMORY_SCOPE_AGENT);
            if (t == call || __builtin_amdgcn_s_memrealtime() - t0 > DEADLINE) {
                __hip_atomic_store((u32*)tmo, call, __ATOMIC_RELAXED, __HIP_MEMORY_SCOPE_AGENT);
                return false;
            }
        }
        __builtin_amdgcn_s_sleep(SLEEP);
    }
}

__device__ __forceinline__ void elect_invalidate(unsigned char* myws, u32 call, u32 xcc, int lane) {
    u32 old = 0;
    if (lane == 0)
        old = __hip_atomic_fetch_max((u32*)(myws + OFF_ELECT + xcc * 128), call,
                                     __ATOMIC_RELAXED, __HIP_MEMORY_SCOPE_AGENT);
    old = __builtin_amdgcn_readfirstlane(old);
    if (old < call) {
        __builtin_amdgcn_fence(__ATOMIC_ACQUIRE, "");
        wait_vm0();
        if (lane == 0) st_sys(myws + OFF_INVD + xcc * 128, call);
    }
}

__global__ void __launch_bounds__(512, 2) fused_gemm_rs(Args a) {
    __shared__ __attribute__((aligned(16))) float red[2 * 8 * 2048 + 4];
    const int lane = threadIdx.x & 63;
    const int wv = __builtin_amdgcn_readfirstlane(threadIdx.x >> 6);
    const int bid = blockIdx.x;
    const int xg = bid & 7;
    const int idx = bid >> 3;
    const int rank = a.rank;
    unsigned char* myws = a.myws;
    const unsigned char* xin = a.x;
    const unsigned char* win = a.w;
    unsigned char* pws = a.rot[((idx - NGM) >> 1) & 7];
    asm volatile("" :: "s"(rank), "s"(myws), "s"(xin), "s"(win), "s"(pws));

    unsigned char* ctr = myws + OFF_CTR + bid * 64;
    unsigned char* tmo = myws + OFF_TMO;
    const u32 lo = lane * 16;

    if (LIKELY(idx >= NGM)) {
        const int hi = idx - NGM;
        const int peer = (rank + 1 + (hi >> 1)) & 7;
        const int blk = (hi & 1) * 8 + xg;
        if (wv > 3) return;
        unsigned char* myctr = ctr + wv * 4;
        if (LIKELY(wv == 0 || wv == 3)) {
            __amdgpu_buffer_rsrc_t rx = RSRC(xin + (size_t)(peer * 512 + blk * 32) * KS, 32 * KS);
            const u32 o = (u32)(lane & 31) * KS + (lane >> 5) * 16;
            const u32 cv = ld_sys(myctr);
            __builtin_amdgcn_sched_barrier(0);
            if (wv == 0) {
                u32x4 v[X_PUSH];
#pragma unroll
                for (int c = 0; c < X_PUSH; ++c) v[c] = __builtin_amdgcn_raw_buffer_load_b128(rx, o + c * 32, 0, 0);
                __builtin_amdgcn_sched_barrier(0);
                const u32 call = __builtin_amdgcn_readfirstlane(cv) + 1u;
                __amdgpu_buffer_rsrc_t rd = RSRC(pws + CTRL_BYTES + (call % NBUF) * BUF_BYTES + WST_BYTES
                                                 + (rank * 16 + blk) * 16384, 16384);
#pragma unroll
                for (int c = 0; c < X_PUSH; ++c) {
                    if (c >= X_WIN) asm volatile("s_waitcnt vmcnt(%0)" :: "i"(X_WIN - 1) : "memory");
                    __builtin_amdgcn_raw_buffer_store_b128(v[c], rd, c * 1024 + lo, 0, AUX_SYS);
                }
                wait_vm0();
                if (lane == 0) {
                    st_sys(pws + OFF_XFLAG + (blk * 8 + rank) * 128, call);
                    st_sys(myctr, call);
                }
            } else {
                u32x4 v[NX];
#pragma unroll
                for (int c = 0; c < NX; ++c)
                    v[c] = __builtin_amdgcn_raw_buffer_load_b128(rx, o + (X_PUSH + c) * 32, 0, 0);
                __builtin_amdgcn_sched_barrier(0);
                const u32 call = __builtin_amdgcn_readfirstlane(cv) + 1u;
                __amdgpu_buffer_rsrc_t rl = RSRC(myws + CTRL_BYTES + (call % NBUF) * BUF_BYTES + WST_BYTES + XB_BYTES
                                                 + (peer * 16 + blk) * 16384, 16384);
#pragma unroll
                for (int c = 0; c < NX; ++c)
                    __builtin_amdgcn_raw_buffer_store_b128(v[c], rl, (X_PUSH + c) * 1024 + lo, 0, AUX_SYS);
                wait_vm0();
                if (lane == 0) {
                    st_sys(pws + OFF_ZFLAG + (blk * 8 + rank) * 128, call);
                    st_sys(myctr, call);
                }
            }
            return;
        }
        if (wv == 2) __builtin_amdgcn_s_sleep(64);
        const u32 call = __builtin_amdgcn_readfirstlane(ld_sys(myctr)) + 1u;
        if (lane == 0) st_sys(myctr, call);
        if (wv == 2) {
            elect_invalidate(myws, call, xcc_id(), lane);
            return;
        }
        const int stream = xg * 2 + (hi & 1);
        const u32 bufo = CTRL_BYTES + (call % NBUF) * BUF_BYTES;
        const unsigned char* base = pws + bufo + stream * STR_PITCH;
        const unsigned char* fw = base + STR_BYTES + (lane & 7) * 4;
        if (lane >= 8) fw = myws + OFF_INVD + xcc_id() * 128;
        __amdgpu_buffer_rsrc_t rs  = RSRC(base, STR_BYTES);
        __amdgpu_buffer_rsrc_t rxs = RSRC(pws + bufo + WST_BYTES + XB_BYTES + (rank * 16 + blk) * 16384, 16384);
        __amdgpu_buffer_rsrc_t rxd = RSRC(myws + bufo + WST_BYTES + (peer * 16 + blk) * 16384, 16384);
        u32x4 v[PF_WIN];
        static_assert(NX >= 2 * PF_WIN, "the x phase covers the w flag check");
        if (!poll_flags<2>(myws + OFF_ZFLAG + (blk * 8 + peer) * 128, tmo, call)) return;
#pragma unroll
        for (int k = 0; k < PF_WIN; ++k)
            v[k] = __builtin_amdgcn_raw_buffer_load_b128(rxs, lo + (X_PUSH + k) * 1024, 0, AUX_SYS);
        u32 fv = 0;
#pragma unroll
        for (int i = 0; i < NX - PF_WIN; ++i) {
            const int k = i % PF_WIN;
            __builtin_amdgcn_raw_buffer_store_b128(v[k], rxd, lo + (X_PUSH + i) * 1024, 0, AUX_SYS);
            if (i == NX - 2 * PF_WIN) fv = ld_sys(fw);
            v[k] = __builtin_amdgcn_raw_buffer_load_b128(rxs, lo + (X_PUSH + i + PF_WIN) * 1024, 0, AUX_SYS);
            __builtin_amdgcn_sched_barrier(0);
        }
        if (!__all(fv >= call) && !poll_flags<2>(fw, tmo, call)) return;
        asm volatile("" ::: "memory");
        if (lane == 0) st_sys(myws + OFF_RFLAG + (stream * 8 + peer) * 128, call);
#pragma unroll
        for (int i = 0; i < PF_WIN; ++i) {
            const int k = (NX + i) % PF_WIN;
            __builtin_amdgcn_raw_buffer_store_b128(v[k], rxd, lo + (16 - PF_WIN + i) * 1024, 0, AUX_SYS);
            v[k] = __builtin_amdgcn_raw_buffer_load_b128(rs, lo + i * 1024, 0, 0);
            __builtin_amdgcn_sched_barrier(0);
        }
#pragma unroll
        for (int i = 0; i < PF_WIN; ++i) {
            const int k = (NX + i) % PF_WIN;
            asm volatile("" :: "v"(v[k]) : "memory");
            v[k] = __builtin_amdgcn_raw_buffer_load_b128(rs, lo + (PF_WIN + i) * 1024, 0, 0);
            __builtin_amdgcn_sched_barrier(0);
        }
        asm volatile("s_waitcnt vmcnt(%0)" :: "i"(PF_WIN) : "memory");
        if (lane == 0) st_sys(myws + OFF_YFLAG + (blk * 8 + peer) * 128, call);
        for (int n = 2 * PF_WIN; n < 128 + PF_WIN; n += PF_WIN) {
#pragma unroll
            for (int i = 0; i < PF_WIN; ++i) {
                const int k = (NX + i) % PF_WIN;
                asm volatile("" :: "v"(v[k]) : "memory");
                v[k] = __builtin_amdgcn_raw_buffer_load_b128(rs, lo + (n + i) * 1024, 0, 0);
                __builtin_amdgcn_sched_barrier(0);
            }
        }
        return;
    }

    const int gi = idx;
    const int stream = xg * 2 + (gi & 1);
    const int slab = gi >> 1;
    const u32 xcc = xcc_id();

    __builtin_amdgcn_s_sleep(64);
    u32 call;
    if (LIKELY(wv < 4)) {
        const int cq = lane & 7;
        const int G = xg * 4 + wv;
        __amdgpu_buffer_rsrc_t rw = RSRC(win, KS * NDIM);
        const u32 o = (u32)(G * 16) * NDIM + gi * 256 + lane * 4;
        u32 d[16];
#pragma unroll
        for (int j = 0; j < 16; ++j) d[j] = __builtin_amdgcn_raw_buffer_load_b32(rw, o + j * NDIM, 0, 0);
        call = __builtin_amdgcn_readfirstlane(ld_sys(ctr)) + 1u;
        u32x4 q[4];
#pragma unroll
        for (int u = 0; u < 4; ++u) {
            u32 t0 = __builtin_amdgcn_perm(d[4 * u + 1], d[4 * u + 0], 0x05010400u);
            u32 t1 = __builtin_amdgcn_perm(d[4 * u + 1], d[4 * u + 0], 0x07030602u);
            u32 t2 = __builtin_amdgcn_perm(d[4 * u + 3], d[4 * u + 2], 0x05010400u);
            u32 t3 = __builtin_amdgcn_perm(d[4 * u + 3], d[4 * u + 2], 0x07030602u);
            q[0][u] = __builtin_amdgcn_perm(t2, t0, 0x05040100u);
            q[1][u] = __builtin_amdgcn_perm(t2, t0, 0x07060302u);
            q[2][u] = __builtin_amdgcn_perm(t3, t1, 0x05040100u);
            q[3][u] = __builtin_amdgcn_perm(t3, t1, 0x07060302u);
        }
        __amdgpu_buffer_rsrc_t rd = RSRC(myws + CTRL_BYTES + (call % NBUF) * BUF_BYTES + gi * STR_PITCH, STR_BYTES);
        const u32 so = (u32)(lane >> 3) * 16384 + (G >> 1) * 1024 + ((G & 1) * 32 + cq) * 16;
#pragma unroll
        for (int c = 0; c < 4; ++c) __builtin_amdgcn_raw_buffer_store_b128(q[c], rd, so + c * 128, 0, AUX_SYS);
    } else {
        call = __builtin_amdgcn_readfirstlane(ld_sys(ctr)) + 1u;
        if (wv == 7) elect_invalidate(myws, call, xcc, lane);
    }
    const float scale = a.sx[0] * a.sw[0];
    wait_vm0();
    wg_barrier();
    int* okw = (int*)&red[2 * 8 * 2048];
    const u32 bufo = CTRL_BYTES + (call % NBUF) * BUF_BYTES;
    if (wv == 0) {
        if (lane == 0) {
            st_sys(myws + bufo + gi * STR_PITCH + STR_BYTES + xg * 4, call);
            st_sys(ctr, call);
        }
    } else if (wv == 7) {
        const int l7 = lane & 7;
        const unsigned char* own = myws + bufo + stream * STR_PITCH + STR_BYTES + l7 * 4;
        const unsigned char* f = myws + OFF_INVD + xcc * 128;
        if (lane < 16) f = myws + OFF_XFLAG + (slab * 16 + lane) * 128;
        else if (lane < 32) f = myws + OFF_YFLAG + (slab * 16 + lane - 16) * 128;
        else if (lane < 40) f = myws + OFF_RFLAG + (stream * 8 + l7) * 128;
        if ((lane < 40 && l7 == rank) || (lane >= 40 && lane < 48)) f = own;
        __builtin_amdgcn_s_sleep(100);
        bool ok = poll_flags<16>(f, tmo, call);
        if (ok) ok = __hip_atomic_load((u32*)tmo, __ATOMIC_RELAXED, __HIP_MEMORY_SCOPE_AGENT) != call;
        if (lane == 0) *okw = ok ? 1 : 0;
    }
    wg_barrier();
    const int ok = __builtin_amdgcn_readfirstlane(*okw);

    float* outp = a.out + (size_t)(slab * 64) * NDIM + stream * 256;
    __amdgpu_buffer_rsrc_t ro = RSRC(outp, 63 * NDIM * 4 + 1024);
    if (!ok) {
        const float qnan = __builtin_nanf("");
        const f32x4 nn = {qnan, qnan, qnan, qnan};
#pragma unroll
        for (int r = 0; r < 8; ++r)
            __builtin_amdgcn_raw_buffer_store_b128(__builtin_bit_cast(u32x4, nn), ro,
                                                   (u32)(wv * 8 + r) * NDIM * 4 + lo, 0, 0);
        return;
    }

    __amdgpu_buffer_rsrc_t rw = RSRC(a.rot[(wv - rank - 1) & 7] + bufo + stream * STR_PITCH, STR_BYTES);
    const bool ownk = (wv == rank);
    __amdgpu_buffer_rsrc_t rx = RSRC(ownk ? xin + (size_t)(rank * 512 + slab * 64) * KS
                                          : myws + bufo + WST_BYTES + (wv * 16 + slab * 2) * 16384, 32768);
    const u32 xo = ownk ? (u32)(lane & 31) * KS + (lane >> 5) * 16 : lo;
    const u32 xstep = ownk ? 32u : 1024u;

    u32x4 wf[8];
#pragma unroll
    for (int k = 0; k < 2 * C_PAIRS; ++k) wf[k] = __builtin_amdgcn_raw_buffer_load_b128(rw, lo + k * 1024, 0, 0);
    u32x4 xf[32];
#pragma unroll
    for (int i = 0; i < 32; ++i)
        xf[i] = __builtin_amdgcn_raw_buffer_load_b128(rx, xo + (i >> 4) * 16384, (i & 15) * xstep, AUX_AGT);

    const int m = lane & 31, hh = lane >> 5;
    const u32 wr_base = wv * 8192 + m * 128;
    const int rrow = wv * 8 + (lane >> 3);
    const int re = lane & 7;
    const u32 rd_base = rrow * 128 + ((re ^ (rrow & 7)) * 16);
    const u32 out_off = (u32)rrow * NDIM * 4 + re * 16;
    const int one = 0x7F7F7F7F;

#pragma unroll 1
    for (int j = 0; j < 8; ++j) {
        f32x16 acc0 = {0.f, 0.f, 0.f, 0.f, 0.f, 0.f, 0.f, 0.f, 0.f, 0.f, 0.f, 0.f, 0.f, 0.f, 0.f, 0.f};
        f32x16 acc1 = acc0;
#pragma unroll
        for (int t = 0; t < 8; ++t) {
            const int s = (t & 3) * 2, sn = ((t + C_PAIRS) & 3) * 2;
            asm volatile("" :: "v"(wf[s]), "v"(wf[s + 1]) : "memory");
            const u32 n = (u32)(j * 16 + 2 * t + 2 * C_PAIRS) * 1024 + lo;
            wf[sn]     = __builtin_amdgcn_raw_buffer_load_b128(rw, n, 0, 0);
            wf[sn + 1] = __builtin_amdgcn_raw_buffer_load_b128(rw, n + 1024, 0, 0);
            __builtin_amdgcn_sched_barrier(0);
            i32x8 A = CAT8(wf[s], wf[s + 1]);
            acc0 = __builtin_amdgcn_mfma_scale_f32_32x32x64_f8f6f4(A, CAT8(xf[2 * t], xf[2 * t + 1]), acc0,
                                                                   1, 1, 0, one, 0, one);
            acc1 = __builtin_amdgcn_mfma_scale_f32_32x32x64_f8f6f4(A, CAT8(xf[16 + 2 * t], xf[17 + 2 * t]), acc1,
                                                                   1, 1, 0, one, 0, one);
            asm volatile("" : "+v"(acc0), "+v"(acc1));
        }
        char* slot = (char*)red + (j & 1) * 65536;
#pragma unroll
        for (int b = 0; b < 4; ++b) {
            f32x4 p = {acc0[b], acc0[4 + b], acc0[8 + b], acc0[12 + b]};
            *(f32x4*)(slot + wr_base + (((hh * 4 + b) ^ (m & 7)) * 16)) = p;
        }
#pragma unroll
        for (int b = 0; b < 4; ++b) {
            f32x4 p = {acc1[b], acc1[4 + b], acc1[8 + b], acc1[12 + b]};
            *(f32x4*)(slot + wr_base + 4096 + (((hh * 4 + b) ^ (m & 7)) * 16)) = p;
        }
        wg_barrier();
        f32x4 sv[8];
#pragma unroll
        for (int w = 0; w < 8; ++w) sv[w] = *(const f32x4*)(slot + rd_base + w * 8192);
        f32x4 s = ((sv[0] + sv[1]) + (sv[2] + sv[3])) + ((sv[4] + sv[5]) + (sv[6] + sv[7]));
        f32x4 r;
#pragma unroll
        for (int c = 0; c < 4; ++c) {
            float y = s[c] * scale;
            float z = fminf(fmaxf(y, -60.f), 60.f);
            r[c] = y * __builtin_amdgcn_rcpf(1.f + __expf(-z));
        }
        __builtin_amdgcn_raw_buffer_store_b128(__builtin_bit_cast(u32x4, r), ro, out_off + j * 128, 0, AUX_AGT);
    }
}

extern "C" void kernel_launch(int rank, int nranks, ncclComm_t comm,
                              hipStream_t stream,
                              void* const* d_in, const int* in_sizes, int n_in,
                              void* d_out, int out_size,
                              const int* in_dtypes, int out_dtype,
                              void* const* d_out_all,
                              void* const* d_ws_all, size_t ws_bytes) {
    (void)comm; (void)in_sizes; (void)in_dtypes; (void)out_dtype; (void)d_out_all;
    if (nranks != NR || n_in < 4 || (unsigned)rank >= (unsigned)NR || ws_bytes < WS_NEED ||
        out_size != 512 * NDIM) {
        fprintf(stderr, "kernel_launch: unexpected problem shape\n");
        return;
    }
    Args a;
    a.x = (const unsigned char*)d_in[0];
    a.w = (const unsigned char*)d_in[1];
    a.sx = (const float*)d_in[2];
    a.sw = (const float*)d_in[3];
    a.out = (float*)d_out;
    for (int d = 0; d < NR; ++d) a.rot[d] = (unsigned char*)d_ws_all[(rank + 1 + d) & 7];
    a.myws = (unsigned char*)d_ws_all[rank];
    a.pad = 0;
    a.rank = rank;
    hipLaunchKernelGGL(fused_gemm_rs, dim3(NWG), dim3(512), 0, stream, a);
    HIP_CHECK(hipGetLastError());
}
```

```cpp
#include <hip/hip_runtime.h>
#include <rccl/rccl.h>
#include <cstdio>
#include <cstdint>

#define HIP_CHECK(expr) do { hipError_t _e = (expr); if (_e != hipSuccess) { \
    fprintf(stderr, "HIP error %s at %s:%d\n", hipGetErrorString(_e), __FILE__, __LINE__); \
    return; } } while (0)

typedef unsigned int u32;
typedef unsigned long long u64;
typedef u32   u32x4  __attribute__((ext_vector_type(4)));
typedef int   i32x4  __attribute__((ext_vector_type(4)));
typedef int   i32x8  __attribute__((ext_vector_type(8)));
typedef float f32x4  __attribute__((ext_vector_type(4)));
typedef float f32x16 __attribute__((ext_vector_type(16)));

constexpr int NR   = 8;
constexpr int NGM  = 16;
constexpr int NAX  = 14;
constexpr int NWG  = 8 * (NGM + NAX);
constexpr int NDIM = 4096;
constexpr int KS   = 512;
constexpr u64 DEADLINE = 100000000ull;

constexpr int PF_WIN  = 3;
constexpr int C_PAIRS = 2;
constexpr int X_PUSH  = 5;
constexpr int X_WIN   = 2;
constexpr int NX      = 16 - X_PUSH;
constexpr u32 NBUF    = 2;

constexpr u32 OFF_CTR   = 0;
constexpr u32 OFF_TMO   = 32768;
constexpr u32 OFF_ELECT = 32768 + 1024;
constexpr u32 OFF_INVD  = 32768 + 4096;
constexpr u32 OFF_XFLAG = 65536;
constexpr u32 OFF_YFLAG = 98304;
constexpr u32 OFF_RFLAG = 131072;
constexpr u32 OFF_ZFLAG = 163840;
constexpr u32 CTRL_BYTES = 1u << 20;
constexpr u32 STR_BYTES = 131072;
constexpr u32 STR_PITCH = STR_BYTES + 4096;
constexpr u32 WST_BYTES = 16 * STR_PITCH;
constexpr u32 XB_BYTES  = 2u << 20;
constexpr u32 XS_BYTES  = 2u << 20;
constexpr u32 BUF_BYTES = WST_BYTES + XB_BYTES + XS_BYTES;
constexpr u32 WS_NEED   = CTRL_BYTES + NBUF * BUF_BYTES;

struct Args {
    const unsigned char* x;
    const unsigned char* w;
    unsigned char* myws;
    int rank;
    int pad;
    unsigned char* rot[NR];
    const float* sx;
    const float* sw;
    float* out;
};

#define LIKELY(c) __builtin_expect(!!(c), 1)
#define RSRC(p, bytes) __builtin_amdgcn_make_buffer_rsrc((void*)(p), 0, (int)(bytes), 0x00020000)
#define AUX_SYS 17
#define AUX_AGT 16
#define CAT8(lo4, hi4) __builtin_shufflevector(__builtin_bit_cast(i32x4, lo4), __builtin_bit_cast(i32x4, hi4), \
                                               0, 1, 2, 3, 4, 5, 6, 7)

__device__ __forceinline__ u32 ld_sys(const unsigned char* p) {
    return __hip_atomic_load((const u32*)p, __ATOMIC_RELAXED, __HIP_MEMORY_SCOPE_SYSTEM);
}
__device__ __forceinline__ void st_sys(unsigned char* p, u32 v) {
    __hip_atomic_store((u32*)p, v, __ATOMIC_RELAXED, __HIP_MEMORY_SCOPE_SYSTEM);
}
__device__ __forceinline__ u32 xcc_id() {
    return (u32)__builtin_amdgcn_s_getreg((3 << 11) | 20) & 0xFu;
}
__device__ __forceinline__ void wait_vm0() { asm volatile("s_waitcnt vmcnt(0)" ::: "memory"); }
__device__ __forceinline__ void wg_barrier() {
    asm volatile("s_waitcnt lgkmcnt(0)" ::: "memory");
    __builtin_amdgcn_s_barrier();
    asm volatile("" ::: "memory");
}

template <int SLEEP>
__device__ __forceinline__ bool poll_flags(const unsigned char* p, unsigned char* tmo, u32 call) {
    const u64 t0 = __builtin_amdgcn_s_memrealtime();
    for (u32 it = 1;; ++it) {
        u32 v = ld_sys(p);
        if (__all(v >= call)) { asm volatile("" ::: "memory"); return true; }
        if ((it & 15u) == 0u) {
            u32 t = __hip_atomic_load((u32*)tmo, __ATOMIC_RELAXED, __HIP_MEMORY_SCOPE_AGENT);
            if (t == call || __builtin_amdgcn_s_memrealtime() - t0 > DEADLINE) {
                __hip_atomic_store((u32*)tmo, call, __ATOMIC_RELAXED, __HIP_MEMORY_SCOPE_AGENT);
                return false;
            }
        }
        __builtin_amdgcn_s_sleep(SLEEP);
    }
}

__device__ __forceinline__ void elect_invalidate(unsigned char* myws, u32 call, u32 xcc, int lane) {
    u32 old = 0;
    if (lane == 0)
        old = __hip_atomic_fetch_max((u32*)(myws + OFF_ELECT + xcc * 128), call,
                                     __ATOMIC_RELAXED, __HIP_MEMORY_SCOPE_AGENT);
    old = __builtin_amdgcn_readfirstlane(old);
    if (old < call) {
        __builtin_amdgcn_fence(__ATOMIC_ACQUIRE, "");
        wait_vm0();
        if (lane == 0) st_sys(myws + OFF_INVD + xcc * 128, call);
    }
}

__global__ void __launch_bounds__(512, 2) fused_gemm_rs(Args a) {
    __shared__ __attribute__((aligned(16))) float red[2 * 8 * 2048 + 4];
    const int lane = threadIdx.x & 63;
    const int wv = __builtin_amdgcn_readfirstlane(threadIdx.x >> 6);
    const int bid = blockIdx.x;
    const int xg = bid & 7;
    const int idx = bid >> 3;
    const int rank = a.rank;
    unsigned char* myws = a.myws;
    const unsigned char* xin = a.x;
    const unsigned char* win = a.w;
    unsigned char* pws = a.rot[((idx - NGM) >> 1) & 7];
    asm volatile("" :: "s"(rank), "s"(myws), "s"(xin), "s"(win), "s"(pws));

    unsigned char* ctr = myws + OFF_CTR + bid * 64;
    unsigned char* tmo = myws + OFF_TMO;
    const u32 lo = lane * 16;

    if (LIKELY(idx >= NGM)) {
        const int hi = idx - NGM;
        const int peer = (rank + 1 + (hi >> 1)) & 7;
        const int blk = (hi & 1) * 8 + xg;
        if (wv > 3) return;
        unsigned char* myctr = ctr + wv * 4;
        if (LIKELY(wv == 0 || wv == 3)) {
            __amdgpu_buffer_rsrc_t rx = RSRC(xin + (size_t)(peer * 512 + blk * 32) * KS, 32 * KS);
            const u32 o = (u32)(lane & 31) * KS + (lane >> 5) * 16;
            const u32 cv = ld_sys(myctr);
            __builtin_amdgcn_sched_barrier(0);
            if (wv == 0) {
                u32x4 v[X_PUSH];
#pragma unroll
                for (int c = 0; c < X_PUSH; ++c) v[c] = __builtin_amdgcn_raw_buffer_load_b128(rx, o + c * 32, 0, 0);
                __builtin_amdgcn_sched_barrier(0);
                const u32 call = __builtin_amdgcn_readfirstlane(cv) + 1u;
                __amdgpu_buffer_rsrc_t rd = RSRC(pws + CTRL_BYTES + (call % NBUF) * BUF_BYTES + WST_BYTES
                                                 + (rank * 16 + blk) * 16384, 16384);
#pragma unroll
                for (int c = 0; c < X_PUSH; ++c) {
                    if (c >= X_WIN) asm volatile("s_waitcnt vmcnt(%0)" :: "i"(X_WIN - 1) : "memory");
                    __builtin_amdgcn_raw_buffer_store_b128(v[c], rd, c * 1024 + lo, 0, AUX_SYS);
                }
                wait_vm0();
                if (lane == 0) {
                    st_sys(pws + OFF_XFLAG + (blk * 8 + rank) * 128, call);
                    st_sys(myctr, call);
                }
            } else {
                u32x4 v[NX];
#pragma unroll
                for (int c = 0; c < NX; ++c)
                    v[c] = __builtin_amdgcn_raw_buffer_load_b128(rx, o + (X_PUSH + c) * 32, 0, 0);
                __builtin_amdgcn_sched_barrier(0);
                const u32 call = __builtin_amdgcn_readfirstlane(cv) + 1u;
                __amdgpu_buffer_rsrc_t rl = RSRC(myws + CTRL_BYTES + (call % NBUF) * BUF_BYTES + WST_BYTES + XB_BYTES
                                                 + (peer * 16 + blk) * 16384, 16384);
#pragma unroll
                for (int c = 0; c < NX; ++c)
                    __builtin_amdgcn_raw_buffer_store_b128(v[c], rl, (X_PUSH + c) * 1024 + lo, 0, AUX_SYS);
                wait_vm0();
                if (lane == 0) {
                    st_sys(pws + OFF_ZFLAG + (blk * 8 + rank) * 128, call);
                    st_sys(myctr, call);
                }
            }
            return;
        }
        if (wv == 2) __builtin_amdgcn_s_sleep(64);
        const u32 call = __builtin_amdgcn_readfirstlane(ld_sys(myctr)) + 1u;
        if (lane == 0) st_sys(myctr, call);
        if (wv == 2) {
            elect_invalidate(myws, call, xcc_id(), lane);
            return;
        }
        const int stream = xg * 2 + (hi & 1);
        const u32 bufo = CTRL_BYTES + (call % NBUF) * BUF_BYTES;
        const unsigned char* base = pws + bufo + stream * STR_PITCH;
        const unsigned char* fw = base + STR_BYTES + (lane & 7) * 4;
        if (lane >= 8) fw = myws + OFF_INVD + xcc_id() * 128;
        __amdgpu_buffer_rsrc_t rs  = RSRC(base, STR_BYTES);
        __amdgpu_buffer_rsrc_t rxs = RSRC(pws + bufo + WST_BYTES + XB_BYTES + (rank * 16 + blk) * 16384, 16384);
        __amdgpu_buffer_rsrc_t rxd = RSRC(myws + bufo + WST_BYTES + (peer * 16 + blk) * 16384, 16384);
        u32x4 v[PF_WIN];
        static_assert(NX >= 2 * PF_WIN, "the x phase covers the w flag check");
        if (!poll_flags<2>(myws + OFF_ZFLAG + (blk * 8 + peer) * 128, tmo, call)) return;
#pragma unroll
        for (int k = 0; k < PF_WIN; ++k)
            v[k] = __builtin_amdgcn_raw_buffer_load_b128(rxs, lo + (X_PUSH + k) * 1024, 0, AUX_SYS);
        u32 fv = 0;
#pragma unroll
        for (int i = 0; i < NX - PF_WIN; ++i) {
            const int k = i % PF_WIN;
            __builtin_amdgcn_raw_buffer_store_b128(v[k], rxd, lo + (X_PUSH + i) * 1024, 0, AUX_SYS);
            if (i == NX - 2 * PF_WIN) fv = ld_sys(fw);
            v[k] = __builtin_amdgcn_raw_buffer_load_b128(rxs, lo + (X_PUSH + i + PF_WIN) * 1024, 0, AUX_SYS);
            __builtin_amdgcn_sched_barrier(0);
        }
        if (!__all(fv >= call) && !poll_flags<2>(fw, tmo, call)) return;
        asm volatile("" ::: "memory");
        if (lane == 0) st_sys(myws + OFF_RFLAG + (stream * 8 + peer) * 128, call);
#pragma unroll
        for (int i = 0; i < PF_WIN; ++i) {
            const int k = (NX + i) % PF_WIN;
            __builtin_amdgcn_raw_buffer_store_b128(v[k], rxd, lo + (16 - PF_WIN + i) * 1024, 0, AUX_SYS);
            v[k] = __builtin_amdgcn_raw_buffer_load_b128(rs, lo + i * 1024, 0, 0);
            __builtin_amdgcn_sched_barrier(0);
        }
#pragma unroll
        for (int i = 0; i < PF_WIN; ++i) {
            const int k = (NX + i) % PF_WIN;
            asm volatile("" :: "v"(v[k]) : "memory");
            v[k] = __builtin_amdgcn_raw_buffer_load_b128(rs, lo + (PF_WIN + i) * 1024, 0, 0);
            __builtin_amdgcn_sched_barrier(0);
        }
        asm volatile("s_waitcnt vmcnt(%0)" :: "i"(PF_WIN) : "memory");
        if (lane == 0) st_sys(myws + OFF_YFLAG + (blk * 8 + peer) * 128, call);
        asm volatile(".Lgemmrs_pull_loop:");
        for (int n = 2 * PF_WIN; n < 128 + PF_WIN; n += PF_WIN) {
#pragma unroll
            for (int i = 0; i < PF_WIN; ++i) {
                const int k = (NX + i) % PF_WIN;
                asm volatile("" :: "v"(v[k]) : "memory");
                v[k] = __builtin_amdgcn_raw_buffer_load_b128(rs, lo + (n + i) * 1024, 0, 0);
                __builtin_amdgcn_sched_barrier(0);
            }
        }
        return;
    }

    const int gi = idx;
    const int stream = xg * 2 + (gi & 1);
    const int slab = gi >> 1;
    const u32 xcc = xcc_id();

    if (wv == 4) {
        if (gi < NAX && !(gi & 1)) {
            const int peer = (rank + 1 + (gi >> 1)) & 7;
            u32 bo = CTRL_BYTES + (lane & 1) * BUF_BYTES;
            const unsigned char* t = pws + OFF_XFLAG + (xg * 8 + rank) * 128 + 64;
            if ((lane >> 1) == 1) t = pws + bo + WST_BYTES + (rank * 16 + xg) * 16384;
            if ((lane >> 1) == 2) t = myws + bo + WST_BYTES + XB_BYTES + (peer * 16 + xg) * 16384;
            u32 v = 0;
            if (lane < 6) v = ld_sys(t);
            asm volatile("" :: "v"(v));
            bo = CTRL_BYTES + ((__builtin_amdgcn_readfirstlane(ld_sys(ctr)) + 1u) % NBUF) * BUF_BYTES;
            t = pws + bo + WST_BYTES + XB_BYTES + (rank * 16 + xg) * 16384;
            if (lane == 1) t = myws + bo + WST_BYTES + (rank * 16 + xg) * 16384;
            if (lane == 2) t = pws + bo + (xg * 2 + 1) * STR_PITCH + STR_BYTES + 256;
            if (lane == 3) t = pws + bo + (xg * 2) * STR_PITCH - 256;
            if (lane == 4) t = win + (size_t)(xg * 64) * NDIM;
            if (lane < 5) v = ld_sys(t);
            asm volatile("" :: "v"(v));
        }
        else if (gi == 1) {
            u64 pc; u32 off, len;
            asm volatile("s_getpc_b64 %0\n1:\n\ts_mov_b32 %1, 1b-_Z13fused_gemm_rs4Args\n\t"
                         "s_mov_b32 %2, .Lgemmrs_pull_loop-_Z13fused_gemm_rs4Args" : "=s"(pc), "=s"(off), "=s"(len));
            __amdgpu_buffer_rsrc_t rc = RSRC(pc - off, len);
            u32 v = __builtin_amdgcn_raw_buffer_load_b32(rc, lane * 128, 0, 0);
            asm volatile("" :: "v"(v));
        }
    } else {
        __builtin_amdgcn_s_sleep(64);
    }
    u32 call;
    if (LIKELY(wv < 4)) {
        const int cq = lane & 7;
        const int G = xg * 4 + wv;
        __amdgpu_buffer_rsrc_t rw = RSRC(win, KS * NDIM);
        const u32 o = (u32)(G * 16) * NDIM + gi * 256 + lane * 4;
        u32 d[16];
#pragma unroll
        for (int j = 0; j < 16; ++j) d[j] = __builtin_amdgcn_raw_buffer_load_b32(rw, o + j * NDIM, 0, 0);
        call = __builtin_amdgcn_readfirstlane(ld_sys(ctr)) + 1u;
        u32x4 q[4];
#pragma unroll
        for (int u = 0; u < 4; ++u) {
            u32 t0 = __builtin_amdgcn_perm(d[4 * u + 1], d[4 * u + 0], 0x05010400u);
            u32 t1 = __builtin_amdgcn_perm(d[4 * u + 1], d[4 * u + 0], 0x07030602u);
            u32 t2 = __builtin_amdgcn_perm(d[4 * u + 3], d[4 * u + 2], 0x05010400u);
            u32 t3 = __builtin_amdgcn_perm(d[4 * u + 3], d[4 * u + 2], 0x07030602u);
            q[0][u] = __builtin_amdgcn_perm(t2, t0, 0x05040100u);
            q[1][u] = __builtin_amdgcn_perm(t2, t0, 0x07060302u);
            q[2][u] = __builtin_amdgcn_perm(t3, t1, 0x05040100u);
            q[3][u] = __builtin_amdgcn_perm(t3, t1, 0x07060302u);
        }
        __amdgpu_buffer_rsrc_t rd = RSRC(myws + CTRL_BYTES + (call % NBUF) * BUF_BYTES + gi * STR_PITCH, STR_BYTES);
        const u32 so = (u32)(lane >> 3) * 16384 + (G >> 1) * 1024 + ((G & 1) * 32 + cq) * 16;
#pragma unroll
        for (int c = 0; c < 4; ++c) __builtin_amdgcn_raw_buffer_store_b128(q[c], rd, so + c * 128, 0, AUX_SYS);
    } else {
        call = __builtin_amdgcn_readfirstlane(ld_sys(ctr)) + 1u;
        if (wv == 7) elect_invalidate(myws, call, xcc, lane);
    }
    const float scale = a.sx[0] * a.sw[0];
    wait_vm0();
    wg_barrier();
    int* okw = (int*)&red[2 * 8 * 2048];
    const u32 bufo = CTRL_BYTES + (call % NBUF) * BUF_BYTES;
    if (wv == 0) {
        if (lane == 0) {
            st_sys(myws + bufo + gi * STR_PITCH + STR_BYTES + xg * 4, call);
            st_sys(ctr, call);
        }
    } else if (wv == 7) {
        const int l7 = lane & 7;
        const unsigned char* own = myws + bufo + stream * STR_PITCH + STR_BYTES + l7 * 4;
        const unsigned char* f = myws + OFF_INVD + xcc * 128;
        if (lane < 16) f = myws + OFF_XFLAG + (slab * 16 + lane) * 128;
        else if (lane < 32) f = myws + OFF_YFLAG + (slab * 16 + lane - 16) * 128;
        else if (lane < 40) f = myws + OFF_RFLAG + (stream * 8 + l7) * 128;
        if ((lane < 40 && l7 == rank) || (lane >= 40 && lane < 48)) f = own;
        __builtin_amdgcn_s_sleep(100);
        bool ok = poll_flags<16>(f, tmo, call);
        if (ok) ok = __hip_atomic_load((u32*)tmo, __ATOMIC_RELAXED, __HIP_MEMORY_SCOPE_AGENT) != call;
        if (lane == 0) *okw = ok ? 1 : 0;
    }
    wg_barrier();
    const int ok = __builtin_amdgcn_readfirstlane(*okw);

    float* outp = a.out + (size_t)(slab * 64) * NDIM + stream * 256;
    __amdgpu_buffer_rsrc_t ro = RSRC(outp, 63 * NDIM * 4 + 1024);
    if (!ok) {
        const float qnan = __builtin_nanf("");
        const f32x4 nn = {qnan, qnan, qnan, qnan};
#pragma unroll
        for (int r = 0; r < 8; ++r)
            __builtin_amdgcn_raw_buffer_store_b128(__builtin_bit_cast(u32x4, nn), ro,
                                                   (u32)(wv * 8 + r) * NDIM * 4 + lo, 0, 0);
        return;
    }

    __amdgpu_buffer_rsrc_t rw = RSRC(a.rot[(wv - rank - 1) & 7] + bufo + stream * STR_PITCH, STR_BYTES);
    const bool ownk = (wv == rank);
    __amdgpu_buffer_rsrc_t rx = RSRC(ownk ? xin + (size_t)(rank * 512 + slab * 64) * KS
                                          : myws + bufo + WST_BYTES + (wv * 16 + slab * 2) * 16384, 32768);
    const u32 xo = ownk ? (u32)(lane & 31) * KS + (lane >> 5) * 16 : lo;
    const u32 xstep = ownk ? 32u : 1024u;

    u32x4 wf[8];
#pragma unroll
    for (int k = 0; k < 2 * C_PAIRS; ++k) wf[k] = __builtin_amdgcn_raw_buffer_load_b128(rw, lo + k * 1024, 0, 0);
    u32x4 xf[32];
#pragma unroll
    for (int i = 0; i < 32; ++i)
        xf[i] = __builtin_amdgcn_raw_buffer_load_b128(rx, xo + (i >> 4) * 16384, (i & 15) * xstep, AUX_AGT);

    const int m = lane & 31, hh = lane >> 5;
    const u32 wr_base = wv * 8192 + m * 128;
    const int rrow = wv * 8 + (lane >> 3);
    const int re = lane & 7;
    const u32 rd_base = rrow * 128 + ((re ^ (rrow & 7)) * 16);
    const u32 out_off = (u32)rrow * NDIM * 4 + re * 16;
    const int one = 0x7F7F7F7F;

#pragma unroll 1
    for (int j = 0; j < 8; ++j) {
        f32x16 acc0 = {0.f, 0.f, 0.f, 0.f, 0.f, 0.f, 0.f, 0.f, 0.f, 0.f, 0.f, 0.f, 0.f, 0.f, 0.f, 0.f};
        f32x16 acc1 = acc0;
#pragma unroll
        for (int t = 0; t < 8; ++t) {
            const int s = (t & 3) * 2, sn = ((t + C_PAIRS) & 3) * 2;
            asm volatile("" :: "v"(wf[s]), "v"(wf[s + 1]) : "memory");
            const u32 n = (u32)(j * 16 + 2 * t + 2 * C_PAIRS) * 1024 + lo;
            wf[sn]     = __builtin_amdgcn_raw_buffer_load_b128(rw, n, 0, 0);
            wf[sn + 1] = __builtin_amdgcn_raw_buffer_load_b128(rw, n + 1024, 0, 0);
            __builtin_amdgcn_sched_barrier(0);
            i32x8 A = CAT8(wf[s], wf[s + 1]);
            acc0 = __builtin_amdgcn_mfma_scale_f32_32x32x64_f8f6f4(A, CAT8(xf[2 * t], xf[2 * t + 1]), acc0,
                                                                   1, 1, 0, one, 0, one);
            acc1 = __builtin_amdgcn_mfma_scale_f32_32x32x64_f8f6f4(A, CAT8(xf[16 + 2 * t], xf[17 + 2 * t]), acc1,
                                                                   1, 1, 0, one, 0, one);
            asm volatile("" : "+v"(acc0), "+v"(acc1));
        }
        char* slot = (char*)red + (j & 1) * 65536;
#pragma unroll
        for (int b = 0; b < 4; ++b) {
            f32x4 p = {acc0[b], acc0[4 + b], acc0[8 + b], acc0[12 + b]};
            *(f32x4*)(slot + wr_base + (((hh * 4 + b) ^ (m & 7)) * 16)) = p;
        }
#pragma unroll
        for (int b = 0; b < 4; ++b) {
            f32x4 p = {acc1[b], acc1[4 + b], acc1[8 + b], acc1[12 + b]};
            *(f32x4*)(slot + wr_base + 4096 + (((hh * 4 + b) ^ (m & 7)) * 16)) = p;
        }
        wg_barrier();
        f32x4 sv[8];
#pragma unroll
        for (int w = 0; w < 8; ++w) sv[w] = *(const f32x4*)(slot + rd_base + w * 8192);
        f32x4 s = ((sv[0] + sv[1]) + (sv[2] + sv[3])) + ((sv[4] + sv[5]) + (sv[6] + sv[7]));
        f32x4 r;
#pragma unroll
        for (int c = 0; c < 4; ++c) {
            float y = s[c] * scale;
            float z = fminf(fmaxf(y, -60.f), 60.f);
            r[c] = y * __builtin_amdgcn_rcpf(1.f + __expf(-z));
        }
        __builtin_amdgcn_raw_buffer_store_b128(__builtin_bit_cast(u32x4, r), ro, out_off + j * 128, 0, AUX_AGT);
    }
}

extern "C" void kernel_launch(int rank, int nranks, ncclComm_t comm,
                              hipStream_t stream,
                              void* const* d_in, const int* in_sizes, int n_in,
                              void* d_out, int out_size,
                              const int* in_dtypes, int out_dtype,
                              void* const* d_out_all,
                              void* const* d_ws_all, size_t ws_bytes) {
    (void)comm; (void)in_sizes; (void)in_dtypes; (void)out_dtype; (void)d_out_all;
    if (nranks != NR || n_in < 4 || (unsigned)rank >= (unsigned)NR || ws_bytes < WS_NEED ||
        out_size != 512 * NDIM) {
        fprintf(stderr, "kernel_launch: unexpected problem shape\n");
        return;
    }
    Args a;
    a.x = (const unsigned char*)d_in[0];
    a.w = (const unsigned char*)d_in[1];
    a.sx = (const float*)d_in[2];
    a.sw = (const float*)d_in[3];
    a.out = (float*)d_out;
    for (int d = 0; d < NR; ++d) a.rot[d] = (unsigned char*)d_ws_all[(rank + 1 + d) & 7];
    a.myws = (unsigned char*)d_ws_all[rank];
    a.pad = 0;
    a.rank = rank;
    hipLaunchKernelGGL(fused_gemm_rs, dim3(NWG), dim3(512), 0, stream, a);
    HIP_CHECK(hipGetLastError());
}
```

```cpp
#include <hip/hip_runtime.h>
#include <rccl/rccl.h>
#include <cstdio>
#include <cstdint>

#define HIP_CHECK(expr) do { hipError_t _e = (expr); if (_e != hipSuccess) { \
    fprintf(stderr, "HIP error %s at %s:%d\n", hipGetErrorString(_e), __FILE__, __LINE__); \
    return; } } while (0)

typedef unsigned int u32;
typedef unsigned long long u64;
typedef u32   u32x4  __attribute__((ext_vector_type(4)));
typedef int   i32x4  __attribute__((ext_vector_type(4)));
typedef int   i32x8  __attribute__((ext_vector_type(8)));
typedef float f32x4  __attribute__((ext_vector_type(4)));
typedef float f32x16 __attribute__((ext_vector_type(16)));

#define PF_WIN 4
#define CP_WIN 4
#define OC_WIN 4
#define X_AUX 16
#define PF_B 4
#define LATE_J 9
#define FW_PAIRS 8
#define C_PAIRS 3
#define X_PUSH 5
#define X_WIN 2
constexpr int NR   = 8;
constexpr int NGM  = 16;
constexpr int NAX  = 16;
constexpr int NWG  = 8 * (NGM + NAX);
constexpr int NDIM = 4096;
constexpr int KS   = 512;
constexpr u64 DEADLINE = 100000000ull;
constexpr int NX      = 16 - X_PUSH;
constexpr u32 NBUF    = 2;

constexpr u32 OFF_CTR   = 0;
constexpr u32 OFF_TMO   = 32768;
constexpr u32 OFF_ELECT = 32768 + 1024;
constexpr u32 OFF_INVD  = 32768 + 4096;
constexpr u32 OFF_XFLAG = 65536;
constexpr u32 OFF_RFLAG = 65536 + 16384;
constexpr u32 OFF_OFLAG = 65536 + 24576;
constexpr u32 OFF_YFLAG = 98304;
constexpr u32 OFF_ZFLAG = 131072;
constexpr u32 OFF_FFLAG = 163840;
constexpr u32 OFF_STAMP = 524288;
constexpr u32 CTRL_BYTES = 1u << 20;
constexpr u32 STR_BYTES = 262144;
constexpr u32 STR_PITCH = STR_BYTES + 4096;
constexpr u32 O_XB = 8 * STR_PITCH;
constexpr u32 O_XS = O_XB + (4u << 20);
constexpr u32 O_FW = O_XS + (2u << 20);
constexpr u32 O_OS = O_FW + (512u << 10);
constexpr u32 BUF_BYTES = O_OS + (4u << 20);
constexpr u32 WS_NEED   = CTRL_BYTES + NBUF * BUF_BYTES;

struct Args {
    const unsigned char* x;
    const unsigned char* w;
    unsigned char* myws;
    int rank;
    int pad;
    unsigned char* rot[NR];
    const float* sx;
    const float* sw;
    float* out;
};

#define LIKELY(c) __builtin_expect(!!(c), 1)
#define RSRC(p, bytes) __builtin_amdgcn_make_buffer_rsrc((void*)(p), 0, (int)(bytes), 0x00020000)
#define AUX_SYS 17
#define AUX_AGT 16
#define CAT8(lo4, hi4) __builtin_shufflevector(__builtin_bit_cast(i32x4, lo4), __builtin_bit_cast(i32x4, hi4), \
                                               0, 1, 2, 3, 4, 5, 6, 7)

__device__ __forceinline__ u32 ld_sys(const unsigned char* p) {
    return __hip_atomic_load((const u32*)p, __ATOMIC_RELAXED, __HIP_MEMORY_SCOPE_SYSTEM);
}
__device__ __forceinline__ void st_sys(unsigned char* p, u32 v) {
    __hip_atomic_store((u32*)p, v, __ATOMIC_RELAXED, __HIP_MEMORY_SCOPE_SYSTEM);
}
__device__ __forceinline__ u32 xcc_id() {
    return (u32)__builtin_amdgcn_s_getreg((3 << 11) | 20) & 0xFu;
}
__device__ __forceinline__ void wait_vm0() { asm volatile("s_waitcnt vmcnt(0)" ::: "memory"); }
__device__ __forceinline__ void wg_barrier() {
    asm volatile("s_waitcnt lgkmcnt(0)" ::: "memory");
    __builtin_amdgcn_s_barrier();
    asm volatile("" ::: "memory");
}

#define STAMP(i) do {} while (0)
#define LD16(r, off, aux) __builtin_amdgcn_raw_buffer_load_b128(r, off, 0, aux)
#define ST16(v, r, off, aux) __builtin_amdgcn_raw_buffer_store_b128(v, r, off, 0, aux)

template <int SLEEP>
__device__ __forceinline__ bool poll_flags(const unsigned char* p, unsigned char* tmo, u32 call) {
    const u64 t0 = __builtin_amdgcn_s_memrealtime();
    for (u32 it = 1;; ++it) {
        u32 v = ld_sys(p);
        if (__all(v >= call)) { asm volatile("" ::: "memory"); return true; }
        if ((it & 15u) == 0u) {
            u32 t = __hip_atomic_load((u32*)tmo, __ATOMIC_RELAXED, __HIP_MEMORY_SCOPE_AGENT);
            if (t == call || __builtin_amdgcn_s_memrealtime() - t0 > DEADLINE) {
                __hip_atomic_store((u32*)tmo, call, __ATOMIC_RELAXED, __HIP_MEMORY_SCOPE_AGENT);
                return false;
            }
        }
        __builtin_amdgcn_s_sleep(SLEEP);
    }
}

__device__ __forceinline__ void elect_invalidate(unsigned char* myws, u32 call, u32 xcc, int lane) {
    u32 old = 0;
    if (lane == 0)
        old = __hip_atomic_fetch_max((u32*)(myws + OFF_ELECT + xcc * 128), call,
                                     __ATOMIC_RELAXED, __HIP_MEMORY_SCOPE_AGENT);
    old = __builtin_amdgcn_readfirstlane(old);
    if (old < call) {
        __builtin_amdgcn_fence(__ATOMIC_ACQUIRE, "");
        wait_vm0();
        if (lane == 0) st_sys(myws + OFF_INVD + xcc * 128, call);
    }
}

template <int WIN>
__device__ __forceinline__ void touch_chunks(const unsigned char* base, u32 off, u32 step, int n) {
    u32x4 dsc;
    dsc.x = __builtin_amdgcn_readfirstlane((u32)(size_t)base);
    dsc.y = __builtin_amdgcn_readfirstlane((u32)((size_t)base >> 32) & 0xffffu);
    dsc.z = STR_BYTES;
    dsc.w = 0x00020000u;
#pragma unroll 2
    for (int c = 0; c < n; ++c) {
        u32 keep;
        asm volatile("s_mov_b32 %0, m0\n\ts_mov_b32 m0, 0\n\ts_waitcnt vmcnt(%3)\n\ts_nop 2\n\t"
                     "buffer_load_dwordx4 %1, %2, 0 offen lds\n\ts_mov_b32 m0, %0"
                     : "=&s"(keep) : "v"(off), "s"(dsc), "i"(WIN - 1) : "memory");
        off += step;
    }
}

__global__ void __launch_bounds__(512, 2) fused_gemm_rs(Args a) {
    __shared__ __attribute__((aligned(16))) float red[2 * 8 * 2048 + 4];
    const int lane = threadIdx.x & 63;
    const int wv = __builtin_amdgcn_readfirstlane(threadIdx.x >> 6);
    const int xg = blockIdx.x & 7;
    const int idx = ((blockIdx.x >> 3) + 30) & 31;
    const int bid = idx * 8 + xg;
    const int rank = a.rank;
    unsigned char* myws = a.myws;
    const unsigned char* xin = a.x;
    const unsigned char* win = a.w;
    unsigned char* pws = a.rot[((idx - NGM) >> 1) & 7];
    asm volatile("" :: "s"(rank), "s"(myws), "s"(xin), "s"(win), "s"(pws));
    STAMP(0);

    unsigned char* ctr = myws + OFF_CTR + bid * 64;
    unsigned char* tmo = myws + OFF_TMO;
    const u32 lo = lane * 16;
    const int g = rank >> 2, gi4 = rank & 3;
    const float qnan = __builtin_nanf("");
    const f32x4 nan4 = {qnan, qnan, qnan, qnan};

    if (LIKELY(idx >= NGM)) {
        const int hi = idx - NGM;
        const int h = hi & 1;
        const int peer = (rank + 1 + (hi >> 1)) & 7;
        const int blk = h * 8 + xg;
        if (wv > 3) return;
        unsigned char* myctr = ctr + wv * 4;
        if (LIKELY(wv == 0 || wv == 3)) {
            if (wv == 0 && peer == rank) return;
            const int bo_ = 4 * (1 - (peer >> 2)) + (blk >> 2), bb_ = 4 * (peer & 3) + (blk & 3);
            __amdgpu_buffer_rsrc_t rx = RSRC(xin + (size_t)(bo_ * 512 + bb_ * 32) * KS, 32 * KS);
            const u32 o = (u32)(lane & 31) * KS + (lane >> 5) * 16;
            const u32 cv = ld_sys(myctr);
            __builtin_amdgcn_sched_barrier(0);
            if (wv == 0) {
                u32x4 v[X_PUSH];
#pragma unroll
                for (int c = 0; c < X_PUSH; ++c) v[c] = LD16(rx, o + c * 32, 0);
                __builtin_amdgcn_sched_barrier(0);
                const u32 call = __builtin_amdgcn_readfirstlane(cv) + 1u;
                __amdgpu_buffer_rsrc_t rd = RSRC(pws + CTRL_BYTES + (call % NBUF) * BUF_BYTES + O_XB
                                                 + (rank * 32 + 16 + blk) * 16384, 16384);
#pragma unroll
                for (int c = 0; c < X_PUSH; ++c) {
                    if (c >= X_WIN) asm volatile("s_waitcnt vmcnt(%0)" :: "i"(X_WIN - 1) : "memory");
                    ST16(v[c], rd, c * 1024 + lo, AUX_SYS);
                }
                wait_vm0();
                STAMP(2);
                if (lane == 0) {
                    st_sys(pws + OFF_XFLAG + (blk * 8 + rank) * 128, call);
                    st_sys(myctr, call);
                }
            } else {
                u32x4 v[16];
#pragma unroll
                for (int c = 0; c < 16; ++c) {
                    v[c] = LD16(rx, o + ((X_PUSH + c) & 15) * 32, 0);
                    __builtin_amdgcn_sched_barrier(0);
                }
                __builtin_amdgcn_sched_barrier(0);
                const u32 call = __builtin_amdgcn_readfirstlane(cv) + 1u;
                __amdgpu_buffer_rsrc_t rl = RSRC(myws + CTRL_BYTES + (call % NBUF) * BUF_BYTES + O_XS
                                                 + (bo_ * 16 + bb_) * 16384, 16384);
                unsigned char* ows = a.rot[(bo_ - rank - 1) & 7];
#pragma unroll
                for (int c = 0; c < 16; ++c) ST16(v[c], rl, ((X_PUSH + c) & 15) * 1024 + lo, AUX_SYS);
                asm volatile("s_waitcnt vmcnt(%0)" :: "i"(X_PUSH) : "memory");
                if (lane == 0 && peer != rank) st_sys(pws + OFF_ZFLAG + ((16 + blk) * 8 + rank) * 128, call);
                asm volatile("s_waitcnt vmcnt(1)" ::: "memory");
                if (peer == rank) wait_vm0();
                STAMP(2);
                if (lane == 0) {
                    if (bo_ != rank) st_sys(ows + OFF_ZFLAG + (bb_ * 8 + rank) * 128, call);
                    st_sys(myctr, call);
                }
            }
            return;
        }
        if (peer == rank) return;
        if (wv == 2) __builtin_amdgcn_s_sleep(64);
        const u32 call = __builtin_amdgcn_readfirstlane(ld_sys(myctr)) + 1u;
        if (lane == 0) st_sys(myctr, call);
        if (wv == 2) {
            elect_invalidate(myws, call, xcc_id(), lane);
            return;
        }
        const bool cross = (peer >> 2) != g;
        const u32 bufo = CTRL_BYTES + (call % NBUF) * BUF_BYTES;
        const unsigned char* wbase = pws + bufo + xg * STR_PITCH;
        const unsigned char* fw = wbase + STR_BYTES + (lane & 15) * 4;
        if (lane >= 16) fw = myws + OFF_INVD + xcc_id() * 128;
        if (lane >= 24) fw = myws + OFF_ZFLAG + (blk * 8 + peer) * 128;
        const int fo = 4 * (1 - g) + (blk >> 2), fb = 4 * gi4 + (blk & 3);
        __amdgpu_buffer_rsrc_t rs1 = RSRC(pws + bufo + O_XS + (rank * 16 + blk) * 16384, 16384);
        __amdgpu_buffer_rsrc_t rs0 = RSRC(pws + bufo + O_XS + (fo * 16 + fb) * 16384, 16384);
        __amdgpu_buffer_rsrc_t rd1 = RSRC(myws + bufo + O_XB + (peer * 32 + blk) * 16384, 16384);
        __amdgpu_buffer_rsrc_t rd0 = RSRC(myws + bufo + O_XB + (peer * 32 + 16 + blk) * 16384, 16384);
        const int te = 5 + ((gi4 - peer - xg) & 3);
        const int ue = te < 7 ? 7 * g + te : 7 + te;
        __amdgpu_buffer_rsrc_t rs2 = RSRC(wbase + ue * 16384 + h * FW_PAIRS * 1024, FW_PAIRS * 1024);
        __amdgpu_buffer_rsrc_t rd2 = RSRC(myws + bufo + O_FW + ((peer & 3) * 8 + xg) * 16384 + h * FW_PAIRS * 1024, FW_PAIRS * 1024);
        __amdgpu_buffer_rsrc_t rdo = RSRC(a.out + (size_t)((peer & 3) * 128 + h * 64) * NDIM + (xg * 16 + 7 * (1 - g)) * 32,
                                          63 * NDIM * 4 + 7 * 128);
        const u32 oo = (u32)(lane >> 3) * NDIM * 4 + (lane & 7) * 16;
        u64 gm = (u64)(a.rot[(4 * g + (lane & 3) - rank - 1) & 7]
                       + OFF_FFLAG + ((((peer & 3) * 8 + xg) * 4 + gi4) * 2 + h) * 128);
        asm volatile("" : "+v"(gm));
        bool good = poll_flags<2>(myws + OFF_ZFLAG + ((16 + blk) * 8 + peer) * 128, tmo, call);
        STAMP(3);
        if (good) {
            constexpr int N0 = NX, N1 = NX + 16, N2 = NX + 16 + FW_PAIRS;
#define ITEM_LD(i) ((i) < N0 ? LD16(rs0, lo + (X_PUSH + (i)) * 1024, AUX_SYS) : (i) < N1 ? LD16(rs1, lo + ((i) - N0) * 1024, AUX_SYS) \
                             : LD16(rs2, lo + ((i) - N1) * 1024, AUX_SYS))
#define ITEM_ST(v, i) do { if ((i) < N0) ST16(v, rd0, lo + (X_PUSH + (i)) * 1024, AUX_SYS); \
                           else if ((i) < N1) ST16(v, rd1, lo + ((i) - N0) * 1024, AUX_SYS); \
                           else ST16(v, rd2, lo + ((i) - N1) * 1024, AUX_SYS); } while (0)
            static_assert(8 % OC_WIN == 0, "rounds align with tiles");
            static_assert(NX >= 2 * CP_WIN, "the first x block covers the flag check");
            u32x4 v[CP_WIN];
#pragma unroll
            for (int k = 0; k < CP_WIN; ++k) v[k] = ITEM_LD(k);
            u32 fv = 0;
#pragma unroll
            for (int i = 0; i < N0 - CP_WIN; ++i) {
                ITEM_ST(v[i % CP_WIN], i);
                if (i == N0 - 2 * CP_WIN) fv = ld_sys(fw);
                v[i % CP_WIN] = ITEM_LD(i + CP_WIN);
                __builtin_amdgcn_sched_barrier(0);
            }
            good = __all(fv >= call) || poll_flags<2>(fw, tmo, call);
            asm volatile("" ::: "memory");
            if (good) {
                if (lane == 0 && h == 0) st_sys(myws + OFF_RFLAG + (xg * 8 + peer) * 128, call);
#define COPY_STEPS(from, to, NE) \
                _Pragma("unroll") \
                for (int i = (from); i < (to); ++i) { \
                    ITEM_ST(v[i % CP_WIN], i); \
                    if (i + CP_WIN < (NE)) v[i % CP_WIN] = ITEM_LD(i + CP_WIN); \
                    else touch_chunks<64>(wbase, 7 * g * 16384 + (2 * (i + CP_WIN - (NE)) + h) * 1024 + lo, 0, 1); \
                    __builtin_amdgcn_sched_barrier(0); \
                    if (i == N0 + CP_WIN - 1 || i == N1 + CP_WIN - 1) { \
                          \
                        asm volatile("s_waitcnt vmcnt(%0)" :: "i"(2 * CP_WIN) : "memory"); \
                        if (lane == 0) st_sys(myws + OFF_YFLAG + (((i < N1 ? 16 : 0) + blk) * 8 + peer) * 128, call); \
                    } \
                }
                COPY_STEPS(N0 - CP_WIN, N1 - CP_WIN, N2)
                if (cross) { COPY_STEPS(N1 - CP_WIN, N2, N2) } else { COPY_STEPS(N1 - CP_WIN, N1, N1) }
                STAMP(2);
                touch_chunks<PF_WIN>(wbase, 7 * g * 16384 + (2 * CP_WIN + h) * 1024 + lo, 2048, PF_WIN);
                asm volatile("s_waitcnt vmcnt(%0)" :: "i"(PF_WIN) : "memory");
                if (cross) {
                    if (lane < 4)
                        __hip_atomic_store((__attribute__((address_space(1))) u32*)gm, call, __ATOMIC_RELAXED, __HIP_MEMORY_SCOPE_SYSTEM);
                } else if (lane == 0) {
                    st_sys(myws + OFF_YFLAG + (blk * 8 + peer) * 128, call);
                }
                touch_chunks<PF_WIN>(wbase, 7 * g * 16384 + (2 * (CP_WIN + PF_WIN) + h) * 1024 + lo, 2048,
                                     40 - CP_WIN - 2 * PF_WIN);
                const int sl = (peer - xg - ((lane >> 1) & 3)) & 3;
                const unsigned char* of = myws + OFF_OFLAG + (((peer & 3) * 2 + h) * 8 + xg) * 128;
                const unsigned char* nf = cross ? of + (lane & 7) * 4
                                                : myws + OFF_FFLAG + (((sl * 8 + xg) * 4 + (peer & 3)) * 2 + (lane & 1)) * 128;
                u32 fl;
                asm volatile("global_load_dword %0, %1, off sc0 sc1" : "=v"(fl) : "v"(nf) : "memory");
                touch_chunks<PF_WIN + 1>(wbase, 7 * g * 16384 + (2 * (40 - PF_WIN) + h) * 1024 + lo, 2048, PF_WIN);
                asm volatile("s_waitcnt vmcnt(%1)" : "+v"(fl) : "i"(PF_WIN) : "memory");
                STAMP(5);
                if (!cross) {
                    if (h == 0) {
                        touch_chunks<PF_B>(wbase, (7 * g + 5) * 16384 + lo, 1024, 32);
                        touch_chunks<PF_B>(wbase, 14 * 16384 + lo, 1024, 32);
                    } else {
                        good = __all(fl >= call) || poll_flags<2>(nf, tmo, call);
                        if (good) {
                            for (int t = 0; t < 4; ++t)
                                touch_chunks<PF_B>(pws + bufo + O_FW + ((((peer - xg - t) & 3) * 8 + xg) * 16384), lo, 1024, 2 * FW_PAIRS);
                        }
                    }
                    wait_vm0();
                    STAMP(4);
                    return;
                }
                const unsigned char* osrc = pws + bufo + O_OS + ((gi4 * 2 + h) * 8 + xg) * 65536;
                int done = 0;
                u32x4 w[OC_WIN];
#pragma unroll
                for (int k = 0; k < OC_WIN; ++k) w[k] = u32x4{0u, 0u, 0u, 0u};
                while (done < 7) {
                    int n = __builtin_ctzll(~(__ballot(fl >= call) & 0x7full));
                    if (n <= done) {
                        good = poll_flags<2>(of + done * 4, tmo, call);
                        if (!good) break;
                        n = done + 1;
                    }
                    asm volatile("" ::: "memory");
                    for (int c = done * 8 - OC_WIN; c < n * 8; c += OC_WIN) {
                        const int n2 = __builtin_ctzll(~(__ballot(fl >= call) & 0x7full));
                        n = n2 > n ? n2 : n;
                        asm volatile("" ::: "memory");
                        fl = ld_sys(nf);
                        __amdgpu_buffer_rsrc_t rso = RSRC(osrc, n * 8192);
#pragma unroll
                        for (int k = 0; k < OC_WIN; ++k) {
                            const int cc = c + k;
                            const u32 so = cc >= done * 8 ? (cc & 7) * (8 * NDIM * 4) + (cc >> 3) * 128 : 0x7ff00000u;
                            ST16(w[k], rdo, oo + so, AUX_AGT);
                            w[k] = LD16(rso, (cc + OC_WIN) * 1024 + lo, AUX_SYS);
                            __builtin_amdgcn_sched_barrier(0);
                        }
                    }
                    done = n;
                }
                wait_vm0();
                STAMP(4);
            }
        }
        if (!good && cross) {
            for (int c = 0; c < 56; ++c)
                ST16(__builtin_bit_cast(u32x4, nan4), rdo, oo + (c & 7) * (8 * NDIM * 4) + (c >> 3) * 128, 0);
        }
        return;
    }

    const int gi = idx;
    const bool ownrows = gi < 8;
    const int nt = ownrows ? 9 : 7;
    const u32 xcc = xcc_id();

    if (wv == 4) {
        if (gi < 14 && !(gi & 1)) {
            u32 bo = CTRL_BYTES + (lane & 1) * BUF_BYTES;
            const unsigned char* t = pws + OFF_XFLAG + (xg * 8 + rank) * 128 + 64;
            if ((lane >> 1) == 1) t = pws + bo + O_XB + (rank * 32 + 16 + xg) * 16384;
            if ((lane >> 1) == 2) t = myws + bo + O_XS + (gi * 8 + xg) * 16384;
            u32 v = 0;
            if (lane < 6) v = ld_sys(t);
            asm volatile("" :: "v"(v));
            bo = CTRL_BYTES + ((__builtin_amdgcn_readfirstlane(ld_sys(ctr)) + 1u) % NBUF) * BUF_BYTES;
            t = pws + bo + O_XS + (lane & 1) * ((2u << 20) - 64);
            if (lane == 2) t = myws + bo + O_XB + (rank * 32 + xg) * 16384;
            if (lane == 3) t = pws + bo + xg * STR_PITCH + STR_BYTES + 256;
            if (lane == 4) t = pws + bo + xg * STR_PITCH - 256;
            if (lane == 5) t = win + (size_t)(xg * 64) * NDIM;
            if (lane < 6) v = ld_sys(t);
            asm volatile("" :: "v"(v));
        }
    } else {
        __builtin_amdgcn_s_sleep(64);
    }
    u32 call;
    if (LIKELY(wv < 4)) {
        const int cq = lane & 7;
        const int G = xg * 4 + wv;
        __amdgpu_buffer_rsrc_t rw = RSRC(win, KS * NDIM);
        const u32 o = (u32)(G * 16) * NDIM + gi * 256 + lane * 4;
        u32 d[16];
#pragma unroll
        for (int j = 0; j < 16; ++j) d[j] = __builtin_amdgcn_raw_buffer_load_b32(rw, o + j * NDIM, 0, 0);
        call = __builtin_amdgcn_readfirstlane(ld_sys(ctr)) + 1u;
        u32x4 q[4];
#pragma unroll
        for (int u = 0; u < 4; ++u) {
            u32 t0 = __builtin_amdgcn_perm(d[4 * u + 1], d[4 * u + 0], 0x05010400u);
            u32 t1 = __builtin_amdgcn_perm(d[4 * u + 1], d[4 * u + 0], 0x07030602u);
            u32 t2 = __builtin_amdgcn_perm(d[4 * u + 3], d[4 * u + 2], 0x05010400u);
            u32 t3 = __builtin_amdgcn_perm(d[4 * u + 3], d[4 * u + 2], 0x07030602u);
            q[0][u] = __builtin_amdgcn_perm(t2, t0, 0x05040100u);
            q[1][u] = __builtin_amdgcn_perm(t2, t0, 0x07060302u);
            q[2][u] = __builtin_amdgcn_perm(t3, t1, 0x05040100u);
            q[3][u] = __builtin_amdgcn_perm(t3, t1, 0x07060302u);
        }
        __amdgpu_buffer_rsrc_t rd = RSRC(myws + CTRL_BYTES + (call % NBUF) * BUF_BYTES + (gi >> 1) * STR_PITCH
                                         + (gi & 1) * 131072, 131072);
        const u32 so = (u32)(lane >> 3) * 16384 + (G >> 1) * 1024 + ((G & 1) * 32 + cq) * 16;
#pragma unroll
        for (int c = 0; c < 4; ++c) ST16(q[c], rd, so + c * 128, AUX_SYS);
    } else {
        call = __builtin_amdgcn_readfirstlane(ld_sys(ctr)) + 1u;
        if (wv == 7) elect_invalidate(myws, call, xcc, lane);
    }
    const float scale = a.sx[0] * a.sw[0];
    wait_vm0();
    wg_barrier();
    STAMP(2);
    int* okw = (int*)&red[2 * 8 * 2048];
    const u32 bufo = CTRL_BYTES + (call % NBUF) * BUF_BYTES;
    if (wv == 0) {
        if (lane == 0) {
            st_sys(myws + bufo + (gi >> 1) * STR_PITCH + STR_BYTES + ((gi & 1) * 8 + xg) * 4, call);
            st_sys(ctr, call);
        }
    } else if (wv == 7) {
        const int l = lane & 31;
        int src = l & 7;
        if (src == rank) src = (rank + 1) & 7;
        const unsigned char* f = myws + OFF_YFLAG + (gi * 16 + (l & 8) + src) * 128;
        if (!ownrows && l >= 16) f = myws + OFF_XFLAG + ((gi - 8) * 16 + (l & 8) + src) * 128;
        __builtin_amdgcn_s_sleep(64);
        bool ok = poll_flags<8>(f, tmo, call);
        if (lane == 0) *okw = ok ? 1 : 0;
    }
    wg_barrier();
    STAMP(3);
    int ok = __builtin_amdgcn_readfirstlane(*okw);

    const int fq = 4 * (1 - g) + ((gi - 8) >> 1);
    const bool ownk = (wv == rank);
    const int row0 = ownrows ? rank * 512 + gi * 64 : fq * 512 + gi4 * 128 + (gi & 1) * 64;
    __amdgpu_buffer_rsrc_t rx = RSRC(ownk ? xin + (size_t)row0 * KS
                                          : myws + bufo + O_XB + (wv * 32 + gi * 2) * 16384, ok ? 32768 : 0);
    const u32 xo = ownk ? (u32)(lane & 31) * KS + (lane >> 5) * 16 : lo;
    const u32 xstep = ownk ? 32u : 1024u;
    u32x4 xf[32];
    if (wv == 7 && ok) {
        const unsigned char* f = myws + OFF_INVD + xcc * 128;
        if (lane < 8 && lane != rank) f = myws + OFF_RFLAG + (xg * 8 + lane) * 128;
        if (lane >= 8 && lane < 24) f = myws + bufo + xg * STR_PITCH + STR_BYTES + (lane - 8) * 4;
        bool o2 = poll_flags<8>(f, tmo, call);
        if (o2) o2 = __hip_atomic_load((u32*)tmo, __ATOMIC_RELAXED, __HIP_MEMORY_SCOPE_AGENT) != call;
        if (lane == 0) okw[1] = o2 ? 1 : 0;
    }
#pragma unroll
    for (int i = 0; i < 32; ++i)
        xf[i] = __builtin_amdgcn_raw_buffer_load_b128(rx, xo + (i >> 4) * 16384, (i & 15) * xstep, X_AUX);
    wg_barrier();
    if (ok) ok = __builtin_amdgcn_readfirstlane(okw[1]);

    const int rrow = wv * 8 + (lane >> 3);
    const int re = lane & 7;
    __amdgpu_buffer_rsrc_t ro = ownrows ? RSRC(a.out + (size_t)(gi * 64) * NDIM + xg * 512, 63 * NDIM * 4 + 2048)
                                        : RSRC(myws + bufo + O_OS + ((gi - 8) * 8 + xg) * 65536, 65536);
    const u32 out_off = ownrows ? (u32)rrow * NDIM * 4 + re * 16 : (u32)wv * 1024 + lo;
    if (!ok) {
        if (ownrows)
            for (int t = 0; t < 9; ++t)
                ST16(__builtin_bit_cast(u32x4, nan4), ro, out_off + (t < 7 ? 7 * g + t : 7 + t) * 128, 0);
        return;
    }

    auto tile_base = [&](int t) -> const unsigned char* {
        const bool fwd = (wv >> 2) != g && t >= 5;
        const int src = fwd ? 4 * g + ((t - 5 + wv + xg) & 3) : wv;
        const u32 off = fwd ? O_FW + ((wv & 3) * 8 + xg) * 16384 : xg * STR_PITCH + (t < 7 ? 7 * g + t : 7 + t) * 16384;
        return a.rot[(src - rank - 1) & 7] + bufo + off;
    };
    unsigned char* oflag = a.rot[(fq - rank - 1) & 7] + OFF_OFLAG + ((gi4 * 2 + (gi & 1)) * 8 + xg) * 128;

    const unsigned char* wsrc = a.rot[(wv - rank - 1) & 7] + bufo + xg * STR_PITCH;
    static_assert(FW_PAIRS >= C_PAIRS && FW_PAIRS >= CP_WIN, "");
    const unsigned char* cur = tile_base(0);
    u32x4 wf[8];
    {
        __amdgpu_buffer_rsrc_t r0 = RSRC(cur, 16384);
#pragma unroll
        for (int k = 0; k < 2 * C_PAIRS; ++k) wf[k] = LD16(r0, lo + k * 1024, 0);
    }

    const int m = lane & 31, hh = lane >> 5;
    const u32 wr_base = wv * 8192 + m * 128;
    const u32 rd_base = rrow * 128 + ((re ^ (rrow & 7)) * 16);
    const int one = 0x7F7F7F7F;

#pragma unroll 1
    for (int j = 0; j < nt; ++j) {
        const unsigned char* nxt = tile_base(j + 1 < 9 ? j + 1 : 8);
        __amdgpu_buffer_rsrc_t rc = RSRC(cur, ok ? 16384 : 0);
        __amdgpu_buffer_rsrc_t rn = RSRC(nxt, (ok && j + 1 < nt) ? 16384 : 0);
        __amdgpu_buffer_rsrc_t rdir = RSRC(wsrc + (j < 7 ? 7 * g + j : 7 + j) * 16384, ok ? 16384 : 0);
        f32x16 acc0 = {0.f, 0.f, 0.f, 0.f, 0.f, 0.f, 0.f, 0.f, 0.f, 0.f, 0.f, 0.f, 0.f, 0.f, 0.f, 0.f};
        f32x16 acc1 = acc0;
#pragma unroll
        for (int t = 0; t < 8; ++t) {
            const int s = (t & 3) * 2, sn = ((t + C_PAIRS) & 3) * 2;
            asm volatile("" :: "v"(wf[s]), "v"(wf[s + 1]) : "memory");
            if (C_PAIRS == 3 && j >= LATE_J) asm volatile("s_waitcnt vmcnt(2)" ::: "memory");
            const int p = t + C_PAIRS;
            wf[sn]     = LD16(p < FW_PAIRS ? rc : p < 8 ? rdir : rn, (p & 7) * 2048 + lo, 0);
            wf[sn + 1] = LD16(p < FW_PAIRS ? rc : p < 8 ? rdir : rn, (p & 7) * 2048 + 1024 + lo, 0);
            __builtin_amdgcn_sched_barrier(0);
            i32x8 A = CAT8(wf[s], wf[s + 1]);
            acc0 = __builtin_amdgcn_mfma_scale_f32_32x32x64_f8f6f4(A, CAT8(xf[2 * t], xf[2 * t + 1]), acc0,
                                                                   1, 1, 0, one, 0, one);
            acc1 = __builtin_amdgcn_mfma_scale_f32_32x32x64_f8f6f4(A, CAT8(xf[16 + 2 * t], xf[17 + 2 * t]), acc1,
                                                                   1, 1, 0, one, 0, one);
            asm volatile("" : "+v"(acc0), "+v"(acc1));
        }
        cur = nxt;
        char* slot = (char*)red + (j & 1) * 65536;
#pragma unroll
        for (int b = 0; b < 4; ++b) {
            f32x4 p = {acc0[b], acc0[4 + b], acc0[8 + b], acc0[12 + b]};
            *(f32x4*)(slot + wr_base + (((hh * 4 + b) ^ (m & 7)) * 16)) = p;
        }
#pragma unroll
        for (int b = 0; b < 4; ++b) {
            f32x4 p = {acc1[b], acc1[4 + b], acc1[8 + b], acc1[12 + b]};
            *(f32x4*)(slot + wr_base + 4096 + (((hh * 4 + b) ^ (m & 7)) * 16)) = p;
        }
        if (j == 3 && wv == 7) {
            bool o3 = poll_flags<2>(myws + OFF_FFLAG + ((((lane >> 3) & 3) * 8 + xg) * 8 + (lane & 7)) * 128, tmo, call);
            if (lane == 0) okw[2] = o3 ? 1 : 0;
        }
        asm volatile("s_waitcnt vmcnt(%0)" :: "i"(2 * C_PAIRS) : "memory");
        wg_barrier();
        if (j == 3) ok = __builtin_amdgcn_readfirstlane(okw[2]);
        if (!ownrows && ownk && j > 0 && lane == 0) st_sys(oflag + (j - 1) * 4, call);
        f32x4 sv[8];
#pragma unroll
        for (int w = 0; w < 8; ++w) sv[w] = *(const f32x4*)(slot + rd_base + w * 8192);
        f32x4 s = ((sv[0] + sv[1]) + (sv[2] + sv[3])) + ((sv[4] + sv[5]) + (sv[6] + sv[7]));
        f32x4 r;
#pragma unroll
        for (int c = 0; c < 4; ++c) {
            float y = s[c] * scale;
            float z = fminf(fmaxf(y, -60.f), 60.f);
            r[c] = y * __builtin_amdgcn_rcpf(1.f + __expf(-z));
        }
        if (!ok) r = nan4;
        const u32 toff = ownrows ? (j < 7 ? 7 * g + j : 7 + j) * 128 : j * 8192;
        __builtin_amdgcn_raw_buffer_store_b128(__builtin_bit_cast(u32x4, r), ro, out_off, toff, AUX_SYS);
    }
    if (!ownrows) {
        wait_vm0();
        wg_barrier();
        if (ownk && lane == 0) st_sys(oflag + 6 * 4, call);
    }
    STAMP(7);
}

extern "C" void kernel_launch(int rank, int nranks, ncclComm_t comm,
                              hipStream_t stream,
                              void* const* d_in, const int* in_sizes, int n_in,
                              void* d_out, int out_size,
                              const int* in_dtypes, int out_dtype,
                              void* const* d_out_all,
                              void* const* d_ws_all, size_t ws_bytes) {
    (void)comm; (void)in_sizes; (void)in_dtypes; (void)out_dtype; (void)d_out_all;
    if (nranks != NR || n_in < 4 || (unsigned)rank >= (unsigned)NR || ws_bytes < WS_NEED ||
        out_size != 512 * NDIM) {
        fprintf(stderr, "kernel_launch: unexpected problem shape\n");
        return;
    }
    Args a;
    a.x = (const unsigned char*)d_in[0];
    a.w = (const unsigned char*)d_in[1];
    a.sx = (const float*)d_in[2];
    a.sw = (const float*)d_in[3];
    a.out = (float*)d_out;
    for (int d = 0; d < NR; ++d) a.rot[d] = (unsigned char*)d_ws_all[(rank + 1 + d) & 7];
    a.myws = (unsigned char*)d_ws_all[rank];
    a.pad = 0;
    a.rank = rank;
    hipLaunchKernelGGL(fused_gemm_rs, dim3(NWG), dim3(512), 0, stream, a);
    HIP_CHECK(hipGetLastError());
}
```

```cpp
#include <hip/hip_runtime.h>
#include <rccl/rccl.h>
#include <cstdio>
#include <cstdint>

#define HIP_CHECK(expr) do { hipError_t _e = (expr); if (_e != hipSuccess) { \
    fprintf(stderr, "HIP error %s at %s:%d\n", hipGetErrorString(_e), __FILE__, __LINE__); \
    return; } } while (0)

typedef unsigned int u32;
typedef unsigned long long u64;
typedef u32   u32x4  __attribute__((ext_vector_type(4)));
typedef int   i32x4  __attribute__((ext_vector_type(4)));
typedef int   i32x8  __attribute__((ext_vector_type(8)));
typedef float f32x4  __attribute__((ext_vector_type(4)));
typedef float f32x16 __attribute__((ext_vector_type(16)));

constexpr int NR   = 8;
constexpr int NGM  = 16;
constexpr int NAX  = 16;
constexpr int NWG  = 8 * (NGM + NAX);
constexpr int NDIM = 4096;
constexpr int KS   = 512;
constexpr u64 DEADLINE = 100000000ull;
constexpr int WIN     = 4;
constexpr int C_PAIRS = 3;
constexpr int X_PUSH  = 5;
constexpr int X_WIN   = 2;
constexpr int NX      = 16 - X_PUSH;
constexpr u32 NBUF    = 2;

constexpr u32 OFF_CTR   = 0;
constexpr u32 OFF_TMO   = 32768;
constexpr u32 OFF_ELECT = 32768 + 1024;
constexpr u32 OFF_INVD  = 32768 + 4096;
constexpr u32 OFF_XFLAG = 65536;
constexpr u32 OFF_RFLAG = 65536 + 16384;
constexpr u32 OFF_OFLAG = 65536 + 24576;
constexpr u32 OFF_YFLAG = 98304;
constexpr u32 OFF_ZFLAG = 131072;
constexpr u32 OFF_FFLAG = 163840;
constexpr u32 CTRL_BYTES = 1u << 20;
constexpr u32 STR_BYTES = 262144;
constexpr u32 STR_PITCH = STR_BYTES + 4096;
constexpr u32 O_XB = 8 * STR_PITCH;
constexpr u32 O_XS = O_XB + (4u << 20);
constexpr u32 O_FW = O_XS + (2u << 20);
constexpr u32 O_OS = O_FW + (512u << 10);
constexpr u32 BUF_BYTES = O_OS + (4u << 20);
constexpr u32 WS_NEED   = CTRL_BYTES + NBUF * BUF_BYTES;

struct Args {
    const unsigned char* x;
    const unsigned char* w;
    unsigned char* myws;
    int rank;
    int pad;
    unsigned char* rot[NR];
    const float* sx;
    const float* sw;
    float* out;
};

#define LIKELY(c) __builtin_expect(!!(c), 1)
#define RSRC(p, bytes) __builtin_amdgcn_make_buffer_rsrc((void*)(p), 0, (int)(bytes), 0x00020000)
#define AUX_SYS 17
#define AUX_AGT 16
#define CAT8(lo4, hi4) __builtin_shufflevector(__builtin_bit_cast(i32x4, lo4), __builtin_bit_cast(i32x4, hi4), \
                                               0, 1, 2, 3, 4, 5, 6, 7)

__device__ __forceinline__ u32 ld_sys(const unsigned char* p) {
    return __hip_atomic_load((const u32*)p, __ATOMIC_RELAXED, __HIP_MEMORY_SCOPE_SYSTEM);
}
__device__ __forceinline__ void st_sys(unsigned char* p, u32 v) {
    __hip_atomic_store((u32*)p, v, __ATOMIC_RELAXED, __HIP_MEMORY_SCOPE_SYSTEM);
}
__device__ __forceinline__ u32 xcc_id() {
    return (u32)__builtin_amdgcn_s_getreg((3 << 11) | 20) & 0xFu;
}
__device__ __forceinline__ void wait_vm0() { asm volatile("s_waitcnt vmcnt(0)" ::: "memory"); }
__device__ __forceinline__ void wg_barrier() {
    asm volatile("s_waitcnt lgkmcnt(0)" ::: "memory");
    __builtin_amdgcn_s_barrier();
    asm volatile("" ::: "memory");
}

#define LD16(r, off, aux) __builtin_amdgcn_raw_buffer_load_b128(r, off, 0, aux)
#define ST16(v, r, off, aux) __builtin_amdgcn_raw_buffer_store_b128(v, r, off, 0, aux)

template <int SLEEP>
__device__ __forceinline__ bool poll_flags(const unsigned char* p, unsigned char* tmo, u32 call) {
    const u64 t0 = __builtin_amdgcn_s_memrealtime();
    for (u32 it = 1;; ++it) {
        u32 v = ld_sys(p);
        if (__all(v >= call)) { asm volatile("" ::: "memory"); return true; }
        if ((it & 15u) == 0u) {
            u32 t = __hip_atomic_load((u32*)tmo, __ATOMIC_RELAXED, __HIP_MEMORY_SCOPE_AGENT);
            if (t == call || __builtin_amdgcn_s_memrealtime() - t0 > DEADLINE) {
                __hip_atomic_store((u32*)tmo, call, __ATOMIC_RELAXED, __HIP_MEMORY_SCOPE_AGENT);
                return false;
            }
        }
        __builtin_amdgcn_s_sleep(SLEEP);
    }
}

__device__ __forceinline__ void elect_invalidate(unsigned char* myws, u32 call, u32 xcc, int lane) {
    u32 old = 0;
    if (lane == 0)
        old = __hip_atomic_fetch_max((u32*)(myws + OFF_ELECT + xcc * 128), call,
                                     __ATOMIC_RELAXED, __HIP_MEMORY_SCOPE_AGENT);
    old = __builtin_amdgcn_readfirstlane(old);
    if (old < call) {
        __builtin_amdgcn_fence(__ATOMIC_ACQUIRE, "");
        wait_vm0();
        if (lane == 0) st_sys(myws + OFF_INVD + xcc * 128, call);
    }
}

template <int W>
__device__ __forceinline__ void touch_chunks(const unsigned char* base, u32 off, u32 step, int n) {
    u32x4 dsc;
    dsc.x = __builtin_amdgcn_readfirstlane((u32)(size_t)base);
    dsc.y = __builtin_amdgcn_readfirstlane((u32)((size_t)base >> 32) & 0xffffu);
    dsc.z = STR_BYTES;
    dsc.w = 0x00020000u;
#pragma unroll 2
    for (int c = 0; c < n; ++c) {
        u32 keep;
        asm volatile("s_mov_b32 %0, m0\n\ts_mov_b32 m0, 0\n\ts_waitcnt vmcnt(%3)\n\ts_nop 2\n\t"
                     "buffer_load_dwordx4 %1, %2, 0 offen lds\n\ts_mov_b32 m0, %0"
                     : "=&s"(keep) : "v"(off), "s"(dsc), "i"(W - 1) : "memory");
        off += step;
    }
}

__global__ void __launch_bounds__(512, 2) fused_gemm_rs(Args a) {
    __shared__ __attribute__((aligned(16))) float red[2 * 8 * 2048 + 4];
    const int lane = threadIdx.x & 63;
    const int wv = __builtin_amdgcn_readfirstlane(threadIdx.x >> 6);
    const int xg = blockIdx.x & 7;
    const int idx = ((blockIdx.x >> 3) + 30) & 31;
    const int bid = idx * 8 + xg;
    const int rank = a.rank;
    unsigned char* myws = a.myws;
    const unsigned char* xin = a.x;
    const unsigned char* win = a.w;
    unsigned char* pws = a.rot[((idx - NGM) >> 1) & 7];
    asm volatile("" :: "s"(rank), "s"(myws), "s"(xin), "s"(win), "s"(pws));

    unsigned char* ctr = myws + OFF_CTR + bid * 64;
    unsigned char* tmo = myws + OFF_TMO;
    const u32 lo = lane * 16;
    const int g = rank >> 2, gi4 = rank & 3;
    const float qnan = __builtin_nanf("");
    const f32x4 nan4 = {qnan, qnan, qnan, qnan};

    const int hi = idx - NGM;
    const int h = hi & 1;
    const int peer = (rank + 1 + (hi >> 1)) & 7;
    const int blk = h * 8 + xg;
    unsigned char* myctr = ctr + (wv & 3) * 4;
    {
        if (LIKELY(idx >= NGM && (wv == 3 || (wv == 0 && peer != rank)))) {
            const int bo_ = 4 * (1 - (peer >> 2)) + (blk >> 2), bb_ = 4 * (peer & 3) + (blk & 3);
            __amdgpu_buffer_rsrc_t rx = RSRC(xin + (size_t)(bo_ * 512 + bb_ * 32) * KS, 32 * KS);
            const u32 o = (u32)(lane & 31) * KS + (lane >> 5) * 16;
            const u32 cv = ld_sys(myctr);
            __builtin_amdgcn_sched_barrier(0);
            if (wv == 0) {
                u32x4 v[X_PUSH];
#pragma unroll
                for (int c = 0; c < X_PUSH; ++c) v[c] = LD16(rx, o + c * 32, 0);
                __builtin_amdgcn_sched_barrier(0);
                const u32 call = __builtin_amdgcn_readfirstlane(cv) + 1u;
                __amdgpu_buffer_rsrc_t rd = RSRC(pws + CTRL_BYTES + (call % NBUF) * BUF_BYTES + O_XB
                                                 + (rank * 32 + 16 + blk) * 16384, 16384);
#pragma unroll
                for (int c = 0; c < X_PUSH; ++c) {
                    if (c >= X_WIN) asm volatile("s_waitcnt vmcnt(%0)" :: "i"(X_WIN - 1) : "memory");
                    ST16(v[c], rd, c * 1024 + lo, AUX_SYS);
                }
                wait_vm0();
                if (lane == 0) {
                    st_sys(pws + OFF_XFLAG + (blk * 8 + rank) * 128, call);
                    st_sys(myctr, call);
                }
            } else {
                u32x4 v[16];
#pragma unroll
                for (int c = 0; c < 16; ++c) {
                    v[c] = LD16(rx, o + ((X_PUSH + c) & 15) * 32, 0);
                    __builtin_amdgcn_sched_barrier(0);
                }
                __builtin_amdgcn_sched_barrier(0);
                const u32 call = __builtin_amdgcn_readfirstlane(cv) + 1u;
                __amdgpu_buffer_rsrc_t rl = RSRC(myws + CTRL_BYTES + (call % NBUF) * BUF_BYTES + O_XS
                                                 + (bo_ * 16 + bb_) * 16384, 16384);
                unsigned char* ows = a.rot[(bo_ - rank - 1) & 7];
#pragma unroll
                for (int c = 0; c < 16; ++c) {
                    ST16(v[c], rl, ((X_PUSH + c) & 15) * 1024 + lo, AUX_SYS);
                    __builtin_amdgcn_sched_barrier(0);
                }
                asm volatile("s_waitcnt vmcnt(%0)" :: "i"(X_PUSH) : "memory");
                if (lane == 0 && peer != rank) st_sys(pws + OFF_ZFLAG + ((16 + blk) * 8 + rank) * 128, call);
                asm volatile("s_waitcnt vmcnt(1)" ::: "memory");
                if (peer == rank) wait_vm0();
                if (lane == 0) {
                    if (bo_ != rank) st_sys(ows + OFF_ZFLAG + (bb_ * 8 + rank) * 128, call);
                    st_sys(myctr, call);
                }
            }
            return;
        }
    }
    if (idx >= NGM) {
        if (peer == rank || wv == 0 || wv > 2) return;
        if (wv == 2) __builtin_amdgcn_s_sleep(64);
        const u32 call = __builtin_amdgcn_readfirstlane(ld_sys(myctr)) + 1u;
        if (lane == 0) st_sys(myctr, call);
        if (wv == 2) {
            elect_invalidate(myws, call, xcc_id(), lane);
            return;
        }
        const bool cross = (peer >> 2) != g;
        const u32 bufo = CTRL_BYTES + (call % NBUF) * BUF_BYTES;
        const unsigned char* wbase = pws + bufo + xg * STR_PITCH;
        const unsigned char* fw = wbase + STR_BYTES + (lane & 15) * 4;
        if (lane >= 16) fw = myws + OFF_INVD + xcc_id() * 128;
        if (lane >= 24) fw = myws + OFF_ZFLAG + (blk * 8 + peer) * 128;
        const int fo = 4 * (1 - g) + (blk >> 2), fb = 4 * gi4 + (blk & 3);
        __amdgpu_buffer_rsrc_t rs1 = RSRC(pws + bufo + O_XS + (rank * 16 + blk) * 16384, 16384);
        __amdgpu_buffer_rsrc_t rs0 = RSRC(pws + bufo + O_XS + (fo * 16 + fb) * 16384, 16384);
        __amdgpu_buffer_rsrc_t rd1 = RSRC(myws + bufo + O_XB + (peer * 32 + blk) * 16384, 16384);
        __amdgpu_buffer_rsrc_t rd0 = RSRC(myws + bufo + O_XB + (peer * 32 + 16 + blk) * 16384, 16384);
        const int te = 5 + ((gi4 - peer - xg) & 3);
        const int ue = te < 7 ? 7 * g + te : 7 + te;
        __amdgpu_buffer_rsrc_t rs2 = RSRC(wbase + ue * 16384 + h * 8192, 8192);
        __amdgpu_buffer_rsrc_t rd2 = RSRC(myws + bufo + O_FW + ((peer & 3) * 8 + xg) * 16384 + h * 8192, 8192);
        __amdgpu_buffer_rsrc_t rdo = RSRC(a.out + (size_t)((peer & 3) * 128 + h * 64) * NDIM + (xg * 16 + 7 * (1 - g)) * 32,
                                          63 * NDIM * 4 + 7 * 128);
        const u32 oo = (u32)(lane >> 3) * NDIM * 4 + (lane & 7) * 16;
        u64 gm = (u64)(a.rot[(4 * g + (lane & 3) - rank - 1) & 7]
                       + OFF_FFLAG + ((((peer & 3) * 8 + xg) * 4 + gi4) * 2 + h) * 128);
        asm volatile("" : "+v"(gm));
        bool good = poll_flags<2>(myws + OFF_ZFLAG + ((16 + blk) * 8 + peer) * 128, tmo, call);
        if (good) {
            constexpr int N0 = NX, N1 = NX + 16, N2 = NX + 24;
#define ITEM_LD(i) ((i) < N0 ? LD16(rs0, lo + (X_PUSH + (i)) * 1024, AUX_SYS) : (i) < N1 ? LD16(rs1, lo + ((i) - N0) * 1024, AUX_SYS) \
                             : LD16(rs2, lo + ((i) - N1) * 1024, AUX_SYS))
#define ITEM_ST(v, i) do { if ((i) < N0) ST16(v, rd0, lo + (X_PUSH + (i)) * 1024, AUX_SYS); \
                           else if ((i) < N1) ST16(v, rd1, lo + ((i) - N0) * 1024, AUX_SYS); \
                           else ST16(v, rd2, lo + ((i) - N1) * 1024, AUX_SYS); } while (0)
            static_assert(8 % WIN == 0, "rounds align with tiles");
            static_assert(NX >= 2 * WIN, "the first x block covers the flag check");
            u32x4 v[WIN];
#pragma unroll
            for (int k = 0; k < WIN; ++k) v[k] = ITEM_LD(k);
            u32 fv = 0;
#pragma unroll
            for (int i = 0; i < N0 - WIN; ++i) {
                ITEM_ST(v[i % WIN], i);
                if (i == N0 - 2 * WIN) fv = ld_sys(fw);
                v[i % WIN] = ITEM_LD(i + WIN);
                __builtin_amdgcn_sched_barrier(0);
            }
            good = __all(fv >= call) || poll_flags<2>(fw, tmo, call);
            asm volatile("" ::: "memory");
            if (good) {
                if (lane == 0 && h == 0) st_sys(myws + OFF_RFLAG + (xg * 8 + peer) * 128, call);
#define COPY_STEPS(from, to, NE) \
                _Pragma("unroll") \
                for (int i = (from); i < (to); ++i) { \
                    ITEM_ST(v[i % WIN], i); \
                    if (i + WIN < (NE)) v[i % WIN] = ITEM_LD(i + WIN); \
                    else touch_chunks<64>(wbase, 7 * g * 16384 + (2 * (i + WIN - (NE)) + h) * 1024 + lo, 0, 1); \
                    __builtin_amdgcn_sched_barrier(0); \
                    if (i == N0 + WIN - 1 || i == N1 + WIN - 1) { \
                          \
                        asm volatile("s_waitcnt vmcnt(%0)" :: "i"(2 * WIN) : "memory"); \
                        if (lane == 0) st_sys(myws + OFF_YFLAG + (((i < N1 ? 16 : 0) + blk) * 8 + peer) * 128, call); \
                    } \
                }
                COPY_STEPS(N0 - WIN, N1 - WIN, N2)
                if (cross) { COPY_STEPS(N1 - WIN, N2, N2) } else { COPY_STEPS(N1 - WIN, N1, N1) }
                touch_chunks<WIN>(wbase, 7 * g * 16384 + (2 * WIN + h) * 1024 + lo, 2048, WIN);
                asm volatile("s_waitcnt vmcnt(%0)" :: "i"(WIN) : "memory");
                if (cross) {
                    if (lane < 4)
                        __hip_atomic_store((__attribute__((address_space(1))) u32*)gm, call, __ATOMIC_RELAXED, __HIP_MEMORY_SCOPE_SYSTEM);
                } else if (lane == 0) {
                    st_sys(myws + OFF_YFLAG + (blk * 8 + peer) * 128, call);
                }
                touch_chunks<WIN>(wbase, 7 * g * 16384 + (2 * (WIN + WIN) + h) * 1024 + lo, 2048,
                                     40 - WIN - 2 * WIN);
                const int sl = (peer - xg - ((lane >> 1) & 3)) & 3;
                const unsigned char* of = myws + OFF_OFLAG + (((peer & 3) * 2 + h) * 8 + xg) * 128;
                const unsigned char* nf = cross ? of + (lane & 7) * 4
                                                : myws + OFF_FFLAG + (((sl * 8 + xg) * 4 + (peer & 3)) * 2 + (lane & 1)) * 128;
                u32 fl;
                asm volatile("global_load_dword %0, %1, off sc0 sc1" : "=v"(fl) : "v"(nf) : "memory");
                touch_chunks<WIN + 1>(wbase, 7 * g * 16384 + (2 * (40 - WIN) + h) * 1024 + lo, 2048, WIN);
                asm volatile("s_waitcnt vmcnt(%1)" : "+v"(fl) : "i"(WIN) : "memory");
                if (!cross) {
                    if (h == 0) {
                        touch_chunks<WIN>(wbase, (7 * g + 5) * 16384 + lo, 1024, 32);
                        touch_chunks<WIN>(wbase, 14 * 16384 + lo, 1024, 32);
                    } else {
                        good = __all(fl >= call) || poll_flags<2>(nf, tmo, call);
                        if (good) {
                            for (int t = 0; t < 4; ++t)
                                touch_chunks<WIN>(pws + bufo + O_FW + ((((peer - xg - t) & 3) * 8 + xg) * 16384), lo, 1024, 16);
                        }
                    }
                    wait_vm0();
                    return;
                }
                const unsigned char* osrc = pws + bufo + O_OS + ((gi4 * 2 + h) * 8 + xg) * 65536;
                int done = 0;
                u32x4 w[WIN];
#pragma unroll
                for (int k = 0; k < WIN; ++k) w[k] = u32x4{0u, 0u, 0u, 0u};
                while (done < 7) {
                    int n = __builtin_ctzll(~(__ballot(fl >= call) & 0x7full));
                    if (n <= done) {
                        good = poll_flags<2>(of + done * 4, tmo, call);
                        if (!good) break;
                        n = done + 1;
                    }
                    asm volatile("" ::: "memory");
                    for (int c = done * 8 - WIN; c < n * 8; c += WIN) {
                        const int n2 = __builtin_ctzll(~(__ballot(fl >= call) & 0x7full));
                        n = n2 > n ? n2 : n;
                        asm volatile("" ::: "memory");
                        fl = ld_sys(nf);
                        __amdgpu_buffer_rsrc_t rso = RSRC(osrc, n * 8192);
#pragma unroll
                        for (int k = 0; k < WIN; ++k) {
                            const int cc = c + k;
                            const u32 so = cc >= done * 8 ? (cc & 7) * (8 * NDIM * 4) + (cc >> 3) * 128 : 0x7ff00000u;
                            ST16(w[k], rdo, oo + so, AUX_AGT);
                            w[k] = LD16(rso, (cc + WIN) * 1024 + lo, AUX_SYS);
                            __builtin_amdgcn_sched_barrier(0);
                        }
                    }
                    done = n;
                }
                wait_vm0();
            }
        }
        if (!good && cross) {
            for (int c = 0; c < 56; ++c)
                ST16(__builtin_bit_cast(u32x4, nan4), rdo, oo + (c & 7) * (8 * NDIM * 4) + (c >> 3) * 128, 0);
        }
        return;
    }

    const int gi = idx;
    const bool ownrows = gi < 8;
    const int nt = ownrows ? 9 : 7;
    const u32 xcc = xcc_id();

    if (wv == 4) {
        if (gi < 14 && !(gi & 1)) {
            u32 bo = CTRL_BYTES + (lane & 1) * BUF_BYTES;
            const unsigned char* t = pws + OFF_XFLAG + (xg * 8 + rank) * 128 + 64;
            if ((lane >> 1) == 1) t = pws + bo + O_XB + (rank * 32 + 16 + xg) * 16384;
            if ((lane >> 1) == 2) t = myws + bo + O_XS + (gi * 8 + xg) * 16384;
            u32 v = 0;
            if (lane < 6) v = ld_sys(t);
            asm volatile("" :: "v"(v));
            bo = CTRL_BYTES + ((__builtin_amdgcn_readfirstlane(ld_sys(ctr)) + 1u) % NBUF) * BUF_BYTES;
            t = pws + bo + O_XS + (lane & 1) * ((2u << 20) - 64);
            if (lane == 2) t = myws + bo + O_XB + (rank * 32 + xg) * 16384;
            if (lane == 3) t = pws + bo + xg * STR_PITCH + STR_BYTES + 256;
            if (lane == 4) t = pws + bo + xg * STR_PITCH - 256;
            if (lane == 5) t = win + (size_t)(xg * 64) * NDIM;
            if (lane < 6) v = ld_sys(t);
            asm volatile("" :: "v"(v));
        }
    } else {
        __builtin_amdgcn_s_sleep(64);
    }
    u32 call;
    if (LIKELY(wv < 4)) {
        const int cq = lane & 7;
        const int G = xg * 4 + wv;
        __amdgpu_buffer_rsrc_t rw = RSRC(win, KS * NDIM);
        const u32 o = (u32)(G * 16) * NDIM + gi * 256 + lane * 4;
        u32 d[16];
#pragma unroll
        for (int j = 0; j < 16; ++j) d[j] = __builtin_amdgcn_raw_buffer_load_b32(rw, o + j * NDIM, 0, 0);
        call = __builtin_amdgcn_readfirstlane(ld_sys(ctr)) + 1u;
        u32x4 q[4];
#pragma unroll
        for (int u = 0; u < 4; ++u) {
            u32 t0 = __builtin_amdgcn_perm(d[4 * u + 1], d[4 * u + 0], 0x05010400u);
            u32 t1 = __builtin_amdgcn_perm(d[4 * u + 1], d[4 * u + 0], 0x07030602u);
            u32 t2 = __builtin_amdgcn_perm(d[4 * u + 3], d[4 * u + 2], 0x05010400u);
            u32 t3 = __builtin_amdgcn_perm(d[4 * u + 3], d[4 * u + 2], 0x07030602u);
            q[0][u] = __builtin_amdgcn_perm(t2, t0, 0x05040100u);
            q[1][u] = __builtin_amdgcn_perm(t2, t0, 0x07060302u);
            q[2][u] = __builtin_amdgcn_perm(t3, t1, 0x05040100u);
            q[3][u] = __builtin_amdgcn_perm(t3, t1, 0x07060302u);
        }
        __amdgpu_buffer_rsrc_t rd = RSRC(myws + CTRL_BYTES + (call % NBUF) * BUF_BYTES + (gi >> 1) * STR_PITCH
                                         + (gi & 1) * 131072, 131072);
        const u32 so = (u32)(lane >> 3) * 16384 + (G >> 1) * 1024 + ((G & 1) * 32 + cq) * 16;
#pragma unroll
        for (int c = 0; c < 4; ++c) ST16(q[c], rd, so + c * 128, AUX_SYS);
    } else {
        call = __builtin_amdgcn_readfirstlane(ld_sys(ctr)) + 1u;
        if (wv == 7) elect_invalidate(myws, call, xcc, lane);
    }
    const float scale = a.sx[0] * a.sw[0];
    wait_vm0();
    wg_barrier();
    int* okw = (int*)&red[2 * 8 * 2048];
    const u32 bufo = CTRL_BYTES + (call % NBUF) * BUF_BYTES;
    if (wv == 0) {
        if (lane == 0) {
            st_sys(myws + bufo + (gi >> 1) * STR_PITCH + STR_BYTES + ((gi & 1) * 8 + xg) * 4, call);
            st_sys(ctr, call);
            okw[3] = 0;
        }
    } else if (wv == 7) {
        const int l = lane & 31;
        int src = l & 7;
        if (src == rank) src = (rank + 1) & 7;
        const unsigned char* f = myws + OFF_YFLAG + (gi * 16 + (l & 8) + src) * 128;
        if (!ownrows && l >= 16) f = myws + OFF_XFLAG + ((gi - 8) * 16 + (l & 8) + src) * 128;
        __builtin_amdgcn_s_sleep(64);
        bool ok = poll_flags<8>(f, tmo, call);
        if (lane == 0) *okw = ok ? 1 : 0;
    }
    wg_barrier();
    int ok = __builtin_amdgcn_readfirstlane(*okw);

    const int fq = 4 * (1 - g) + ((gi - 8) >> 1);
    const bool ownk = (wv == rank);
    const int row0 = ownrows ? rank * 512 + gi * 64 : fq * 512 + gi4 * 128 + (gi & 1) * 64;
    __amdgpu_buffer_rsrc_t rx = RSRC(ownk ? xin + (size_t)row0 * KS
                                          : myws + bufo + O_XB + (wv * 32 + gi * 2) * 16384, ok ? 32768 : 0);
    const u32 xo = ownk ? (u32)(lane & 31) * KS + (lane >> 5) * 16 : lo;
    const u32 xstep = ownk ? 32u : 1024u;
    u32x4 xf[32];
    if (wv == 7 && ok) {
        const unsigned char* f = myws + OFF_INVD + xcc * 128;
        if (lane < 8 && lane != rank) f = myws + OFF_RFLAG + (xg * 8 + lane) * 128;
        if (lane >= 8 && lane < 24) f = myws + bufo + xg * STR_PITCH + STR_BYTES + (lane - 8) * 4;
        bool o2 = poll_flags<8>(f, tmo, call);
        if (o2) o2 = __hip_atomic_load((u32*)tmo, __ATOMIC_RELAXED, __HIP_MEMORY_SCOPE_AGENT) != call;
        if (lane == 0) okw[1] = o2 ? 1 : 0;
    }
#pragma unroll
    for (int i = 0; i < 32; ++i)
        xf[i] = __builtin_amdgcn_raw_buffer_load_b128(rx, xo + (i >> 4) * 16384, (i & 15) * xstep, AUX_AGT);
    wg_barrier();
    if (ok) ok = __builtin_amdgcn_readfirstlane(okw[1]);

    const int rrow = wv * 8 + (lane >> 3);
    const int re = lane & 7;
    __amdgpu_buffer_rsrc_t ro = ownrows ? RSRC(a.out + (size_t)(gi * 64) * NDIM + xg * 512, 63 * NDIM * 4 + 2048)
                                        : RSRC(myws + bufo + O_OS + ((gi - 8) * 8 + xg) * 65536, 65536);
    const u32 out_off = ownrows ? (u32)rrow * NDIM * 4 + re * 16 : (u32)wv * 1024 + lo;
    if (!ok) {
        if (ownrows)
            for (int t = 0; t < 9; ++t)
                ST16(__builtin_bit_cast(u32x4, nan4), ro, out_off + (t < 7 ? 7 * g + t : 7 + t) * 128, 0);
        return;
    }

    auto tile_base = [&](int t) -> const unsigned char* {
        const bool fwd = (wv >> 2) != g && t >= 5;
        const int src = fwd ? 4 * g + ((t - 5 + wv + xg) & 3) : wv;
        const u32 off = fwd ? O_FW + ((wv & 3) * 8 + xg) * 16384 : xg * STR_PITCH + (t < 7 ? 7 * g + t : 7 + t) * 16384;
        return a.rot[(src - rank - 1) & 7] + bufo + off;
    };
    unsigned char* oflag = a.rot[(fq - rank - 1) & 7] + OFF_OFLAG + ((gi4 * 2 + (gi & 1)) * 8 + xg) * 128;

    const unsigned char* cur = tile_base(0);
    u32x4 wf[8];
    {
        __amdgpu_buffer_rsrc_t r0 = RSRC(cur, 16384);
#pragma unroll
        for (int k = 0; k < 2 * C_PAIRS; ++k) wf[k] = LD16(r0, lo + k * 1024, 0);
    }

    const int m = lane & 31, hh = lane >> 5;
    const u32 wr_base = wv * 8192 + m * 128;
    const u32 rd_base = rrow * 128 + ((re ^ (rrow & 7)) * 16);
    const int one = 0x7F7F7F7F;

#pragma unroll 1
    for (int j = 0; j < nt; ++j) {
        const unsigned char* nxt = tile_base(j + 1 < 9 ? j + 1 : 8);
        __amdgpu_buffer_rsrc_t rc = RSRC(cur, ok ? 16384 : 0);
        __amdgpu_buffer_rsrc_t rn = RSRC(nxt, (ok && j + 1 < nt) ? 16384 : 0);
        f32x16 acc0 = {0.f, 0.f, 0.f, 0.f, 0.f, 0.f, 0.f, 0.f, 0.f, 0.f, 0.f, 0.f, 0.f, 0.f, 0.f, 0.f};
        f32x16 acc1 = acc0;
#pragma unroll
        for (int t = 0; t < 8; ++t) {
            const int s = (t & 3) * 2, sn = ((t + C_PAIRS) & 3) * 2;
            asm volatile("" :: "v"(wf[s]), "v"(wf[s + 1]) : "memory");
            if (t == C_PAIRS - 1 && !ownrows && j > 0) {
                asm volatile("s_waitcnt vmcnt(%0)" :: "i"(2 * (C_PAIRS - 1)) : "memory");
                if (lane == 0 &&
                    __hip_atomic_fetch_add(&okw[3], 1, __ATOMIC_RELAXED, __HIP_MEMORY_SCOPE_WORKGROUP) == 8 * j - 1)
                    st_sys(oflag + (j - 1) * 4, call);
            }
            const int p = t + C_PAIRS;
            wf[sn]     = LD16(p < 8 ? rc : rn, (p & 7) * 2048 + lo, 0);
            wf[sn + 1] = LD16(p < 8 ? rc : rn, (p & 7) * 2048 + 1024 + lo, 0);
            __builtin_amdgcn_sched_barrier(0);
            i32x8 A = CAT8(wf[s], wf[s + 1]);
            acc0 = __builtin_amdgcn_mfma_scale_f32_32x32x64_f8f6f4(A, CAT8(xf[2 * t], xf[2 * t + 1]), acc0,
                                                                   1, 1, 0, one, 0, one);
            acc1 = __builtin_amdgcn_mfma_scale_f32_32x32x64_f8f6f4(A, CAT8(xf[16 + 2 * t], xf[17 + 2 * t]), acc1,
                                                                   1, 1, 0, one, 0, one);
            asm volatile("" : "+v"(acc0), "+v"(acc1));
        }
        cur = nxt;
        char* slot = (char*)red + (j & 1) * 65536;
#pragma unroll
        for (int b = 0; b < 4; ++b) {
            f32x4 p = {acc0[b], acc0[4 + b], acc0[8 + b], acc0[12 + b]};
            *(f32x4*)(slot + wr_base + (((hh * 4 + b) ^ (m & 7)) * 16)) = p;
        }
#pragma unroll
        for (int b = 0; b < 4; ++b) {
            f32x4 p = {acc1[b], acc1[4 + b], acc1[8 + b], acc1[12 + b]};
            *(f32x4*)(slot + wr_base + 4096 + (((hh * 4 + b) ^ (m & 7)) * 16)) = p;
        }
        if (j == 3 && wv == 7) {
            bool o3 = poll_flags<2>(myws + OFF_FFLAG + ((((lane >> 3) & 3) * 8 + xg) * 8 + (lane & 7)) * 128, tmo, call);
            if (lane == 0) okw[2] = o3 ? 1 : 0;
        }
        wg_barrier();
        if (j == 3) ok = __builtin_amdgcn_readfirstlane(okw[2]);
        f32x4 sv[8];
#pragma unroll
        for (int w = 0; w < 8; ++w) sv[w] = *(const f32x4*)(slot + rd_base + w * 8192);
        f32x4 s = ((sv[0] + sv[1]) + (sv[2] + sv[3])) + ((sv[4] + sv[5]) + (sv[6] + sv[7]));
        f32x4 r;
#pragma unroll
        for (int c = 0; c < 4; ++c) {
            float y = s[c] * scale;
            float z = fminf(fmaxf(y, -60.f), 60.f);
            r[c] = y * __builtin_amdgcn_rcpf(1.f + __expf(-z));
        }
        if (!ok) r = nan4;
        const u32 toff = ownrows ? (j < 7 ? 7 * g + j : 7 + j) * 128 : j * 8192;
        __builtin_amdgcn_raw_buffer_store_b128(__builtin_bit_cast(u32x4, r), ro, out_off, toff, AUX_SYS);
    }
    if (!ownrows) {
        wait_vm0();
        wg_barrier();
        if (ownk && lane == 0) st_sys(oflag + 6 * 4, call);
    }
}

extern "C" void kernel_launch(int rank, int nranks, ncclComm_t comm,
                              hipStream_t stream,
                              void* const* d_in, const int* in_sizes, int n_in,
                              void* d_out, int out_size,
                              const int* in_dtypes, int out_dtype,
                              void* const* d_out_all,
                              void* const* d_ws_all, size_t ws_bytes) {
    (void)comm; (void)in_sizes; (void)in_dtypes; (void)out_dtype; (void)d_out_all;
    if (nranks != NR || n_in < 4 || (unsigned)rank >= (unsigned)NR || ws_bytes < WS_NEED ||
        out_size != 512 * NDIM) {
        fprintf(stderr, "kernel_launch: unexpected problem shape\n");
        return;
    }
    Args a;
    a.x = (const unsigned char*)d_in[0];
    a.w = (const unsigned char*)d_in[1];
    a.sx = (const float*)d_in[2];
    a.sw = (const float*)d_in[3];
    a.out = (float*)d_out;
    for (int d = 0; d < NR; ++d) a.rot[d] = (unsigned char*)d_ws_all[(rank + 1 + d) & 7];
    a.myws = (unsigned char*)d_ws_all[rank];
    a.pad = 0;
    a.rank = rank;
    hipLaunchKernelGGL(fused_gemm_rs, dim3(NWG), dim3(512), 0, stream, a);
    HIP_CHECK(hipGetLastError());
}
```

```cpp
#include <hip/hip_runtime.h>
#include <rccl/rccl.h>
#include <cstdio>
#include <cstdint>

#define HIP_CHECK(expr) do { hipError_t _e = (expr); if (_e != hipSuccess) { \
    fprintf(stderr, "HIP error %s at %s:%d\n", hipGetErrorString(_e), __FILE__, __LINE__); \
    return; } } while (0)

typedef unsigned int u32;
typedef unsigned long long u64;
typedef u32   u32x4  __attribute__((ext_vector_type(4)));
typedef int   i32x4  __attribute__((ext_vector_type(4)));
typedef int   i32x8  __attribute__((ext_vector_type(8)));
typedef float f32x4  __attribute__((ext_vector_type(4)));
typedef float f32x16 __attribute__((ext_vector_type(16)));

constexpr int NR   = 8;
constexpr int NGM  = 16;
constexpr int NAX  = 16;
constexpr int NWG  = 8 * (NGM + NAX);
constexpr int NDIM = 4096;
constexpr int KS   = 512;
constexpr u64 DEADLINE = 100000000ull;
constexpr int WIN     = 4;
constexpr int C_PAIRS = 3;
constexpr int X_PUSH  = 5;
constexpr int X_WIN   = 2;
constexpr int NX      = 16 - X_PUSH;
constexpr u32 NBUF    = 2;

constexpr u32 OFF_CTR   = 0;
constexpr u32 OFF_TMO   = 32768;
constexpr u32 OFF_ELECT = 32768 + 1024;
constexpr u32 OFF_INVD  = 32768 + 4096;
constexpr u32 OFF_XFLAG = 65536;
constexpr u32 OFF_RFLAG = 65536 + 16384;
constexpr u32 OFF_OFLAG = 65536 + 24576;
constexpr u32 OFF_YFLAG = 98304;
constexpr u32 OFF_ZFLAG = 131072;
constexpr u32 OFF_FFLAG = 163840;
constexpr u32 CTRL_BYTES = 1u << 20;
constexpr u32 STR_BYTES = 262144;
constexpr u32 STR_PITCH = STR_BYTES + 4096;
constexpr u32 O_XB = 8 * STR_PITCH;
constexpr u32 O_XS = O_XB + (4u << 20);
constexpr u32 O_FW = O_XS + (2u << 20);
constexpr u32 O_OS = O_FW + (512u << 10);
constexpr u32 BUF_BYTES = O_OS + (4u << 20);
constexpr u32 WS_NEED   = CTRL_BYTES + NBUF * BUF_BYTES;

struct Args {
    const unsigned char* x;
    const unsigned char* w;
    unsigned char* myws;
    int rank;
    int pad;
    unsigned char* rot[NR];
    const float* sx;
    const float* sw;
    float* out;
};

#define LIKELY(c) __builtin_expect(!!(c), 1)
#define RSRC(p, bytes) __builtin_amdgcn_make_buffer_rsrc((void*)(p), 0, (int)(bytes), 0x00020000)
#define AUX_SYS 17
#define AUX_AGT 16
#define CAT8(lo4, hi4) __builtin_shufflevector(__builtin_bit_cast(i32x4, lo4), __builtin_bit_cast(i32x4, hi4), \
                                               0, 1, 2, 3, 4, 5, 6, 7)

__device__ __forceinline__ u32 ld_sys(const unsigned char* p) {
    return __hip_atomic_load((const u32*)p, __ATOMIC_RELAXED, __HIP_MEMORY_SCOPE_SYSTEM);
}
__device__ __forceinline__ void st_sys(unsigned char* p, u32 v) {
    __hip_atomic_store((u32*)p, v, __ATOMIC_RELAXED, __HIP_MEMORY_SCOPE_SYSTEM);
}
__device__ __forceinline__ u32 xcc_id() {
    return (u32)__builtin_amdgcn_s_getreg((3 << 11) | 20) & 0xFu;
}
__device__ __forceinline__ void wait_vm0() { asm volatile("s_waitcnt vmcnt(0)" ::: "memory"); }
__device__ __forceinline__ void wg_barrier() {
    asm volatile("s_waitcnt lgkmcnt(0)" ::: "memory");
    __builtin_amdgcn_s_barrier();
    asm volatile("" ::: "memory");
}

#define LD16(r, off, aux) __builtin_amdgcn_raw_buffer_load_b128(r, off, 0, aux)
#define ST16(v, r, off, aux) __builtin_amdgcn_raw_buffer_store_b128(v, r, off, 0, aux)

template <int SLEEP>
__device__ __forceinline__ bool poll_flags(const unsigned char* p, unsigned char* tmo, u32 call) {
    const u64 t0 = __builtin_amdgcn_s_memrealtime();
    for (u32 it = 1;; ++it) {
        u32 v = ld_sys(p);
        if (__all(v >= call)) { asm volatile("" ::: "memory"); return true; }
        if ((it & 15u) == 0u) {
            u32 t = __hip_atomic_load((u32*)tmo, __ATOMIC_RELAXED, __HIP_MEMORY_SCOPE_AGENT);
            if (t == call || __builtin_amdgcn_s_memrealtime() - t0 > DEADLINE) {
                __hip_atomic_store((u32*)tmo, call, __ATOMIC_RELAXED, __HIP_MEMORY_SCOPE_AGENT);
                return false;
            }
        }
        __builtin_amdgcn_s_sleep(SLEEP);
    }
}

__device__ __forceinline__ void elect_invalidate(unsigned char* myws, u32 call, u32 xcc, int lane) {
    u32 old = 0;
    if (lane == 0)
        old = __hip_atomic_fetch_max((u32*)(myws + OFF_ELECT + xcc * 128), call,
                                     __ATOMIC_RELAXED, __HIP_MEMORY_SCOPE_AGENT);
    old = __builtin_amdgcn_readfirstlane(old);
    if (old < call) {
        __builtin_amdgcn_fence(__ATOMIC_ACQUIRE, "");
        wait_vm0();
        if (lane == 0) st_sys(myws + OFF_INVD + xcc * 128, call);
    }
}

template <int W>
__device__ __forceinline__ void touch_chunks(const unsigned char* base, u32 off, u32 step, int n) {
    u32x4 dsc;
    dsc.x = __builtin_amdgcn_readfirstlane((u32)(size_t)base);
    dsc.y = __builtin_amdgcn_readfirstlane((u32)((size_t)base >> 32) & 0xffffu);
    dsc.z = STR_BYTES;
    dsc.w = 0x00020000u;
#pragma unroll 2
    for (int c = 0; c < n; ++c) {
        u32 keep;
        asm volatile("s_mov_b32 %0, m0\n\ts_mov_b32 m0, 0\n\ts_waitcnt vmcnt(%3)\n\ts_nop 2\n\t"
                     "buffer_load_dwordx4 %1, %2, 0 offen lds\n\ts_mov_b32 m0, %0"
                     : "=&s"(keep) : "v"(off), "s"(dsc), "i"(W - 1) : "memory");
        off += step;
    }
}

__global__ void __launch_bounds__(512, 2) fused_gemm_rs(Args a) {
    __shared__ __attribute__((aligned(16))) float red[2 * 8 * 2048 + 4];
    const int lane = threadIdx.x & 63;
    const int wv = __builtin_amdgcn_readfirstlane(threadIdx.x >> 6);
    const int xg = blockIdx.x & 7;
    const int idx = ((blockIdx.x >> 3) + 30) & 31;
    const int bid = idx * 8 + xg;
    const int rank = a.rank;
    unsigned char* myws = a.myws;
    const unsigned char* xin = a.x;
    const unsigned char* win = a.w;
    unsigned char* pws = a.rot[((idx - NGM) >> 1) & 7];
    asm volatile("" :: "s"(rank), "s"(myws), "s"(xin), "s"(win), "s"(pws));

    unsigned char* ctr = myws + OFF_CTR + bid * 64;
    unsigned char* tmo = myws + OFF_TMO;
    const u32 lo = lane * 16;
    const int g = rank >> 2, gi4 = rank & 3;
    const float qnan = __builtin_nanf("");
    const f32x4 nan4 = {qnan, qnan, qnan, qnan};

    const int hi = idx - NGM;
    const int h = hi & 1;
    const int peer = (rank + 1 + (hi >> 1)) & 7;
    const int blk = h * 8 + xg;
    unsigned char* myctr = ctr + (wv & 3) * 4;
    {
        if (LIKELY(idx >= NGM && (wv == 3 || (wv == 0 && peer != rank)))) {
            const int bo_ = 4 * (1 - (peer >> 2)) + (blk >> 2), bb_ = 4 * (peer & 3) + (blk & 3);
            __amdgpu_buffer_rsrc_t rx = RSRC(xin + (size_t)(bo_ * 512 + bb_ * 32) * KS, 32 * KS);
            const u32 o = (u32)(lane & 31) * KS + (lane >> 5) * 16;
            const u32 cv = ld_sys(myctr);
            __builtin_amdgcn_sched_barrier(0);
            if (wv == 0) {
                u32x4 v[X_PUSH];
#pragma unroll
                for (int c = 0; c < X_PUSH; ++c) v[c] = LD16(rx, o + c * 32, 0);
                __builtin_amdgcn_sched_barrier(0);
                const u32 call = __builtin_amdgcn_readfirstlane(cv) + 1u;
                __amdgpu_buffer_rsrc_t rd = RSRC(pws + CTRL_BYTES + (call % NBUF) * BUF_BYTES + O_XB
                                                 + (rank * 32 + 16 + blk) * 16384, 16384);
#pragma unroll
                for (int c = 0; c < X_PUSH; ++c) {
                    if (c >= X_WIN) asm volatile("s_waitcnt vmcnt(%0)" :: "i"(X_WIN - 1) : "memory");
                    ST16(v[c], rd, c * 1024 + lo, AUX_SYS);
                }
                wait_vm0();
                if (lane == 0) {
                    st_sys(pws + OFF_XFLAG + (blk * 8 + rank) * 128, call);
                    st_sys(myctr, call);
                }
            } else {
                u32x4 v[16];
#pragma unroll
                for (int c = 0; c < 16; ++c) {
                    v[c] = LD16(rx, o + ((X_PUSH + c) & 15) * 32, 0);
                    __builtin_amdgcn_sched_barrier(0);
                }
                __builtin_amdgcn_sched_barrier(0);
                const u32 call = __builtin_amdgcn_readfirstlane(cv) + 1u;
                __amdgpu_buffer_rsrc_t rl = RSRC(myws + CTRL_BYTES + (call % NBUF) * BUF_BYTES + O_XS
                                                 + (bo_ * 16 + bb_) * 16384, 16384);
                unsigned char* ows = a.rot[(bo_ - rank - 1) & 7];
#pragma unroll
                for (int c = 0; c < 16; ++c) {
                    ST16(v[c], rl, ((X_PUSH + c) & 15) * 1024 + lo, AUX_SYS);
                    __builtin_amdgcn_sched_barrier(0);
                }
                asm volatile("s_waitcnt vmcnt(%0)" :: "i"(X_PUSH) : "memory");
                if (lane == 0 && peer != rank) st_sys(pws + OFF_ZFLAG + ((16 + blk) * 8 + rank) * 128, call);
                asm volatile("s_waitcnt vmcnt(1)" ::: "memory");
                if (peer == rank) wait_vm0();
                if (lane == 0) {
                    if (bo_ != rank) st_sys(ows + OFF_ZFLAG + (bb_ * 8 + rank) * 128, call);
                    st_sys(myctr, call);
                }
            }
            return;
        }
    }
    if (idx >= NGM) {
        if (peer == rank || wv == 0 || wv > 2) return;
        if (wv == 2) __builtin_amdgcn_s_sleep(64);
        const u32 call = __builtin_amdgcn_readfirstlane(ld_sys(myctr)) + 1u;
        if (lane == 0) st_sys(myctr, call);
        if (wv == 2) {
            elect_invalidate(myws, call, xcc_id(), lane);
            return;
        }
        const bool cross = (peer >> 2) != g;
        const u32 bufo = CTRL_BYTES + (call % NBUF) * BUF_BYTES;
        const unsigned char* wbase = pws + bufo + xg * STR_PITCH;
        const unsigned char* fw = wbase + STR_BYTES + (lane & 15) * 4;
        if (lane >= 16) fw = myws + OFF_INVD + xcc_id() * 128;
        if (lane >= 24) fw = myws + OFF_ZFLAG + (blk * 8 + peer) * 128;
        const int fo = 4 * (1 - g) + (blk >> 2), fb = 4 * gi4 + (blk & 3);
        __amdgpu_buffer_rsrc_t rs1 = RSRC(pws + bufo + O_XS + (rank * 16 + blk) * 16384, 16384);
        __amdgpu_buffer_rsrc_t rs0 = RSRC(pws + bufo + O_XS + (fo * 16 + fb) * 16384, 16384);
        __amdgpu_buffer_rsrc_t rd1 = RSRC(myws + bufo + O_XB + (peer * 32 + blk) * 16384, 16384);
        __amdgpu_buffer_rsrc_t rd0 = RSRC(myws + bufo + O_XB + (peer * 32 + 16 + blk) * 16384, 16384);
        const int te = 5 + ((gi4 - peer - xg) & 3);
        const int ue = te < 7 ? 7 * g + te : 7 + te;
        __amdgpu_buffer_rsrc_t rs2 = RSRC(wbase + ue * 16384 + h * 8192, 8192);
        __amdgpu_buffer_rsrc_t rd2 = RSRC(myws + bufo + O_FW + ((peer & 3) * 8 + xg) * 16384 + h * 8192, 8192);
        __amdgpu_buffer_rsrc_t rdo = RSRC(a.out + (size_t)((peer & 3) * 128 + h * 64) * NDIM + (xg * 16 + 7 * (1 - g)) * 32,
                                          63 * NDIM * 4 + 7 * 128);
        const u32 oo = (u32)(lane >> 3) * NDIM * 4 + (lane & 7) * 16;
        u64 gm = (u64)(a.rot[(4 * g + (lane & 3) - rank - 1) & 7]
                       + OFF_FFLAG + ((((peer & 3) * 8 + xg) * 4 + gi4) * 2 + h) * 128);
        asm volatile("" : "+v"(gm));
        bool good = poll_flags<2>(myws + OFF_ZFLAG + ((16 + blk) * 8 + peer) * 128, tmo, call);
        if (good) {
            constexpr int N0 = NX, N1 = NX + 16, N2 = NX + 24;
#define ITEM_LD(i) ((i) < N0 ? LD16(rs0, lo + (X_PUSH + (i)) * 1024, AUX_SYS) : (i) < N1 ? LD16(rs1, lo + ((i) - N0) * 1024, AUX_SYS) \
                             : LD16(rs2, lo + ((i) - N1) * 1024, AUX_SYS))
#define ITEM_ST(v, i) do { if ((i) < N0) ST16(v, rd0, lo + (X_PUSH + (i)) * 1024, AUX_SYS); \
                           else if ((i) < N1) ST16(v, rd1, lo + ((i) - N0) * 1024, AUX_SYS); \
                           else ST16(v, rd2, lo + ((i) - N1) * 1024, AUX_SYS); } while (0)
            static_assert(8 % WIN == 0, "rounds align with tiles");
            static_assert(NX >= 2 * WIN, "the first x block covers the flag check");
            u32x4 v[WIN];
#pragma unroll
            for (int k = 0; k < WIN; ++k) v[k] = ITEM_LD(k);
            u32 fv = 0;
#pragma unroll
            for (int i = 0; i < N0 - WIN; ++i) {
                ITEM_ST(v[i % WIN], i);
                if (i == N0 - 2 * WIN) fv = ld_sys(fw);
                v[i % WIN] = ITEM_LD(i + WIN);
                __builtin_amdgcn_sched_barrier(0);
            }
            good = __all(fv >= call) || poll_flags<2>(fw, tmo, call);
            asm volatile("" ::: "memory");
            if (good) {
#define COPY_STEPS(from, to, NE) \
                _Pragma("unroll") \
                for (int i = (from); i < (to); ++i) { \
                    ITEM_ST(v[i % WIN], i); \
                    if (i + WIN < (NE)) v[i % WIN] = ITEM_LD(i + WIN); \
                    else touch_chunks<64>(wbase, 7 * g * 16384 + (2 * (i + WIN - (NE)) + h) * 1024 + lo, 0, 1); \
                    __builtin_amdgcn_sched_barrier(0); \
                    if (i == N0 + WIN - 1 || i == N1 + WIN - 1) { \
                          \
                        asm volatile("s_waitcnt vmcnt(%0)" :: "i"(2 * WIN) : "memory"); \
                        if (lane == 0) st_sys(myws + OFF_YFLAG + (((i < N1 ? 16 : 0) + blk) * 8 + peer) * 128, call); \
                    } \
                }
                COPY_STEPS(N0 - WIN, N1 - WIN, N2)
                if (cross) { COPY_STEPS(N1 - WIN, N2, N2) } else { COPY_STEPS(N1 - WIN, N1, N1) }
                if (lane == 0 && h == 0) st_sys(myws + OFF_RFLAG + (xg * 8 + peer) * 128, call);
                touch_chunks<WIN>(wbase, 7 * g * 16384 + (2 * WIN + h) * 1024 + lo, 2048, WIN);
                asm volatile("s_waitcnt vmcnt(%0)" :: "i"(WIN) : "memory");
                if (cross) {
                    if (lane < 4)
                        __hip_atomic_store((__attribute__((address_space(1))) u32*)gm, call, __ATOMIC_RELAXED, __HIP_MEMORY_SCOPE_SYSTEM);
                } else if (lane == 0) {
                    st_sys(myws + OFF_YFLAG + (blk * 8 + peer) * 128, call);
                }
                touch_chunks<WIN>(wbase, 7 * g * 16384 + (2 * (WIN + WIN) + h) * 1024 + lo, 2048,
                                     40 - WIN - 2 * WIN);
                const int sl = (peer - xg - ((lane >> 1) & 3)) & 3;
                const unsigned char* of = myws + OFF_OFLAG + (((peer & 3) * 2 + h) * 8 + xg) * 128;
                const unsigned char* nf = cross ? of + (lane & 7) * 4
                                                : myws + OFF_FFLAG + (((sl * 8 + xg) * 4 + (peer & 3)) * 2 + (lane & 1)) * 128;
                u32 fl;
                asm volatile("global_load_dword %0, %1, off sc0 sc1" : "=v"(fl) : "v"(nf) : "memory");
                touch_chunks<WIN + 1>(wbase, 7 * g * 16384 + (2 * (40 - WIN) + h) * 1024 + lo, 2048, WIN);
                asm volatile("s_waitcnt vmcnt(%1)" : "+v"(fl) : "i"(WIN) : "memory");
                if (!cross) {
                    if (h == 0) {
                        touch_chunks<WIN>(wbase, (7 * g + 5) * 16384 + lo, 1024, 32);
                        touch_chunks<WIN>(wbase, 14 * 16384 + lo, 1024, 32);
                    } else {
                        good = __all(fl >= call) || poll_flags<2>(nf, tmo, call);
                        if (good) {
                            for (int t = 0; t < 4; ++t)
                                touch_chunks<WIN>(pws + bufo + O_FW + ((((peer - xg - t) & 3) * 8 + xg) * 16384), lo, 1024, 16);
                        }
                    }
                    wait_vm0();
                    return;
                }
                const unsigned char* osrc = pws + bufo + O_OS + ((gi4 * 2 + h) * 8 + xg) * 65536;
                int done = 0;
                u32x4 w[WIN];
#pragma unroll
                for (int k = 0; k < WIN; ++k) w[k] = u32x4{0u, 0u, 0u, 0u};
                while (done < 7) {
                    int n = __builtin_ctzll(~(__ballot(fl >= call) & 0x7full));
                    if (n <= done) {
                        good = poll_flags<2>(of + done * 4, tmo, call);
                        if (!good) break;
                        n = done + 1;
                    }
                    asm volatile("" ::: "memory");
                    for (int c = done * 8 - WIN; c < n * 8; c += WIN) {
                        const int n2 = __builtin_ctzll(~(__ballot(fl >= call) & 0x7full));
                        n = n2 > n ? n2 : n;
                        asm volatile("" ::: "memory");
                        fl = ld_sys(nf);
                        __amdgpu_buffer_rsrc_t rso = RSRC(osrc, n * 8192);
#pragma unroll
                        for (int k = 0; k < WIN; ++k) {
                            const int cc = c + k;
                            const u32 so = cc >= done * 8 ? (cc & 7) * (8 * NDIM * 4) + (cc >> 3) * 128 : 0x7ff00000u;
                            ST16(w[k], rdo, oo + so, AUX_AGT);
                            w[k] = LD16(rso, (cc + WIN) * 1024 + lo, AUX_SYS);
                            __builtin_amdgcn_sched_barrier(0);
                        }
                    }
                    done = n;
                }
                wait_vm0();
            }
        }
        if (!good && cross) {
            for (int c = 0; c < 56; ++c)
                ST16(__builtin_bit_cast(u32x4, nan4), rdo, oo + (c & 7) * (8 * NDIM * 4) + (c >> 3) * 128, 0);
        }
        return;
    }

    const int gi = idx;
    const bool ownrows = gi < 8;
    const int nt = ownrows ? 9 : 7;
    const u32 xcc = xcc_id();

    if (wv == 4) {
        if (gi < 14 && !(gi & 1)) {
            u32 bo = CTRL_BYTES + (lane & 1) * BUF_BYTES;
            const unsigned char* t = pws + OFF_XFLAG + (xg * 8 + rank) * 128 + 64;
            if ((lane >> 1) == 1) t = pws + bo + O_XB + (rank * 32 + 16 + xg) * 16384;
            if ((lane >> 1) == 2) t = myws + bo + O_XS + (gi * 8 + xg) * 16384;
            u32 v = 0;
            if (lane < 6) v = ld_sys(t);
            asm volatile("" :: "v"(v));
            bo = CTRL_BYTES + ((__builtin_amdgcn_readfirstlane(ld_sys(ctr)) + 1u) % NBUF) * BUF_BYTES;
            t = pws + bo + O_XS + (lane & 1) * ((2u << 20) - 64);
            if (lane == 2) t = myws + bo + O_XB + (rank * 32 + xg) * 16384;
            if (lane == 3) t = pws + bo + xg * STR_PITCH + STR_BYTES + 256;
            if (lane == 4) t = pws + bo + xg * STR_PITCH - 256;
            if (lane == 5) t = win + (size_t)(xg * 64) * NDIM;
            if (lane < 6) v = ld_sys(t);
            asm volatile("" :: "v"(v));
        }
    } else {
        __builtin_amdgcn_s_sleep(64);
    }
    u32 call;
    if (LIKELY(wv < 4)) {
        const int cq = lane & 7;
        const int G = xg * 4 + wv;
        __amdgpu_buffer_rsrc_t rw = RSRC(win, KS * NDIM);
        const u32 o = (u32)(G * 16) * NDIM + gi * 256 + lane * 4;
        u32 d[16];
#pragma unroll
        for (int j = 0; j < 16; ++j) d[j] = __builtin_amdgcn_raw_buffer_load_b32(rw, o + j * NDIM, 0, 0);
        call = __builtin_amdgcn_readfirstlane(ld_sys(ctr)) + 1u;
        u32x4 q[4];
#pragma unroll
        for (int u = 0; u < 4; ++u) {
            u32 t0 = __builtin_amdgcn_perm(d[4 * u + 1], d[4 * u + 0], 0x05010400u);
            u32 t1 = __builtin_amdgcn_perm(d[4 * u + 1], d[4 * u + 0], 0x07030602u);
            u32 t2 = __builtin_amdgcn_perm(d[4 * u + 3], d[4 * u + 2], 0x05010400u);
            u32 t3 = __builtin_amdgcn_perm(d[4 * u + 3], d[4 * u + 2], 0x07030602u);
            q[0][u] = __builtin_amdgcn_perm(t2, t0, 0x05040100u);
            q[1][u] = __builtin_amdgcn_perm(t2, t0, 0x07060302u);
            q[2][u] = __builtin_amdgcn_perm(t3, t1, 0x05040100u);
            q[3][u] = __builtin_amdgcn_perm(t3, t1, 0x07060302u);
        }
        __amdgpu_buffer_rsrc_t rd = RSRC(myws + CTRL_BYTES + (call % NBUF) * BUF_BYTES + (gi >> 1) * STR_PITCH
                                         + (gi & 1) * 131072, 131072);
        const u32 so = (u32)(lane >> 3) * 16384 + (G >> 1) * 1024 + ((G & 1) * 32 + cq) * 16;
#pragma unroll
        for (int c = 0; c < 4; ++c) ST16(q[c], rd, so + c * 128, AUX_SYS);
    } else {
        call = __builtin_amdgcn_readfirstlane(ld_sys(ctr)) + 1u;
        if (wv == 7) elect_invalidate(myws, call, xcc, lane);
    }
    const float scale = a.sx[0] * a.sw[0];
    int* okw = (int*)&red[2 * 8 * 2048];
    if (threadIdx.x == 0) { okw[0] = 1; okw[3] = 0; }
    wait_vm0();
    wg_barrier();
    const u32 bufo = CTRL_BYTES + (call % NBUF) * BUF_BYTES;
    if (wv == 0 && lane == 0) {
        st_sys(myws + bufo + (gi >> 1) * STR_PITCH + STR_BYTES + ((gi & 1) * 8 + xg) * 4, call);
        st_sys(ctr, call);
    }

    const int fq = 4 * (1 - g) + ((gi - 8) >> 1);
    const bool ownk = (wv == rank);
    const int row0 = ownrows ? rank * 512 + gi * 64 : fq * 512 + gi4 * 128 + (gi & 1) * 64;
    bool okx = true;
    if (!ownk) {
        const unsigned char* f = myws + OFF_YFLAG + (gi * 16 + (lane & 2) * 4 + wv) * 128;
        if (!ownrows && (lane & 1)) f = myws + OFF_XFLAG + ((gi - 8) * 16 + (lane & 2) * 4 + wv) * 128;
        if (ownrows) __builtin_amdgcn_s_sleep(127);
        __builtin_amdgcn_s_sleep(64);
        okx = poll_flags<16>(f, tmo, call);
    }
    __amdgpu_buffer_rsrc_t rx = RSRC(ownk ? xin + (size_t)row0 * KS
                                          : myws + bufo + O_XB + (wv * 32 + gi * 2) * 16384, okx ? 32768 : 0);
    const u32 xo = ownk ? (u32)(lane & 31) * KS + (lane >> 5) * 16 : lo;
    const u32 xstep = ownk ? 32u : 1024u;
    u32x4 xf[32];
#pragma unroll
    for (int i = 0; i < 32; ++i)
        xf[i] = __builtin_amdgcn_raw_buffer_load_b128(rx, xo + (i >> 4) * 16384, (i & 15) * xstep, AUX_AGT);
    if (ownk) {
        const unsigned char* f = myws + OFF_INVD + xcc * 128;
        if (lane < 8 && lane != rank) f = myws + OFF_RFLAG + (xg * 8 + lane) * 128;
        if (lane >= 8 && lane < 24) f = myws + bufo + xg * STR_PITCH + STR_BYTES + (lane - 8) * 4;
        okx = poll_flags<16>(f, tmo, call);
        if (okx) okx = __hip_atomic_load((u32*)tmo, __ATOMIC_RELAXED, __HIP_MEMORY_SCOPE_AGENT) != call;
    }
    if (!okx && lane == 0) okw[0] = 0;
    wg_barrier();
    int ok = __builtin_amdgcn_readfirstlane(okw[0]);

    const int rrow = wv * 8 + (lane >> 3);
    const int re = lane & 7;
    __amdgpu_buffer_rsrc_t ro = ownrows ? RSRC(a.out + (size_t)(gi * 64) * NDIM + xg * 512, 63 * NDIM * 4 + 2048)
                                        : RSRC(myws + bufo + O_OS + ((gi - 8) * 8 + xg) * 65536, 65536);
    const u32 out_off = ownrows ? (u32)rrow * NDIM * 4 + re * 16 : (u32)wv * 1024 + lo;
    if (!ok) {
        if (ownrows)
            for (int t = 0; t < 9; ++t)
                ST16(__builtin_bit_cast(u32x4, nan4), ro, out_off + (t < 7 ? 7 * g + t : 7 + t) * 128, 0);
        return;
    }

    auto tile_base = [&](int t) -> const unsigned char* {
        const bool fwd = (wv >> 2) != g && t >= 5;
        const int src = fwd ? 4 * g + ((t - 5 + wv + xg) & 3) : wv;
        const u32 off = fwd ? O_FW + ((wv & 3) * 8 + xg) * 16384 : xg * STR_PITCH + (t < 7 ? 7 * g + t : 7 + t) * 16384;
        return a.rot[(src - rank - 1) & 7] + bufo + off;
    };
    unsigned char* oflag = a.rot[(fq - rank - 1) & 7] + OFF_OFLAG + ((gi4 * 2 + (gi & 1)) * 8 + xg) * 128;

    const unsigned char* cur = tile_base(0);
    u32x4 wf[8];
    {
        __amdgpu_buffer_rsrc_t r0 = RSRC(cur, 16384);
#pragma unroll
        for (int k = 0; k < 2 * C_PAIRS; ++k) wf[k] = LD16(r0, lo + k * 1024, 0);
    }

    const int m = lane & 31, hh = lane >> 5;
    const u32 wr_base = wv * 8192 + m * 128;
    const u32 rd_base = rrow * 128 + ((re ^ (rrow & 7)) * 16);
    const int one = 0x7F7F7F7F;

#pragma unroll 1
    for (int j = 0; j < nt; ++j) {
        const unsigned char* nxt = tile_base(j + 1 < 9 ? j + 1 : 8);
        __amdgpu_buffer_rsrc_t rc = RSRC(cur, ok ? 16384 : 0);
        __amdgpu_buffer_rsrc_t rn = RSRC(nxt, (ok && j + 1 < nt) ? 16384 : 0);
        f32x16 acc0 = {0.f, 0.f, 0.f, 0.f, 0.f, 0.f, 0.f, 0.f, 0.f, 0.f, 0.f, 0.f, 0.f, 0.f, 0.f, 0.f};
        f32x16 acc1 = acc0;
#pragma unroll
        for (int t = 0; t < 8; ++t) {
            const int s = (t & 3) * 2, sn = ((t + C_PAIRS) & 3) * 2;
            asm volatile("" :: "v"(wf[s]), "v"(wf[s + 1]) : "memory");
            if (t == C_PAIRS - 1 && !ownrows && j > 0) {
                asm volatile("s_waitcnt vmcnt(%0)" :: "i"(2 * (C_PAIRS - 1)) : "memory");
                if (lane == 0 &&
                    __hip_atomic_fetch_add(&okw[3], 1, __ATOMIC_RELAXED, __HIP_MEMORY_SCOPE_WORKGROUP) == 8 * j - 1)
                    st_sys(oflag + (j - 1) * 4, call);
            }
            const int p = t + C_PAIRS;
            wf[sn]     = LD16(p < 8 ? rc : rn, (p & 7) * 2048 + lo, 0);
            wf[sn + 1] = LD16(p < 8 ? rc : rn, (p & 7) * 2048 + 1024 + lo, 0);
            __builtin_amdgcn_sched_barrier(0);
            i32x8 A = CAT8(wf[s], wf[s + 1]);
            acc0 = __builtin_amdgcn_mfma_scale_f32_32x32x64_f8f6f4(A, CAT8(xf[2 * t], xf[2 * t + 1]), acc0,
                                                                   1, 1, 0, one, 0, one);
            acc1 = __builtin_amdgcn_mfma_scale_f32_32x32x64_f8f6f4(A, CAT8(xf[16 + 2 * t], xf[17 + 2 * t]), acc1,
                                                                   1, 1, 0, one, 0, one);
            asm volatile("" : "+v"(acc0), "+v"(acc1));
        }
        cur = nxt;
        char* slot = (char*)red + (j & 1) * 65536;
#pragma unroll
        for (int b = 0; b < 4; ++b) {
            f32x4 p = {acc0[b], acc0[4 + b], acc0[8 + b], acc0[12 + b]};
            *(f32x4*)(slot + wr_base + (((hh * 4 + b) ^ (m & 7)) * 16)) = p;
        }
#pragma unroll
        for (int b = 0; b < 4; ++b) {
            f32x4 p = {acc1[b], acc1[4 + b], acc1[8 + b], acc1[12 + b]};
            *(f32x4*)(slot + wr_base + 4096 + (((hh * 4 + b) ^ (m & 7)) * 16)) = p;
        }
        if (j == 3 && wv == 7) {
            bool o3 = poll_flags<2>(myws + OFF_FFLAG + ((((lane >> 3) & 3) * 8 + xg) * 8 + (lane & 7)) * 128, tmo, call);
            if (lane == 0) okw[2] = o3 ? 1 : 0;
        }
        wg_barrier();
        if (j == 3) ok = __builtin_amdgcn_readfirstlane(okw[2]);
        f32x4 sv[8];
#pragma unroll
        for (int w = 0; w < 8; ++w) sv[w] = *(const f32x4*)(slot + rd_base + w * 8192);
        f32x4 s = ((sv[0] + sv[1]) + (sv[2] + sv[3])) + ((sv[4] + sv[5]) + (sv[6] + sv[7]));
        f32x4 r;
#pragma unroll
        for (int c = 0; c < 4; ++c) {
            float y = s[c] * scale;
            float z = fminf(fmaxf(y, -60.f), 60.f);
            r[c] = y * __builtin_amdgcn_rcpf(1.f + __expf(-z));
        }
        if (!ok) r = nan4;
        const u32 toff = ownrows ? (j < 7 ? 7 * g + j : 7 + j) * 128 : j * 8192;
        __builtin_amdgcn_raw_buffer_store_b128(__builtin_bit_cast(u32x4, r), ro, out_off, toff, AUX_SYS);
    }
    if (!ownrows) {
        wait_vm0();
        wg_barrier();
        if (ownk && lane == 0) st_sys(oflag + 6 * 4, call);
    }
}

extern "C" void kernel_launch(int rank, int nranks, ncclComm_t comm,
                              hipStream_t stream,
                              void* const* d_in, const int* in_sizes, int n_in,
                              void* d_out, int out_size,
                              const int* in_dtypes, int out_dtype,
                              void* const* d_out_all,
                              void* const* d_ws_all, size_t ws_bytes) {
    (void)comm; (void)in_sizes; (void)in_dtypes; (void)out_dtype; (void)d_out_all;
    if (nranks != NR || n_in < 4 || (unsigned)rank >= (unsigned)NR || ws_bytes < WS_NEED ||
        out_size != 512 * NDIM) {
        fprintf(stderr, "kernel_launch: unexpected problem shape\n");
        return;
    }
    Args a;
    a.x = (const unsigned char*)d_in[0];
    a.w = (const unsigned char*)d_in[1];
    a.sx = (const float*)d_in[2];
    a.sw = (const float*)d_in[3];
    a.out = (float*)d_out;
    for (int d = 0; d < NR; ++d) a.rot[d] = (unsigned char*)d_ws_all[(rank + 1 + d) & 7];
    a.myws = (unsigned char*)d_ws_all[rank];
    a.pad = 0;
    a.rank = rank;
    hipLaunchKernelGGL(fused_gemm_rs, dim3(NWG), dim3(512), 0, stream, a);
    HIP_CHECK(hipGetLastError());
}
```
